```python
import numpy as np

kernel_name = "gpu_ag_gemm_m4096_k4096_n4096_bf16_q8"


ABSMAX_RTOL = 0.02

COMPUTE_DTYPE = "bfloat16"
EPILOGUE = "quant_int8"
IN_QUANT = None
OUT_QUANT = "int8"
IN_DTYPES = {"x": "bfloat16", "w_mat": "bfloat16"}
WORLD_SIZE = 8

def _to_bf16(x):
    u = np.ascontiguousarray(x, dtype=np.float32).view(np.uint32)
    return ((u + 0x7FFF + ((u >> 16) & 1)) & 0xFFFF0000).view(np.float32)

def setup_inputs():

    x = _to_bf16(np.random.randn(4096, 4096).astype(np.float32))
    w_mat = _to_bf16((np.random.randn(4096, 4096) / np.sqrt(4096)).astype(np.float32))
    return {"x": x, "w_mat": w_mat}

def _quant_dequant(y, amax):
    scale = amax / 127.0
    q = np.clip(np.round(y / scale), -127.0, 127.0)
    return (q * scale).astype(np.float32)
def _epilogue(y):
    return _quant_dequant(y, np.abs(y).max())

def reference(x, w_mat):
    y = (x @ w_mat).astype(np.float32)
    return _epilogue(y)
```

```cpp
#include <hip/hip_runtime.h>
#include <rccl/rccl.h>
#include <cstdio>
#include <cstdint>

#define HIP_CHECK(expr) do { hipError_t _e = (expr); if (_e != hipSuccess) { \
    fprintf(stderr, "HIP error %s at %s:%d\n", hipGetErrorString(_e), __FILE__, __LINE__); \
    return; } } while (0)

typedef __attribute__((ext_vector_type(4))) unsigned u32x4;
typedef __attribute__((ext_vector_type(4))) float f32x4;
typedef __attribute__((ext_vector_type(16))) float f32x16;
typedef __attribute__((ext_vector_type(4))) short s16x4;
typedef __attribute__((ext_vector_type(8))) short s16x8;
typedef __attribute__((ext_vector_type(8))) __bf16 bf16x8;
#define LAS __attribute__((address_space(3)))
#define GAS __attribute__((address_space(1)))
typedef GAS unsigned gu32;
typedef GAS unsigned long long gu64;

constexpr int NR = 8;
constexpr int NSTEP = 64;
constexpr int N_WPUSH = 16;
constexpr int N_XCOPY = 64;
constexpr int N_PROD = N_WPUSH + N_XCOPY;
constexpr int N_GEMM = 128;
constexpr int GRID = N_PROD + N_GEMM;
constexpr int MARGIN = 8;
constexpr int WWIN = 3;
constexpr unsigned long long DEADLINE = 100000000ull;

constexpr unsigned OFF_CTR = 0;
constexpr unsigned OFF_TMO = 0x8000;
constexpr unsigned OFF_WFLAG = 0x8080;
constexpr unsigned OFF_XFLAG = 0x9000;
constexpr unsigned OFF_DONE = 0xA000;
constexpr unsigned OFF_AMAX = 0xC000;
constexpr unsigned OFF_DBG = 0x10000;
constexpr unsigned UNIT_BYTES = 64u * 8192u;
constexpr unsigned OFF_XC = 1u << 20;
constexpr unsigned XC_BYTES = 8u * UNIT_BYTES;
constexpr unsigned OFF_XP = OFF_XC + XC_BYTES;
constexpr unsigned XP_UNIT = 4u * 8192u;
constexpr unsigned XP_BYTES = 32u * XP_UNIT;
constexpr unsigned OFF_WS = OFF_XP + XP_BYTES;
constexpr unsigned WQ_BYTES = 1u << 20;
constexpr unsigned WS_NEEDED = OFF_WS + 4u * WQ_BYTES;
constexpr unsigned SHARD_BYTES = 4u << 20;
constexpr unsigned OUT_BYTES = 4096u * 512u;
static_assert(GRID * 128 <= (int)OFF_TMO, "counter area");

constexpr unsigned SLOT = 32768, NSLOT = 4;
constexpr unsigned L_CTRL = SLOT * NSLOT;
constexpr unsigned L_FULL = L_CTRL, L_FREE = L_CTRL + 16, L_AVAIL = L_CTRL + 32, L_ABORT = L_CTRL + 44, L_CALL = L_CTRL + 48,
                   L_WMAX = L_CTRL + 64, L_GMAX = L_CTRL + 96;
constexpr unsigned LDS_BYTES = L_CTRL + 128;

struct Params {
    const char* x;
    const char* w;
    char* ws[NR];
    char* out[NR];
    int rank;
};

__device__ __forceinline__ bool pushed_step(int s) { return (s & 15) == 8; }

__device__ __forceinline__ unsigned rfl(unsigned v) { return __builtin_amdgcn_readfirstlane(v); }
__device__ __forceinline__ u32x4 make_srd(const void* p, unsigned bytes) {
    const unsigned long long a = (unsigned long long)p;
    u32x4 r;
    r.x = rfl((unsigned)a); r.y = rfl((unsigned)(a >> 32) & 0xffffu); r.z = rfl(bytes); r.w = 0x00020000u;
    return r;
}
__device__ __forceinline__ void dma16(unsigned lds, unsigned voff, u32x4 srd, unsigned soff) {
    asm volatile("s_mov_b32 m0, %0\n\ts_nop 0\n\tbuffer_load_dwordx4 %1, %2, %3 offen lds"
                 :: "s"(lds), "v"(voff), "s"(srd), "s"(soff) : "memory");
}
typedef __amdgpu_buffer_rsrc_t rsrc_t;
__device__ __forceinline__ rsrc_t make_rsrc(const void* p, unsigned bytes) {
    const unsigned long long a = (unsigned long long)p;
    const unsigned long long u = ((unsigned long long)rfl((unsigned)(a >> 32)) << 32) | rfl((unsigned)a);
    return __builtin_amdgcn_make_buffer_rsrc((void*)u, 0, rfl(bytes), 0x00020000);
}
__device__ __forceinline__ u32x4 bld16(unsigned voff, rsrc_t r, unsigned soff) { return __builtin_amdgcn_raw_buffer_load_b128(r, voff, soff, 0); }
__device__ __forceinline__ void bst16_sys(u32x4 v, unsigned voff, rsrc_t r, unsigned soff) {
    __builtin_amdgcn_raw_buffer_store_b128(v, r, voff, soff, 17);
}
#define WAIT_VM(n) asm volatile("s_waitcnt vmcnt(" #n ")" ::: "memory")
#define WAIT_LGKM0() asm volatile("s_waitcnt lgkmcnt(0)" ::: "memory")
#define BARRIER() do { asm volatile("s_waitcnt lgkmcnt(0)" ::: "memory"); __builtin_amdgcn_s_barrier(); asm volatile("" ::: "memory"); } while (0)
__device__ __forceinline__ unsigned long long now() { return __builtin_amdgcn_s_memrealtime(); }
__device__ __forceinline__ unsigned ld_sys(const void* p) { return __hip_atomic_load((gu32*)p, __ATOMIC_RELAXED, __HIP_MEMORY_SCOPE_SYSTEM); }
__device__ __forceinline__ void st_sys(void* p, unsigned v) { __hip_atomic_store((gu32*)p, v, __ATOMIC_RELAXED, __HIP_MEMORY_SCOPE_SYSTEM); }
__device__ __forceinline__ unsigned ld_agent(const void* p) { return __hip_atomic_load((gu32*)p, __ATOMIC_RELAXED, __HIP_MEMORY_SCOPE_AGENT); }
__device__ __forceinline__ void st_agent(void* p, unsigned v) { __hip_atomic_store((gu32*)p, v, __ATOMIC_RELAXED, __HIP_MEMORY_SCOPE_AGENT); }
__device__ __forceinline__ unsigned draw_ticket(char* ws, int b) {
    unsigned z = 0;
    asm volatile("" : "+v"(z));
    return __hip_atomic_fetch_add((gu32*)(ws + OFF_CTR + b * 128 + z), 1u, __ATOMIC_RELAXED, __HIP_MEMORY_SCOPE_AGENT);
}
__device__ __forceinline__ unsigned wave_max_u32(unsigned v) {
#pragma unroll
    for (int m = 32; m >= 1; m >>= 1) { const unsigned o = (unsigned)__shfl_xor((int)v, m); v = o > v ? o : v; }
    return v;
}

#define STAMP(i) do { } while (0)

__global__ void __launch_bounds__(512, 2) ag_gemm_q8(Params P) {
    __shared__ __attribute__((aligned(16))) char smem[LDS_BYTES];
    LAS char* const L = (LAS char*)smem;
    const int b = blockIdx.x;
    const int tid = threadIdx.x, lane = tid & 63;
    const int wave = (int)rfl(tid >> 6);
    const unsigned rank = (unsigned)P.rank & 7u;
    const unsigned grp = rank >> 2, me = rank & 3u, og = (grp ^ 1u) * 4u;
    char* const myws = P.ws[rank];

    if (b < N_WPUSH) {
        if (wave == 0) STAMP(0);
        const unsigned di = (((unsigned)b >> 1) + (((unsigned)b >> 3) << 1)) & 3u;
        const unsigned part = ((unsigned)b & 1u) | (((unsigned)b >> 3) << 1);
        const unsigned dst_rank = og + di;
        const unsigned p = 4u * part + ((unsigned)wave & 3u), so = (unsigned)wave >> 2;
        const rsrc_t ssrd = make_rsrc(P.w, SHARD_BYTES);
        const rsrc_t dsrd = make_rsrc(P.ws[dst_rank] + OFF_WS + me * WQ_BYTES, WQ_BYTES);
        const unsigned svoff = (lane >> 4) * 1024u + ((((unsigned)lane & 15u) ^ (((unsigned)lane >> 4) << 2) ^ (p & 3u)) << 4) + di * 256u;
        const unsigned dvoff = lane * 16u;
        char* const flag = P.ws[dst_rank] + OFF_WFLAG + (me * 4u + part) * 4u;
        u32x4 v[WWIN];
#pragma unroll
        for (int i = 0; i < WWIN; ++i) v[i] = bld16(svoff, ssrd, (64u * (2u * i + so) + 4u * p) * 1024u);
        unsigned tk = 0;
        if (tid == 0) tk = draw_ticket(myws, b);
        asm volatile("" ::: "memory");
#pragma unroll
        for (int i = 0; i < 32; ++i) {
            if (i >= WWIN) {
                if (i + WWIN <= 32) asm volatile("s_waitcnt vmcnt(%0)" :: "i"(2 * WWIN - 2) : "memory");
                else                asm volatile("s_waitcnt vmcnt(%0)" :: "i"(WWIN - 1) : "memory");
            }
            bst16_sys(v[i % WWIN], dvoff, dsrd, (2u * i + so) * 16384u + p * 1024u);
            if (i + WWIN < 32) v[i % WWIN] = bld16(svoff, ssrd, (64u * (2u * (i + WWIN) + so) + 4u * p) * 1024u);
            asm volatile("" ::: "memory");
            if (i >= WWIN) {
                __builtin_amdgcn_s_barrier();
                if (i == WWIN) asm volatile("" : "+v"(tk));
                if (tid == 0) st_sys(flag, tk * 64u + 64u + 2u * (unsigned)(i - WWIN + 1));
                asm volatile("" ::: "memory");
            }
        }
        WAIT_VM(0);
        __builtin_amdgcn_s_barrier();
        if (tid == 0) st_sys(flag, tk * 64u + 128u);
        if (wave == 0) STAMP(1);
        return;
    }
    if (b < N_PROD) {
        if (wave == 0) STAMP(0);
        const unsigned id = (unsigned)b - N_WPUSH, lu = id & 7u, j = id >> 3;
        const unsigned s = 8u * j + (unsigned)wave;
        const bool home = lu < 5u;
        const unsigned fr = (home ? grp * 4u : og) + ((unsigned)lane & 3u);
        char* fws = P.ws[fr];
        const rsrc_t ssrd = make_rsrc(P.x, SHARD_BYTES);
        const unsigned sv0 = (lane >> 3) * 8192u + ((((unsigned)lane & 7u) ^ ((unsigned)lane >> 4)) << 4);
        const unsigned sv1 = sv0 ^ 64u;
        const unsigned dvoff = lane * 16u;
        u32x4 v[8];
#pragma unroll
        for (int p = 0; p < 8; ++p) v[p] = bld16((p & 1) ? sv1 : sv0, ssrd, (lu * 64u + 8u * p) * 8192u + s * 128u);
        unsigned tk = 0;
        if (tid == 0) tk = draw_ticket(myws, b);
        if (home && pushed_step((int)s)) {
            const unsigned u = 5u * me + lu;
            for (unsigned d = 1; d < 4; ++d) {
                const unsigned dr = grp * 4u + ((me + d) & 3u);
                const rsrc_t dsrd = make_rsrc(P.ws[dr] + OFF_XP + u * XP_UNIT, XP_UNIT);
#pragma unroll
                for (int p = 0; p < 8; ++p) bst16_sys(v[p], dvoff, dsrd, (s >> 4) * 8192u + p * 1024u);
            }
        } else {
            const rsrc_t dsrd = make_rsrc(myws + OFF_XC, XC_BYTES);
#pragma unroll
            for (int p = 0; p < 8; ++p) bst16_sys(v[p], dvoff, dsrd, lu * UNIT_BYTES + s * 8192u + p * 1024u);
        }
        asm volatile("s_waitcnt vmcnt(0)" : "+v"(fws) :: "memory");
        __builtin_amdgcn_s_barrier();
        if (wave == 0) {
            asm volatile("" : "+v"(tk));
            const unsigned call = rfl(tk) + 1u;
            const unsigned u = home ? 5u * me + lu : 20u + 3u * me + (lu - 5u);
            if (lane < 4 && fr != rank) st_sys(fws + OFF_XFLAG + u * 128u + j * 4u, call);
            STAMP(1);
        }
        return;
    }

    const unsigned gi = (unsigned)b - N_PROD;
    const unsigned t = (gi & 7u) * 2u + ((gi >> 3) & 1u), c = gi >> 4;
    auto unit_src = [&](unsigned u, unsigned& a, unsigned& lu) {
        if (u < 20u) { const unsigned i = u / 5u; a = grp * 4u + i; lu = u - 5u * i; }
        else { const unsigned i = (u - 20u) / 3u; a = og + i; lu = 5u + (u - 20u) - 3u * i; }
    };
    volatile LAS unsigned* const ctl = (volatile LAS unsigned*)(L + L_CTRL);
    if (tid < 32) {
        unsigned v0 = 0;
        if (tid == 8 || tid == 9) { unsigned a, lu; unit_src(2u * t + (tid - 8), a, lu); v0 = (a == rank) ? MARGIN : 0u; }
        if (tid == 10) v0 = c < 4u ? NSTEP : 0u;
        ctl[tid] = v0;
    }
    BARRIER();
    if (wave == 6) STAMP(0);

    f32x16 acc[2][2];
    char* lanews = nullptr;
    if (wave < 4) {
        const unsigned wm = (unsigned)wave >> 1, wn = (unsigned)wave & 1u;
        const unsigned h = (unsigned)lane >> 5;
        unsigned xa[4], wa[2][2];
#pragma unroll
        for (int ks = 0; ks < 4; ++ks) xa[ks] = (wm * 64u + (lane & 31u)) * 128u + (((2u * ks + h) ^ (((unsigned)lane >> 1) & 7u)) << 4);
        {
            const unsigned blk = ((unsigned)lane >> 4) & 1u, q = ((unsigned)lane & 15u) >> 2, pp = (unsigned)lane & 3u;
#pragma unroll
            for (int nt = 0; nt < 2; ++nt)
#pragma unroll
                for (int tt = 0; tt < 2; ++tt) {
                    const unsigned row = 8u * h + 4u * tt + q, ch = 4u * (2u * wn + nt) + 2u * blk + (pp >> 1);
                    wa[nt][tt] = 16384u + 256u * row + ((ch ^ (((row & 3u) << 2) | ((row >> 2) & 3u))) << 4) + 8u * (pp & 1u);
                }
        }
#pragma unroll
        for (int i = 0; i < 2; ++i)
#pragma unroll
            for (int jj = 0; jj < 2; ++jj)
#pragma unroll
                for (int e = 0; e < 16; ++e) acc[i][jj][e] = 0.f;
        unsigned have = 0;
#pragma unroll 1
        for (int k4 = 0; k4 < NSTEP; k4 += NSLOT) {
#pragma unroll
            for (int sl = 0; sl < (int)NSLOT; ++sl) {
                const unsigned k = k4 + sl;
                while (have <= k) {
                    const unsigned fa = ctl[wm], fb = ctl[2];
                    have = rfl(fa < fb ? fa : fb);
                    if (have <= k) __builtin_amdgcn_s_sleep(1);
                }
                asm volatile("" ::: "memory");
                LAS char* const S = L + sl * SLOT;
#pragma unroll
                for (int ks = 0; ks < 4; ++ks) {
                    s16x8 xf[2], wf[2];
#pragma unroll
                    for (int mt = 0; mt < 2; ++mt) xf[mt] = *(LAS s16x8*)(S + xa[ks] + mt * 4096);
#pragma unroll
                    for (int nt = 0; nt < 2; ++nt) {
                        const s16x4 lo = __builtin_amdgcn_ds_read_tr16_b64_v4i16((LAS s16x4*)(S + wa[nt][0] + ks * 4096));
                        const s16x4 hi = __builtin_amdgcn_ds_read_tr16_b64_v4i16((LAS s16x4*)(S + wa[nt][1] + ks * 4096));
                        wf[nt] = __builtin_shufflevector(lo, hi, 0, 1, 2, 3, 4, 5, 6, 7);
                    }
#pragma unroll
                    for (int nt = 0; nt < 2; ++nt)
#pragma unroll
                        for (int mt = 0; mt < 2; ++mt)
                            acc[nt][mt] = __builtin_amdgcn_mfma_f32_32x32x16_bf16((bf16x8)wf[nt], (bf16x8)xf[mt], acc[nt][mt], 0, 0, 0);
                }
                WAIT_LGKM0();
                ctl[4 + wave] = k + 1;
            }
        }
        float m = 0.f;
#pragma unroll
        for (int i = 0; i < 2; ++i)
#pragma unroll
            for (int jj = 0; jj < 2; ++jj)
#pragma unroll
                for (int e = 0; e < 16; ++e) m = fmaxf(m, fabsf(acc[i][jj][e]));
        const unsigned mu = wave_max_u32(__float_as_uint(m));
        if (lane == 0) ctl[16 + wave] = mu;
        if (wave == 0) STAMP(2);
    } else if (wave == 4 || wave == 7) {
        const unsigned hf = wave == 7 ? 1u : 0u, u = 2u * t + hf;
        unsigned a, lu; unit_src(u, a, lu);
        a &= 7u;
        const bool own = a == rank, home = u < 20u;
        const u32x4 srd = own ? make_srd(P.x + lu * UNIT_BYTES, UNIT_BYTES) : make_srd(P.ws[a] + OFF_XC + lu * UNIT_BYTES, UNIT_BYTES);
        const u32x4 psrd = make_srd(myws + OFF_XP + u * XP_UNIT, XP_UNIT);
        const unsigned sw0 = (lane >> 3) * 8192u + ((((unsigned)lane & 7u) ^ ((unsigned)lane >> 4)) << 4);
        const unsigned v0 = own ? sw0 : lane * 16u, v1 = own ? (sw0 ^ 64u) : lane * 16u;
        const unsigned pstr = own ? 65536u : 1024u, sstr = own ? 128u : 8192u;
        const bool has_push = home && !own;
        unsigned avail = 0, freed = 0, pub = 0;
        asm volatile("s_nop 4" ::: "memory");
#pragma unroll 1
        for (unsigned j = 0; j < (unsigned)NSTEP; ++j) {
            if (avail <= j) {
                avail = rfl(ctl[8 + hf]);
                if (avail <= j) {
                    WAIT_VM(0);
                    if (pub < j) { pub = j; ctl[hf] = pub; }
                    do { __builtin_amdgcn_s_sleep(1); avail = rfl(ctl[8 + hf]); } while (avail <= j);
                }
            }
            while (freed + NSLOT <= j) {
                const unsigned f0 = ctl[4 + 2 * hf], f1 = ctl[5 + 2 * hf];
                freed = rfl(f0 < f1 ? f0 : f1);
                if (freed + NSLOT <= j) __builtin_amdgcn_s_sleep(1);
            }
            const unsigned dst = (unsigned)(size_t)(L + (j & (NSLOT - 1)) * SLOT + hf * 8192u);
            if (has_push && pushed_step((int)j)) {
#pragma unroll
                for (int p = 0; p < 8; ++p) dma16(dst + p * 1024u, v0, psrd, (j >> 4) * 8192u + p * 1024u);
            } else {
#pragma unroll
                for (int p = 0; p < 8; ++p) dma16(dst + p * 1024u, (p & 1) ? v1 : v0, srd, j * sstr + p * pstr);
            }
            WAIT_VM(16);
            if (j >= 2 && pub < j - 1) { pub = j - 1; ctl[hf] = pub; }
        }
        WAIT_VM(8); ctl[hf] = NSTEP - 1;
        WAIT_VM(0); ctl[hf] = NSTEP;
    } else if (wave == 5) {
        const bool own = c < 4u;
        const u32x4 srd = own ? make_srd(P.w + c * 256u, SHARD_BYTES - c * 256u) : make_srd(myws + OFF_WS + (c & 3u) * WQ_BYTES, WQ_BYTES);
        const unsigned sw = (lane >> 4) * 1024u + ((((unsigned)lane & 15u) ^ (((unsigned)lane >> 4) << 2)) << 4);
        unsigned vo[4];
#pragma unroll
        for (int i = 0; i < 4; ++i) vo[i] = own ? (sw ^ (i << 4)) : lane * 16u;
        const unsigned pstr = own ? 4096u : 1024u, sstr = own ? 65536u : 16384u;
        unsigned avail = 0, freed = 0, pub = 0;
        asm volatile("s_nop 4" ::: "memory");
#pragma unroll 1
        for (unsigned j = 0; j < (unsigned)NSTEP; ++j) {
            if (avail <= j) {
                avail = rfl(ctl[10]);
                if (avail <= j) {
                    WAIT_VM(0);
                    if (pub < j) { pub = j; ctl[2] = pub; }
                    do { __builtin_amdgcn_s_sleep(1); avail = rfl(ctl[10]); } while (avail <= j);
                }
            }
            while (freed + NSLOT <= j) {
                const unsigned f0 = ctl[4], f1 = ctl[5], f2 = ctl[6], f3 = ctl[7];
                const unsigned m0 = f0 < f1 ? f0 : f1, m1 = f2 < f3 ? f2 : f3;
                freed = rfl(m0 < m1 ? m0 : m1);
                if (freed + NSLOT <= j) __builtin_amdgcn_s_sleep(1);
            }
            const unsigned dst = (unsigned)(size_t)(L + (j & (NSLOT - 1)) * SLOT + 16384u);
#pragma unroll
            for (int p = 0; p < 16; ++p) dma16(dst + p * 1024u, vo[p & 3], srd, j * sstr + p * pstr);
            WAIT_VM(32);
            if (j >= 2 && pub < j - 1) { pub = j - 1; ctl[2] = pub; }
        }
        WAIT_VM(16); ctl[2] = NSTEP - 1;
        WAIT_VM(0); ctl[2] = NSTEP;
    } else {
        __builtin_amdgcn_fence(__ATOMIC_ACQUIRE, "");
        unsigned tk = 0;
        if (lane == 0) tk = draw_ticket(myws, b);
        const unsigned call = rfl(tk) + 1u;
        lanews = P.ws[lane & 7];
        asm volatile("s_waitcnt vmcnt(0)" : "+v"(lanews) :: "memory");
        if (lane == 0) ctl[12] = call;
        unsigned a0, a1, lu0, lu1;
        unit_src(2u * t, a0, lu0); unit_src(2u * t + 1u, a1, lu1);
        const bool own0 = a0 == rank, own1 = a1 == rank;
        const unsigned ln = (unsigned)lane & 31u;
        const char* fp = ln < 16u ? myws + OFF_WFLAG + ln * 4u : myws + OFF_XFLAG + (2u * t + ((ln >> 3) & 1u)) * 128u + (ln & 7u) * 4u;
        if (lane >= 32) fp = myws + OFF_TMO;
        const unsigned long long t0 = now();
        for (;;) {
            const unsigned v = ld_sys(fp);
            const int d = (int)(v - call * 64u);
            const unsigned prog = d < 0 ? 0u : (d > NSTEP ? (unsigned)NSTEP : (unsigned)d);
            unsigned wq[4];
#pragma unroll
            for (int i = 0; i < 4; ++i) {
                const unsigned p0 = __builtin_amdgcn_readlane(prog, 4 * i), p1 = __builtin_amdgcn_readlane(prog, 4 * i + 1),
                               p2 = __builtin_amdgcn_readlane(prog, 4 * i + 2), p3 = __builtin_amdgcn_readlane(prog, 4 * i + 3);
                const unsigned m0 = p0 < p1 ? p0 : p1, m1 = p2 < p3 ? p2 : p3;
                wq[i] = m0 < m1 ? m0 : m1;
            }
            const unsigned mw0 = wq[0] < wq[1] ? wq[0] : wq[1], mw1 = wq[2] < wq[3] ? wq[2] : wq[3];
            const unsigned minw = mw0 < mw1 ? mw0 : mw1;
            const unsigned thr = minw + MARGIN < (unsigned)NSTEP ? minw + MARGIN : (unsigned)NSTEP;
            const unsigned long long rdy = __ballot(v >= call);
            const unsigned r0 = own0 ? 0xffu : (unsigned)(rdy >> 16) & 0xffu, r1 = own1 ? 0xffu : (unsigned)(rdy >> 24) & 0xffu;
            const unsigned x0 = 8u * (unsigned)__builtin_ctz(~r0), x1 = 8u * (unsigned)__builtin_ctz(~r1);
            unsigned av0 = x0 < thr ? x0 : thr, av1 = x1 < thr ? x1 : thr;
            unsigned avb = c < 4u ? (unsigned)NSTEP : ((c & 2u) ? ((c & 1u) ? wq[3] : wq[2]) : ((c & 1u) ? wq[1] : wq[0]));
            const bool dead = __builtin_amdgcn_readlane(v, 32) == call || now() - t0 > DEADLINE;
            if (dead) { if (lane == 0) { st_agent(myws + OFF_TMO, call); ctl[11] = 1u; } av0 = av1 = avb = NSTEP; }
            if (lane == 0) { ctl[8] = av0; ctl[9] = av1; ctl[10] = avb; }
            if (av0 == (unsigned)NSTEP && av1 == (unsigned)NSTEP && avb == (unsigned)NSTEP) break;
            __builtin_amdgcn_s_sleep(4);
        }
        STAMP(1);
    }

    BARRIER();
    const unsigned call = rfl(ctl[12]);
    if (wave == 6) {
        const unsigned m0 = ctl[16], m1 = ctl[17], m2 = ctl[18], m3 = ctl[19];
        const unsigned ma = m0 > m1 ? m0 : m1, mb = m2 > m3 ? m2 : m3, mm = ma > mb ? ma : mb;
        if (lane < NR)
            __hip_atomic_store((gu64*)(lanews + OFF_AMAX + (rank * 128u + gi) * 8u), ((unsigned long long)call << 32) | mm,
                               __ATOMIC_RELAXED, __HIP_MEMORY_SCOPE_SYSTEM);
    }
    if (wave < 4) {
        const unsigned wm = (unsigned)wave >> 1, wn = (unsigned)wave & 1u, h = (unsigned)lane >> 5;
#pragma unroll
        for (int nt = 0; nt < 2; ++nt)
#pragma unroll
            for (int mt = 0; mt < 2; ++mt)
#pragma unroll
                for (int g = 0; g < 4; ++g) {
                    const unsigned m = wm * 64u + mt * 32u + (lane & 31u), ch = 16u * wn + 8u * nt + 2u * g + h;
                    f32x4 q; q.x = acc[nt][mt][4 * g]; q.y = acc[nt][mt][4 * g + 1]; q.z = acc[nt][mt][4 * g + 2]; q.w = acc[nt][mt][4 * g + 3];
                    *(LAS f32x4*)(L + m * 512u + ((ch ^ (m & 7u)) << 4)) = q;
                }
    }
    BARRIER();
    f32x4 y[2][4];
    const unsigned r0 = (unsigned)tid >> 3, q8 = (unsigned)tid & 7u;
#pragma unroll
    for (int hf = 0; hf < 2; ++hf)
#pragma unroll
        for (int e = 0; e < 4; ++e) y[hf][e] = *(LAS f32x4*)(L + (r0 + 64u * hf) * 512u + (((4u * q8 + e) ^ (r0 & 7u)) << 4));
    {
        const char* gp = myws + OFF_AMAX + tid * 16;
        const char* tp = myws + OFF_TMO;
        const unsigned long long t0 = now();
        unsigned mx = 0;
        for (;;) {
            u32x4 g; unsigned tm;
            asm volatile("global_load_dwordx4 %0, %2, off sc0 sc1\n\tglobal_load_dword %1, %3, off sc1\n\ts_waitcnt vmcnt(0)"
                         : "=&v"(g), "=&v"(tm) : "v"(gp), "v"(tp) : "memory");
            mx = g.x > g.z ? g.x : g.z;
            const bool ok = g.y >= call && g.w >= call;
            const bool dead = rfl(tm) == call || now() - t0 > DEADLINE;
            if (dead) { if (lane == 0) { st_agent(myws + OFF_TMO, call); ctl[11] = 1u; } break; }
            if (__all(ok)) break;
            __builtin_amdgcn_s_sleep(1);
        }
        mx = wave_max_u32(mx);
        if (lane == 0) ctl[24 + wave] = mx;
    }
    BARRIER();
    if (wave == 6) STAMP(3);
    unsigned gm = 0;
#pragma unroll
    for (int i = 0; i < 8; ++i) { const unsigned o = ctl[24 + i]; gm = o > gm ? o : gm; }
    const bool aborted = ctl[11] != 0u;
    const float amax = __uint_as_float(gm);
    const float inv = amax > 0.f ? 127.0f / amax : 0.f;
    const unsigned orank = (c < 4u ? rank : og + (c & 3u)) & 7u;
    const unsigned ocol = (c < 4u ? c : me) * 128u;
    const rsrc_t osrd = make_rsrc(P.out[orank], OUT_BYTES);
    constexpr float MAGIC = 12582912.0f;
#pragma unroll
    for (int hf = 0; hf < 2; ++hf) {
        unsigned a, lu; unit_src(2u * t + hf, a, lu);
        u32x4 o;
#pragma unroll
        for (int e = 0; e < 4; ++e) {
            unsigned z[4];
#pragma unroll
            for (int i = 0; i < 4; ++i)
                z[i] = __float_as_uint(__builtin_amdgcn_fmed3f(__builtin_fmaf(y[hf][e][i], inv, MAGIC), MAGIC - 127.0f, MAGIC + 127.0f));
            const unsigned lo = __builtin_amdgcn_perm(z[1], z[0], 0x0c0c0400u), hi = __builtin_amdgcn_perm(z[3], z[2], 0x04000c0cu);
            o[e] = aborted ? 0x80808080u : (lo | hi);
        }
        bst16_sys(o, r0 * 512u + q8 * 16u, osrd, (a * 512u + lu * 64u) * 512u + ocol);
    }
    WAIT_VM(0);
    __builtin_amdgcn_s_barrier();
    if (tid == 0) {
        if (c >= 4u) {
            st_sys(P.ws[orank] + OFF_DONE + (me * 16u + t) * 128u, call);
        } else {
            const char* dp = myws + OFF_DONE + (c * 16u + t) * 128u;
            const unsigned long long t0 = now();
            while (ld_sys(dp) < call) {
                if (ld_agent(myws + OFF_TMO) == call || now() - t0 > DEADLINE) {
                    st_agent(myws + OFF_TMO, call);
                    unsigned a, lu; unit_src(2u * t, a, lu);
                    P.out[rank][(a * 512u + lu * 64u) * 512u + ocol] = (char)-128;
                    break;
                }
                __builtin_amdgcn_s_sleep(1);
            }
            __builtin_amdgcn_fence(__ATOMIC_ACQUIRE, "");
        }
    }
    if (wave == 0) STAMP(4);
}

extern "C" void kernel_launch(int rank, int nranks, ncclComm_t comm,
                              hipStream_t stream,
                              void* const* d_in, const int* in_sizes, int n_in,
                              void* d_out, int out_size,
                              const int* in_dtypes, int out_dtype,
                              void* const* d_out_all,
                              void* const* d_ws_all, size_t ws_bytes) {
    (void)comm; (void)in_dtypes; (void)out_dtype; (void)d_out;
    if (nranks != NR || (unsigned)rank >= (unsigned)NR || n_in < 2 || ws_bytes < WS_NEEDED ||
        in_sizes[0] != 512 * 4096 || in_sizes[1] != 4096 * 512 || out_size != (int)OUT_BYTES) {
        fprintf(stderr, "kernel_launch: unexpected problem shape\n");
        return;
    }
    Params P;
    P.x = (const char*)d_in[0];
    P.w = (const char*)d_in[1];
    for (int p = 0; p < NR; ++p) { P.ws[p] = (char*)d_ws_all[p]; P.out[p] = (char*)d_out_all[p]; }
    P.rank = rank;
    hipLaunchKernelGGL(ag_gemm_q8, dim3(GRID), dim3(512), 0, stream, P);
    HIP_CHECK(hipGetLastError());
}
```

```cpp
#include <hip/hip_runtime.h>
#include <rccl/rccl.h>
#include <cstdio>
#include <cstdint>

#define HIP_CHECK(expr) do { hipError_t _e = (expr); if (_e != hipSuccess) { \
    fprintf(stderr, "HIP error %s at %s:%d\n", hipGetErrorString(_e), __FILE__, __LINE__); \
    return; } } while (0)

typedef __attribute__((ext_vector_type(4))) unsigned u32x4;
typedef __attribute__((ext_vector_type(4))) float f32x4;
typedef __attribute__((ext_vector_type(16))) float f32x16;
typedef __attribute__((ext_vector_type(4))) short s16x4;
typedef __attribute__((ext_vector_type(8))) short s16x8;
typedef __attribute__((ext_vector_type(8))) __bf16 bf16x8;
#define LAS __attribute__((address_space(3)))
#define GAS __attribute__((address_space(1)))
typedef GAS unsigned gu32;
typedef GAS unsigned long long gu64;

constexpr int NR = 8;
constexpr int NSTEP = 64;
constexpr int N_PUSH = 56;
constexpr int N_PROD = N_PUSH;
constexpr int N_GEMM = 128;
constexpr int GRID = N_PROD + N_GEMM;
constexpr int NTHREADS = 576;
#define AG_WIN 2
#define AG_LEAD 64
constexpr int LEAD = AG_LEAD;
#define AG_T0 65
#define AG_PSLEEP 1
#define AG_STAG 1
#define AG_TAU 710
#define AG_FSTR 128
constexpr unsigned FSTR = AG_FSTR;
constexpr int WIN = AG_WIN;
constexpr unsigned long long DEADLINE = 100000000ull;

constexpr unsigned OFF_CTR = 0;
constexpr unsigned OFF_TMO = 0x8000;
constexpr unsigned OFF_PACE = 0x8400;
constexpr unsigned OFF_FLAG = 0xE000;
constexpr unsigned OFF_DONE = 0xA000;
constexpr unsigned OFF_AMAX = 0xC000;
constexpr unsigned OFF_DBG = 0x10000;
constexpr unsigned UNIT_BYTES = 64u * 8192u;
constexpr unsigned OFF_ST = 1u << 20;
constexpr unsigned ST_PIECE = 5u * 1024u, ST_STEP = 8u * ST_PIECE, ST_BYTES = 64u * ST_STEP;
constexpr unsigned WS_NEEDED = OFF_ST + 8u * ST_BYTES;
constexpr unsigned SHARD_BYTES = 4u << 20;
constexpr unsigned OUT_BYTES = 4096u * 512u;
static_assert(GRID * 128 <= (int)OFF_TMO, "counter area");

constexpr unsigned SLOT = 32768, NSLOT = 4;
constexpr unsigned L_CTRL = SLOT * NSLOT;
constexpr unsigned L_FULL = L_CTRL, L_FREE = L_CTRL + 16, L_AVAIL = L_CTRL + 32, L_ABORT = L_CTRL + 44, L_CALL = L_CTRL + 48,
                   L_WMAX = L_CTRL + 64, L_GMAX = L_CTRL + 96;
constexpr unsigned LDS_BYTES = L_CTRL + 128;

struct Params {
    const char* x;
    const char* w;
    char* ws[NR];
    char* out[NR];
    int rank;
};

__device__ __forceinline__ unsigned rfl(unsigned v) { return __builtin_amdgcn_readfirstlane(v); }
__device__ __forceinline__ u32x4 make_srd(const void* p, unsigned bytes) {
    const unsigned long long a = (unsigned long long)p;
    u32x4 r;
    r.x = rfl((unsigned)a); r.y = rfl((unsigned)(a >> 32) & 0xffffu); r.z = rfl(bytes); r.w = 0x00020000u;
    return r;
}
__device__ __forceinline__ void dma16(unsigned lds, unsigned voff, u32x4 srd, unsigned soff) {
    asm volatile("s_mov_b32 m0, %0\n\ts_nop 0\n\tbuffer_load_dwordx4 %1, %2, %3 offen lds"
                 :: "s"(lds), "v"(voff), "s"(srd), "s"(soff) : "memory");
}
typedef __amdgpu_buffer_rsrc_t rsrc_t;
__device__ __forceinline__ rsrc_t make_rsrc(const void* p, unsigned bytes) {
    const unsigned long long a = (unsigned long long)p;
    const unsigned long long u = ((unsigned long long)rfl((unsigned)(a >> 32)) << 32) | rfl((unsigned)a);
    return __builtin_amdgcn_make_buffer_rsrc((void*)u, 0, rfl(bytes), 0x00020000);
}
__device__ __forceinline__ u32x4 bld16(unsigned voff, rsrc_t r, unsigned soff) { return __builtin_amdgcn_raw_buffer_load_b128(r, voff, soff, 0); }
__device__ __forceinline__ void bst16_sys(u32x4 v, unsigned voff, rsrc_t r, unsigned soff) {
    __builtin_amdgcn_raw_buffer_store_b128(v, r, voff, soff, 17);
}
#define WAIT_VM(n) asm volatile("s_waitcnt vmcnt(" #n ")" ::: "memory")
#define WAIT_LGKM0() asm volatile("s_waitcnt lgkmcnt(0)" ::: "memory")
#define BARRIER() do { asm volatile("s_waitcnt lgkmcnt(0)" ::: "memory"); __builtin_amdgcn_s_barrier(); asm volatile("" ::: "memory"); } while (0)
__device__ __forceinline__ unsigned long long now() { return __builtin_amdgcn_s_memrealtime(); }
__device__ __forceinline__ unsigned ld_sys(const void* p) { return __hip_atomic_load((gu32*)p, __ATOMIC_RELAXED, __HIP_MEMORY_SCOPE_SYSTEM); }
__device__ __forceinline__ void st_sys(void* p, unsigned v) { __hip_atomic_store((gu32*)p, v, __ATOMIC_RELAXED, __HIP_MEMORY_SCOPE_SYSTEM); }
__device__ __forceinline__ unsigned ld_agent(const void* p) { return __hip_atomic_load((gu32*)p, __ATOMIC_RELAXED, __HIP_MEMORY_SCOPE_AGENT); }
__device__ __forceinline__ void st_agent(void* p, unsigned v) { __hip_atomic_store((gu32*)p, v, __ATOMIC_RELAXED, __HIP_MEMORY_SCOPE_AGENT); }
__device__ __forceinline__ unsigned draw_ticket(char* ws, int b) {
    unsigned z = 0;
    asm volatile("" : "+v"(z));
    return __hip_atomic_fetch_add((gu32*)(ws + OFF_CTR + b * 128 + z), 1u, __ATOMIC_RELAXED, __HIP_MEMORY_SCOPE_AGENT);
}
__device__ __forceinline__ unsigned wave_max_u32(unsigned v) {
#pragma unroll
    for (int m = 32; m >= 1; m >>= 1) { const unsigned o = (unsigned)__shfl_xor((int)v, m); v = o > v ? o : v; }
    return v;
}

#define STAMP(i) do { } while (0)
#define STORE_DBG(i, v) do { } while (0)
#define TIC() do { } while (0)
#define TOC(acc) do { } while (0)

__global__ void __launch_bounds__(NTHREADS, 3) ag_gemm_q8(Params P) {
    __shared__ __attribute__((aligned(16))) char smem[LDS_BYTES];
    LAS char* const L = (LAS char*)smem;
    const int b = blockIdx.x;
    const int tid = threadIdx.x, lane = tid & 63;
    const int wave = (int)rfl(tid >> 6);
    const unsigned rank = (unsigned)P.rank & 7u;
    const unsigned grp = rank >> 2, me = rank & 3u, og = (grp ^ 1u) * 4u;
    char* const myws = P.ws[rank];

    volatile LAS unsigned* const ctl = (volatile LAS unsigned*)(L + L_CTRL);
    auto unit_src = [&](unsigned u, unsigned& a, unsigned& lu) {
        if (u < 20u) { const unsigned i = u / 5u; a = grp * 4u + i; lu = u - 5u * i; }
        else { const unsigned i = (u - 20u) / 3u; a = og + i; lu = 5u + (u - 20u) - 3u * i; }
        a &= 7u;
    };
    auto min4 = [](unsigned v, int l) {
        const unsigned p0 = __builtin_amdgcn_readlane(v, l), p1 = __builtin_amdgcn_readlane(v, l + 1),
                       p2 = __builtin_amdgcn_readlane(v, l + 2), p3 = __builtin_amdgcn_readlane(v, l + 3);
        const unsigned m0 = p0 < p1 ? p0 : p1, m1 = p2 < p3 ? p2 : p3;
        return m0 < m1 ? m0 : m1;
    };
    if (b < N_PROD) {
        if (wave > 6) return;
        if (wave == 0) STAMP(0);
        const unsigned link = (unsigned)b >> 3, p = (unsigned)b & 7u, st = (unsigned)wave;
        const bool intra = link < 3u, isw = !intra && st >= 3u;
        const unsigned di = intra ? (me + 1u + link) & 3u : (me + link - 3u) & 3u;
        const unsigned dr = ((intra ? grp * 4u : og) + di) & 7u;
        char* const dws = P.ws[dr];
        const unsigned lu = intra ? st : 5u + st;
        const unsigned pw = 8u * (st - 3u) + p;
        const rsrc_t ssrd = make_rsrc(isw ? P.w : P.x, SHARD_BYTES);
        const rsrc_t dsrd = make_rsrc(dws + OFF_ST + rank * ST_BYTES, ST_BYTES);
        const unsigned svoff = isw ? (4u * pw + (lane >> 4)) * 1024u + ((((unsigned)lane & 15u) ^ (((unsigned)lane >> 4) << 2) ^ (pw & 3u)) << 4) + di * 256u
                                   : (lu * 64u + 8u * p + (lane >> 3)) * 8192u + ((((unsigned)lane & 7u) ^ ((unsigned)lane >> 4) ^ ((p & 1u) << 2)) << 4);
        const unsigned dvoff = p * ST_PIECE + st * 1024u + lane * 16u;
        const unsigned sstr = isw ? 65536u : 128u, dstr = ST_STEP;
        char* const flag = dws + OFF_FLAG + (rank * 8u + p) * FSTR;
        auto publish_at = [](int done) { return done > 0 && ((done & 1) == 0 || done > NSTEP - 10); };
        if (wave >= 5) {
            unsigned tk = 0;
            if (tid == 5 * 64) { tk = draw_ticket(myws, b); ctl[0] = tk; }
            BARRIER();
            const unsigned base = rfl(ctl[0]) * 64u + 64u;
            char* const pace = myws + OFF_PACE + link * 128u;
#pragma unroll 1
            for (int done = 1; done <= NSTEP; ++done) {
                if (!publish_at(done)) continue;
                __builtin_amdgcn_s_barrier();
                if (wave == 5) {
                    if (lane == 0) { st_sys(flag, base + (unsigned)done); st_agent(pace + p * 4u, base + (unsigned)done); }
                } else if (LEAD < NSTEP && done <= NSTEP - WIN) {
                    for (int spin = 0; spin < 8; ++spin) {
                        const int lag = done - (int)(ld_agent(pace + (lane & 7) * 4) - base);
                        if (!__any(lag > LEAD && (unsigned)(lane & 7) != p)) break;
                    }
                }
            }
            if (wave == 5)
            STAMP(1);
            return;
        }
        u32x4 v[WIN];
        const unsigned long long tp = now() + (AG_STAG ? (st * 8u + p) * AG_TAU / 400u : 0u);
#pragma unroll
        for (int i = 0; i < WIN; ++i) v[i] = bld16(svoff, ssrd, i * sstr);
        asm volatile("" ::: "memory");
#pragma unroll
        for (int i = 0; i < NSTEP; ++i) {
            if (i >= WIN) {
                const int younger = (WIN - 1) + ((NSTEP - 1 - i) < (WIN - 1) ? (NSTEP - 1 - i) : (WIN - 1));
                asm volatile("s_waitcnt vmcnt(%0)" :: "i"(younger) : "memory");
            }
            if (AG_TAU > 0 && i >= WIN) { while ((long long)(now() - tp) < (long long)((i - WIN + 1) * AG_TAU / 10 + AG_T0)) __builtin_amdgcn_s_sleep(1); }
            bst16_sys(v[i % WIN], dvoff, dsrd, i * dstr);
            if (i + WIN < NSTEP) v[i % WIN] = bld16(svoff, ssrd, (i + WIN) * sstr);
            asm volatile("" ::: "memory");
            if (i == WIN - 1) __builtin_amdgcn_s_barrier();
            if (publish_at(i - WIN + 1)) __builtin_amdgcn_s_barrier();
        }
#pragma unroll
        for (int done = NSTEP - WIN + 1; done <= NSTEP; ++done) {
            asm volatile("s_waitcnt vmcnt(%0)" :: "i"(NSTEP - done) : "memory");
            __builtin_amdgcn_s_barrier();
        }
        return;
    }

    const unsigned gi = (unsigned)b - N_PROD;
    const unsigned t = ((gi & 7u) >> 1) * 4u + ((gi >> 3) & 3u), c = (gi & 1u) * 2u + ((gi >> 5) & 1u) + (gi >> 6) * 4u;
    unsigned a0, a1, lu0, lu1;
    unit_src(2u * t, a0, lu0); unit_src(2u * t + 1u, a1, lu1);
    const bool own0 = a0 == rank, own1 = a1 == rank;
    if (tid < 32) ctl[tid] = ((tid == 8 && own0) || (tid == 9 && own1) || (tid == 10 && c < 4u)) ? (unsigned)NSTEP : 0u;
    BARRIER();
    if (wave == 8) STAMP(0);

    f32x16 acc[2][2];
    char* lanews = nullptr;
    if (wave < 4) {
        const unsigned wm = (unsigned)wave >> 1, wn = (unsigned)wave & 1u;
        const unsigned h = (unsigned)lane >> 5;
        unsigned xa[4], wa[2][2];
#pragma unroll
        for (int ks = 0; ks < 4; ++ks) xa[ks] = (wm * 64u + (lane & 31u)) * 128u + (((2u * ks + h) ^ (((unsigned)lane >> 1) & 7u)) << 4);
        {
            const unsigned blk = ((unsigned)lane >> 4) & 1u, q = ((unsigned)lane & 15u) >> 2, pp = (unsigned)lane & 3u;
#pragma unroll
            for (int nt = 0; nt < 2; ++nt)
#pragma unroll
                for (int tt = 0; tt < 2; ++tt) {
                    const unsigned row = 8u * h + 4u * tt + q, ch = 4u * (2u * wn + nt) + 2u * blk + (pp >> 1);
                    wa[nt][tt] = 16384u + 256u * row + ((ch ^ (((row & 3u) << 2) | ((row >> 2) & 3u))) << 4) + 8u * (pp & 1u);
                }
        }
#pragma unroll
        for (int i = 0; i < 2; ++i)
#pragma unroll
            for (int jj = 0; jj < 2; ++jj)
#pragma unroll
                for (int e = 0; e < 16; ++e) acc[i][jj][e] = 0.f;
        unsigned have = 0;
        [[maybe_unused]] unsigned long long dw = 0;
#pragma unroll 1
        for (int k4 = 0; k4 < NSTEP; k4 += NSLOT) {
#pragma unroll
            for (int sl = 0; sl < (int)NSLOT; ++sl) {
                const unsigned k = k4 + sl;
                { TIC();
                while (have <= k) {
                    const u32x4 f = *(volatile LAS u32x4*)(L + L_FULL);
                    const unsigned fa = wm ? f.y : f.x, fb = f.z < f.w ? f.z : f.w;
                    have = rfl(fa < fb ? fa : fb);
                    if (have <= k) __builtin_amdgcn_s_sleep(1);
                }
                TOC(dw); }
                asm volatile("" ::: "memory");
                LAS char* const S = L + sl * SLOT;
                s16x8 xf[4][2], wf[4][2];
                auto rd = [&](int ks) {
#pragma unroll
                    for (int nt = 0; nt < 2; ++nt) {
                        const s16x4 lo = __builtin_amdgcn_ds_read_tr16_b64_v4i16((LAS s16x4*)(S + wa[nt][0] + ks * 4096));
                        const s16x4 hi = __builtin_amdgcn_ds_read_tr16_b64_v4i16((LAS s16x4*)(S + wa[nt][1] + ks * 4096));
                        wf[ks][nt] = __builtin_shufflevector(lo, hi, 0, 1, 2, 3, 4, 5, 6, 7);
                    }
#pragma unroll
                    for (int mt = 0; mt < 2; ++mt) xf[ks][mt] = *(LAS s16x8*)(S + xa[ks] + mt * 4096);
                    __builtin_amdgcn_sched_barrier(0);
                };
                auto mm = [&](int ks) {
#pragma unroll
                    for (int nt = 0; nt < 2; ++nt)
#pragma unroll
                        for (int mt = 0; mt < 2; ++mt)
                            acc[nt][mt] = __builtin_amdgcn_mfma_f32_32x32x16_bf16((bf16x8)wf[ks][nt], (bf16x8)xf[ks][mt], acc[nt][mt], 0, 0, 0);
                    __builtin_amdgcn_sched_barrier(0);
                };
                rd(0); rd(1); mm(0); rd(2); mm(1); rd(3); mm(2);
                WAIT_LGKM0();
                ctl[4 + wave] = k + 1;
                __builtin_amdgcn_sched_barrier(0);
                mm(3);
            }
        }
        float m = 0.f;
#pragma unroll
        for (int i = 0; i < 2; ++i)
#pragma unroll
            for (int jj = 0; jj < 2; ++jj)
#pragma unroll
                for (int e = 0; e < 16; ++e) m = fmaxf(m, fabsf(acc[i][jj][e]));
        const unsigned mu = wave_max_u32(__float_as_uint(m));
        if (lane == 0) ctl[16 + wave] = mu;
        if (wave == 0) { STAMP(2); STORE_DBG(5, dw); }
    } else if (wave < 8) {
        const unsigned li = (unsigned)wave - 4u, hf = li & 1u;
        const bool isx = li < 2u;
        const bool own = isx ? (hf ? own1 : own0) : (c < 4u);
        const unsigned luh = hf ? lu1 : lu0;
        const unsigned src = isx ? (hf ? a1 : a0) : og + (c & 3u), stn = isx ? (luh < 5u ? luh : luh - 5u) : 3u + hf;
        const char* base; unsigned bytes;
        if (!own) { base = myws + OFF_ST + src * ST_BYTES + stn * 1024u; bytes = ST_BYTES - stn * 1024u; }
        else if (isx) { base = P.x + luh * UNIT_BYTES; bytes = UNIT_BYTES; }
        else { base = P.w + c * 256u + hf * 32768u; bytes = SHARD_BYTES - c * 256u - hf * 32768u; }
        const u32x4 srd = make_srd(base, bytes);
        const unsigned lin = lane * 16u;
        const unsigned swx = (lane >> 3) * 8192u + ((((unsigned)lane & 7u) ^ ((unsigned)lane >> 4)) << 4);
        const unsigned sww = (lane >> 4) * 1024u + ((((unsigned)lane & 15u) ^ (((unsigned)lane >> 4) << 2)) << 4);
        unsigned vo[4];
#pragma unroll
        for (int i = 0; i < 4; ++i) vo[i] = own ? (isx ? (swx ^ ((i & 1) << 6)) : (sww ^ (i << 4))) : lin;
        const unsigned pstr = own ? (isx ? 65536u : 4096u) : ST_PIECE;
        const unsigned sstr = own ? (isx ? 128u : 65536u) : ST_STEP;
        const unsigned avw = isx ? 8u + hf : 10u;
        unsigned avail = 0, freed = 0, pub = 0;
        [[maybe_unused]] unsigned long long da = 0, df = 0, dv = 0;
        asm volatile("s_nop 4" ::: "memory");
#pragma unroll 1
        for (unsigned j = 0; j < (unsigned)NSTEP; ++j) {
            { TIC();
            if (avail <= j) {
                avail = rfl(ctl[avw]);
                if (avail <= j) {
                    WAIT_VM(0);
                    if (pub < j) { pub = j; ctl[li] = pub; }
                    do { __builtin_amdgcn_s_sleep(1); avail = rfl(ctl[avw]); } while (avail <= j);
                }
            }
            TOC(da); }
            { TIC();
            while (freed + NSLOT <= j) {
                const u32x4 f = *(volatile LAS u32x4*)(L + L_FREE);
                const unsigned m0 = f.x < f.y ? f.x : f.y, m1 = f.z < f.w ? f.z : f.w;
                freed = rfl(isx ? (hf ? m1 : m0) : (m0 < m1 ? m0 : m1));
                if (freed + NSLOT <= j) __builtin_amdgcn_s_sleep(1);
            }
            TOC(df); }
            const unsigned dst = (unsigned)(size_t)(L + (j & (NSLOT - 1)) * SLOT) + li * 8192u;
#pragma unroll
            for (int p = 0; p < 8; ++p) dma16(dst + p * 1024u, vo[p & 3], srd, j * sstr + p * pstr);
            { TIC(); WAIT_VM(16); TOC(dv); }
            if (j >= 2 && pub < j - 1) { pub = j - 1; ctl[li] = pub; }
        }
        WAIT_VM(8); ctl[li] = NSTEP - 1;
        WAIT_VM(0); ctl[li] = NSTEP;
        STORE_DBG(8 + li * 3, da); STORE_DBG(9 + li * 3, df); STORE_DBG(10 + li * 3, dv);
    } else {
        __builtin_amdgcn_fence(__ATOMIC_ACQUIRE, "");
        unsigned tk = 0;
        if (lane == 0) tk = draw_ticket(myws, b);
        const unsigned call = rfl(tk) + 1u;
        lanews = P.ws[lane & 7];
        asm volatile("s_waitcnt vmcnt(0)" : "+v"(lanews) :: "memory");
        if (lane == 0) ctl[12] = call;
        const unsigned fsrc = lane < 8 ? a0 : (lane < 16 ? a1 : og + (c & 3u));
        const char* fp = lane < 24 ? myws + OFF_FLAG + (fsrc * 8u + (lane & 7)) * FSTR : myws + OFF_TMO;
        const unsigned long long t0 = now();
        [[maybe_unused]] unsigned npoll = 0;
        for (;;) {
            const unsigned v = ld_sys(fp);
            const int d = (int)(v - call * 64u);
            const unsigned prog = d < 0 ? 0u : (d > NSTEP ? (unsigned)NSTEP : (unsigned)d);
            auto min8 = [&](int l) { const unsigned m0 = min4(prog, l), m1 = min4(prog, l + 4); return m0 < m1 ? m0 : m1; };
            unsigned av0 = own0 ? (unsigned)NSTEP : min8(0), av1 = own1 ? (unsigned)NSTEP : min8(8), avb = c < 4u ? (unsigned)NSTEP : min8(16);
            const bool dead = __builtin_amdgcn_readlane(v, 24) == call || now() - t0 > DEADLINE;
            if (dead) { if (lane == 0) { st_agent(myws + OFF_TMO, call); ctl[11] = 1u; } av0 = av1 = avb = NSTEP; }
            if (lane == 0) { ctl[8] = av0; ctl[9] = av1; ctl[10] = avb; }
            if (av0 == (unsigned)NSTEP && av1 == (unsigned)NSTEP && avb == (unsigned)NSTEP) break;
            __builtin_amdgcn_s_sleep(AG_PSLEEP);
        }
        STORE_DBG(6, npoll);
        STAMP(1);
    }

    BARRIER();
    const unsigned call = rfl(ctl[12]);
    if (wave == 8) {
        const unsigned m0 = ctl[16], m1 = ctl[17], m2 = ctl[18], m3 = ctl[19];
        const unsigned ma = m0 > m1 ? m0 : m1, mb = m2 > m3 ? m2 : m3, mm = ma > mb ? ma : mb;
        if (lane < NR)
            __hip_atomic_store((gu64*)(lanews + OFF_AMAX + (rank * 128u + gi) * 8u), ((unsigned long long)call << 32) | mm,
                               __ATOMIC_RELAXED, __HIP_MEMORY_SCOPE_SYSTEM);
    }
    if (wave >= 8) return;
    if (wave < 4) {
        const unsigned wm = (unsigned)wave >> 1, wn = (unsigned)wave & 1u, h = (unsigned)lane >> 5;
#pragma unroll
        for (int nt = 0; nt < 2; ++nt)
#pragma unroll
            for (int mt = 0; mt < 2; ++mt)
#pragma unroll
                for (int g = 0; g < 4; ++g) {
                    const unsigned m = wm * 64u + mt * 32u + (lane & 31u), ch = 16u * wn + 8u * nt + 2u * g + h;
                    f32x4 q; q.x = acc[nt][mt][4 * g]; q.y = acc[nt][mt][4 * g + 1]; q.z = acc[nt][mt][4 * g + 2]; q.w = acc[nt][mt][4 * g + 3];
                    *(LAS f32x4*)(L + m * 512u + ((ch ^ (m & 7u)) << 4)) = q;
                }
    }
    BARRIER();
    f32x4 y[2][4];
    const unsigned r0 = (unsigned)tid >> 3, q8 = (unsigned)tid & 7u;
#pragma unroll
    for (int hf = 0; hf < 2; ++hf)
#pragma unroll
        for (int e = 0; e < 4; ++e) y[hf][e] = *(LAS f32x4*)(L + (r0 + 64u * hf) * 512u + (((4u * q8 + e) ^ (r0 & 7u)) << 4));
    {
        const char* gp = myws + OFF_AMAX + tid * 16;
        const char* tp = myws + OFF_TMO;
        const unsigned long long t0 = now();
        unsigned mx = 0;
        for (;;) {
            u32x4 g; unsigned tm;
            asm volatile("global_load_dwordx4 %0, %2, off sc0 sc1\n\tglobal_load_dword %1, %3, off sc1\n\ts_waitcnt vmcnt(0)"
                         : "=&v"(g), "=&v"(tm) : "v"(gp), "v"(tp) : "memory");
            mx = g.x > g.z ? g.x : g.z;
            const bool ok = g.y >= call && g.w >= call;
            const bool dead = rfl(tm) == call || now() - t0 > DEADLINE;
            if (dead) { if (lane == 0) { st_agent(myws + OFF_TMO, call); ctl[11] = 1u; } break; }
            if (__all(ok)) break;
            __builtin_amdgcn_s_sleep(1);
        }
        mx = wave_max_u32(mx);
        if (lane == 0) ctl[24 + wave] = mx;
    }
    BARRIER();
    if (wave == 5) STAMP(3);
    unsigned gm = 0;
#pragma unroll
    for (int i = 0; i < 8; ++i) { const unsigned o = ctl[24 + i]; gm = o > gm ? o : gm; }
    const bool aborted = ctl[11] != 0u;
    const float amax = __uint_as_float(gm);
    const float inv = amax > 0.f ? 127.0f / amax : 0.f;
    const unsigned orank = (c < 4u ? rank : og + (c & 3u)) & 7u;
    const unsigned ocol = (c < 4u ? c : me) * 128u;
    const rsrc_t osrd = make_rsrc(P.out[orank], OUT_BYTES);
    constexpr float MAGIC = 12582912.0f;
#pragma unroll
    for (int hf = 0; hf < 2; ++hf) {
        const unsigned a = hf ? a1 : a0, lu = hf ? lu1 : lu0;
        u32x4 o;
#pragma unroll
        for (int e = 0; e < 4; ++e) {
            unsigned z[4];
#pragma unroll
            for (int i = 0; i < 4; ++i)
                z[i] = __float_as_uint(__builtin_amdgcn_fmed3f(__builtin_fmaf(y[hf][e][i], inv, MAGIC), MAGIC - 127.0f, MAGIC + 127.0f));
            const unsigned lo = __builtin_amdgcn_perm(z[1], z[0], 0x0c0c0400u), hi = __builtin_amdgcn_perm(z[3], z[2], 0x04000c0cu);
            o[e] = aborted ? 0x80808080u : (lo | hi);
        }
        bst16_sys(o, r0 * 512u + q8 * 16u, osrd, (a * 512u + lu * 64u) * 512u + ocol);
    }
    WAIT_VM(0);
    __builtin_amdgcn_s_barrier();
    if (tid == 0) {
        if (c >= 4u) {
            st_sys(P.ws[orank] + OFF_DONE + (me * 16u + t) * 128u, call);
        } else {
            const char* dp = myws + OFF_DONE + (c * 16u + t) * 128u;
            const unsigned long long t0 = now();
            while (ld_sys(dp) < call) {
                if (ld_agent(myws + OFF_TMO) == call || now() - t0 > DEADLINE) {
                    st_agent(myws + OFF_TMO, call);
                    P.out[rank][(a0 * 512u + lu0 * 64u) * 512u + ocol] = (char)-128;
                    break;
                }
                __builtin_amdgcn_s_sleep(1);
            }
            __builtin_amdgcn_fence(__ATOMIC_ACQUIRE, "");
        }
    }
    if (wave == 0) STAMP(4);
}

extern "C" void kernel_launch(int rank, int nranks, ncclComm_t comm,
                              hipStream_t stream,
                              void* const* d_in, const int* in_sizes, int n_in,
                              void* d_out, int out_size,
                              const int* in_dtypes, int out_dtype,
                              void* const* d_out_all,
                              void* const* d_ws_all, size_t ws_bytes) {
    (void)comm; (void)in_dtypes; (void)out_dtype; (void)d_out;
    if (nranks != NR || (unsigned)rank >= (unsigned)NR || n_in < 2 || ws_bytes < WS_NEEDED ||
        in_sizes[0] != 512 * 4096 || in_sizes[1] != 4096 * 512 || out_size != (int)OUT_BYTES) {
        fprintf(stderr, "kernel_launch: unexpected problem shape\n");
        return;
    }
    Params P;
    P.x = (const char*)d_in[0];
    P.w = (const char*)d_in[1];
    for (int p = 0; p < NR; ++p) { P.ws[p] = (char*)d_ws_all[p]; P.out[p] = (char*)d_out_all[p]; }
    P.rank = rank;
    hipLaunchKernelGGL(ag_gemm_q8, dim3(GRID), dim3(NTHREADS), 0, stream, P);
    HIP_CHECK(hipGetLastError());
}
```

```cpp
#include <hip/hip_runtime.h>
#include <rccl/rccl.h>
#include <cstdio>
#include <cstdint>

#define HIP_CHECK(expr) do { hipError_t _e = (expr); if (_e != hipSuccess) { \
    fprintf(stderr, "HIP error %s at %s:%d\n", hipGetErrorString(_e), __FILE__, __LINE__); \
    return; } } while (0)

typedef __attribute__((ext_vector_type(4))) unsigned u32x4;
typedef __attribute__((ext_vector_type(4))) float f32x4;
typedef __attribute__((ext_vector_type(16))) float f32x16;
typedef __attribute__((ext_vector_type(4))) short s16x4;
typedef __attribute__((ext_vector_type(8))) short s16x8;
typedef __attribute__((ext_vector_type(8))) __bf16 bf16x8;
#define LAS __attribute__((address_space(3)))
#define GAS __attribute__((address_space(1)))
typedef GAS unsigned gu32;
typedef GAS unsigned long long gu64;

constexpr int NR = 8;
constexpr int NSTEP = 64;
constexpr int N_PUSH = 56;
constexpr int N_PROD = N_PUSH + 8;
constexpr int N_GEMM = 128;
constexpr int GRID = N_PROD + N_GEMM;
constexpr int NTHREADS = 576;
#define AG_WIN 2
#define AG_T0 130
#define AG_PSLEEP 1
#define AG_STAG 2
#define AG_EDELAY 250
#define AG_XPF 0
#define AG_PUB 2
#define AG_PTAIL 2
#define AG_GAP 64
#define AG_MARK0 300
#define AG_MARKD 8
#define AG_LHI 85
#define AG_DINC 2
#define AG_DDEC 1
#define AG_OCAP 80
#define AG_ODELTA 270
#define AG_TAU 685
#define AG_FSTR 128
constexpr unsigned FSTR = AG_FSTR;
#define AG_GSTR 16
constexpr unsigned GSTR = AG_GSTR;
#define AG_NSPEC 16
constexpr int NSPEC = AG_NSPEC;
constexpr int WIN = AG_WIN;
constexpr unsigned long long DEADLINE = 100000000ull;

constexpr unsigned OFF_CTR = 0;
constexpr unsigned OFF_TMO = 0x8000;
constexpr unsigned OFF_FILL = 0x9000;
constexpr unsigned OFF_FLAG = 0xE000;
constexpr unsigned OFF_AMAX = 0x20000;
constexpr unsigned OFF_DBG = 0x10000;
constexpr unsigned UNIT_BYTES = 64u * 8192u;
constexpr unsigned OFF_ST = 1u << 20;
constexpr unsigned ST_PIECE = 5u * 1024u, ST_STEP = 8u * ST_PIECE, ST_BYTES = 64u * ST_STEP;
constexpr unsigned WS_NEEDED = OFF_ST + 8u * ST_BYTES;
constexpr unsigned SHARD_BYTES = 4u << 20;
constexpr unsigned OUT_BYTES = 4096u * 512u;
static_assert((NSTEP - NSPEC - WIN) % AG_PUB == 0 && NSPEC + WIN >= AG_PTAIL - 1 && GRID * 128 <= (int)OFF_TMO && GSTR == 16, "workspace map");

constexpr unsigned SLOT = 32768, NSLOT = 4;
constexpr unsigned L_CTRL = SLOT * NSLOT;
constexpr unsigned L_FULL = L_CTRL, L_FREE = L_CTRL + 16, L_AVAIL = L_CTRL + 32, L_ABORT = L_CTRL + 44, L_CALL = L_CTRL + 48,
                   L_WMAX = L_CTRL + 64, L_GMAX = L_CTRL + 96;
constexpr unsigned LDS_BYTES = L_CTRL + 128;

struct Params {
    const char* x;
    const char* w;
    char* ws[NR];
    char* out[NR];
    int rank;
};

__device__ __forceinline__ unsigned rfl(unsigned v) { return __builtin_amdgcn_readfirstlane(v); }
__device__ __forceinline__ u32x4 make_srd(const void* p, unsigned bytes) {
    const unsigned long long a = (unsigned long long)p;
    u32x4 r;
    r.x = rfl((unsigned)a); r.y = rfl((unsigned)(a >> 32) & 0xffffu); r.z = rfl(bytes); r.w = 0x00020000u;
    return r;
}
__device__ __forceinline__ void dma16(unsigned lds, unsigned voff, u32x4 srd, unsigned soff) {
    asm volatile("s_mov_b32 m0, %0\n\ts_nop 0\n\tbuffer_load_dwordx4 %1, %2, %3 offen lds"
                 :: "s"(lds), "v"(voff), "s"(srd), "s"(soff) : "memory");
}
__device__ __forceinline__ void dma16_sys(unsigned lds, unsigned voff, u32x4 srd, unsigned soff) {
    asm volatile("s_mov_b32 m0, %0\n\ts_nop 0\n\tbuffer_load_dwordx4 %1, %2, %3 offen sc0 sc1 lds"
                 :: "s"(lds), "v"(voff), "s"(srd), "s"(soff) : "memory");
}
typedef __amdgpu_buffer_rsrc_t rsrc_t;
__device__ __forceinline__ rsrc_t make_rsrc(const void* p, unsigned bytes) {
    const unsigned long long a = (unsigned long long)p;
    const unsigned long long u = ((unsigned long long)rfl((unsigned)(a >> 32)) << 32) | rfl((unsigned)a);
    return __builtin_amdgcn_make_buffer_rsrc((void*)u, 0, rfl(bytes), 0x00020000);
}
__device__ __forceinline__ u32x4 bld16(unsigned voff, rsrc_t r, unsigned soff) { return __builtin_amdgcn_raw_buffer_load_b128(r, voff, soff, 0); }
__device__ __forceinline__ void bst16_sys(u32x4 v, unsigned voff, rsrc_t r, unsigned soff) {
    __builtin_amdgcn_raw_buffer_store_b128(v, r, voff, soff, 17);
}
#define WAIT_VM(n) asm volatile("s_waitcnt vmcnt(" #n ")" ::: "memory")
#define WAIT_LGKM0() asm volatile("s_waitcnt lgkmcnt(0)" ::: "memory")
#define BARRIER() do { asm volatile("s_waitcnt lgkmcnt(0)" ::: "memory"); __builtin_amdgcn_s_barrier(); asm volatile("" ::: "memory"); } while (0)
__device__ __forceinline__ unsigned long long now() { return __builtin_amdgcn_s_memrealtime(); }
__device__ __forceinline__ unsigned ld_sys(const void* p) { return __hip_atomic_load((gu32*)p, __ATOMIC_RELAXED, __HIP_MEMORY_SCOPE_SYSTEM); }
__device__ __forceinline__ void st_sys(void* p, unsigned v) { __hip_atomic_store((gu32*)p, v, __ATOMIC_RELAXED, __HIP_MEMORY_SCOPE_SYSTEM); }
__device__ __forceinline__ unsigned ld_agent(const void* p) { return __hip_atomic_load((gu32*)p, __ATOMIC_RELAXED, __HIP_MEMORY_SCOPE_AGENT); }
__device__ __forceinline__ void st_agent(void* p, unsigned v) { __hip_atomic_store((gu32*)p, v, __ATOMIC_RELAXED, __HIP_MEMORY_SCOPE_AGENT); }
__device__ __forceinline__ unsigned draw_ticket(char* ws, int b) {
    unsigned z = 0;
    asm volatile("" : "+v"(z));
    return __hip_atomic_fetch_add((gu32*)(ws + OFF_CTR + b * 128 + z), 1u, __ATOMIC_RELAXED, __HIP_MEMORY_SCOPE_AGENT);
}
__device__ __forceinline__ unsigned wave_max_u32(unsigned v) {
#define DPP_MAX(ctrl, rmask) do { const unsigned o_ = (unsigned)__builtin_amdgcn_update_dpp(0, (int)v, ctrl, rmask, 0xf, false); v = o_ > v ? o_ : v; } while (0)
    DPP_MAX(0x111, 0xf); DPP_MAX(0x112, 0xf); DPP_MAX(0x114, 0xf); DPP_MAX(0x118, 0xf);
    DPP_MAX(0x142, 0xa); DPP_MAX(0x143, 0xc);
#undef DPP_MAX
    return __builtin_amdgcn_readlane(v, 63);
}

#define STAMP(i) do { } while (0)
#define STORE_DBG(i, v) do { } while (0)
#define TIC() do { } while (0)
#define TOC(acc) do { } while (0)

__global__ void __launch_bounds__(NTHREADS, 3) ag_gemm_q8(Params P) {
    __shared__ __attribute__((aligned(16))) char smem[LDS_BYTES];
    LAS char* const L = (LAS char*)smem;
    const int b = blockIdx.x;
    const int tid = threadIdx.x, lane = tid & 63;
    const int wave = (int)rfl(tid >> 6);
    const unsigned rank = (unsigned)P.rank & 7u;
    const unsigned grp = rank >> 2, me = rank & 3u, og = (grp ^ 1u) * 4u;
    char* const myws = P.ws[rank];

    volatile LAS unsigned* const ctl = (volatile LAS unsigned*)(L + L_CTRL);
    auto unit_src = [&](unsigned u, unsigned& a, unsigned& lu) {
        if (u < 20u) { const unsigned i = u / 5u; a = grp * 4u + i; lu = u - 5u * i; }
        else { const unsigned i = (u - 20u) / 3u; a = og + i; lu = 5u + (u - 20u) - 3u * i; }
        a &= 7u;
    };
    auto min4 = [](unsigned v, int l) {
        const unsigned p0 = __builtin_amdgcn_readlane(v, l), p1 = __builtin_amdgcn_readlane(v, l + 1),
                       p2 = __builtin_amdgcn_readlane(v, l + 2), p3 = __builtin_amdgcn_readlane(v, l + 3);
        const unsigned m0 = p0 < p1 ? p0 : p1, m1 = p2 < p3 ? p2 : p3;
        return m0 < m1 ? m0 : m1;
    };
    if (b < N_PUSH) {
        if (wave > 5) return;
        if (wave == 0) STAMP(0);
        const unsigned link = (unsigned)b >> 3, p = (unsigned)b & 7u, st = (unsigned)wave;
        const bool intra = link < 3u, isw = !intra && st >= 3u;
        const unsigned di = intra ? (me + 1u + link) & 3u : (me + link - 3u) & 3u;
        const unsigned dr = ((intra ? grp * 4u : og) + di) & 7u;
        char* const dws = P.ws[dr];
        const unsigned lu = intra ? st : 5u + st;
        const unsigned pw = 8u * (st - 3u) + p;
        const rsrc_t ssrd = make_rsrc(isw ? P.w : P.x, SHARD_BYTES);
        const rsrc_t dsrd = make_rsrc(dws + OFF_ST + rank * ST_BYTES, ST_BYTES);
        const unsigned svoff = isw ? (4u * pw + (lane >> 4)) * 1024u + ((((unsigned)lane & 15u) ^ (((unsigned)lane >> 4) << 2) ^ (pw & 3u)) << 4) + di * 256u
                                   : (lu * 64u + 8u * p + (lane >> 3)) * 8192u + ((((unsigned)lane & 7u) ^ ((unsigned)lane >> 4) ^ ((p & 1u) << 2)) << 4);
        const unsigned dvoff = p * ST_PIECE + st * 1024u + lane * 16u;
        const unsigned sstr = isw ? 65536u : 128u, dstr = ST_STEP;
        char* const flag = dws + OFF_FLAG + (rank * 8u + p) * FSTR;
        auto publish_at = [](int done) { return done > 0 && (done % AG_PUB == 0 || done > NSTEP - AG_PTAIL); };
        if (wave == 5) {
            unsigned tk = 0;
            if (lane == 0) tk = draw_ticket(myws, b);
            const unsigned call = rfl(tk) + 1u, base = call * 64u;
            const char* fillp = lane == 0 ? myws + OFF_FILL + dr * 128u : myws + OFF_TMO;
#pragma unroll 1
            for (int done = 1; done <= NSTEP; ++done) {
                if (!publish_at(done)) continue;
                __builtin_amdgcn_s_barrier();
                if (NSPEC > 0 && done == NSTEP - NSPEC - WIN - AG_PUB) {
                    const unsigned long long t0 = now();
                    for (;;) {
                        const unsigned v = ld_sys(fillp);
                        if (__builtin_amdgcn_readlane(v, 0) >= call) break;
                        if (__builtin_amdgcn_readlane(v, 1) == call || now() - t0 > DEADLINE) { if (lane == 0) st_agent(myws + OFF_TMO, call); break; }
                    }
                }
                if (lane == 0) st_sys(flag, base + (unsigned)done);
            }
            STAMP(1);
            return;
        }
        u32x4 v[WIN];
        int floor_t = -1000, off = 0, tis[WIN];
        [[maybe_unused]] int latsum = 0, latmax = 0;
        const unsigned long long tp = now() + (AG_STAG == 1 ? (st * 8u + p) : (AG_STAG == 2 ? p * 5u + st : ((p + link) & 7u) * 5u + st)) * AG_TAU / 400u;
#pragma unroll
        for (int i = 0; i < WIN; ++i) v[i] = bld16(svoff, ssrd, i * sstr);
        asm volatile("" ::: "memory");
#pragma unroll
        for (int i = 0; i < NSTEP; ++i) {
            if (i >= WIN) {
                const int younger = (WIN - 1) + ((NSTEP - 1 - i) < (WIN - 1) ? (NSTEP - 1 - i) : (WIN - 1));
                asm volatile("s_waitcnt vmcnt(%0)" :: "i"(younger) : "memory");
            }
            if (i < WIN) asm volatile("" : "+v"(v[i % WIN]));
            {
                int tn = (int)(now() - tp);
                if (AG_LHI > 0 && i >= WIN) {
                    const int lat = tn - tis[i % WIN];
                    if (lat > AG_LHI) { if (off < AG_OCAP) off += AG_DINC; } else if (off >= AG_DDEC) off -= AG_DDEC;
                }
                const int sched = (i - WIN + 1) * AG_TAU / 10 + AG_T0 + off;
                const int due = sched > floor_t ? sched : floor_t;
                while (tn < due) { __builtin_amdgcn_s_sleep(1); tn = (int)(now() - tp); }
                floor_t = tn + AG_GAP;
                tis[i % WIN] = tn;
            }
            bst16_sys(v[i % WIN], dvoff, dsrd, i * dstr);
            if (i + WIN < NSTEP) v[i % WIN] = bld16(svoff, ssrd, (i + WIN) * sstr);
            asm volatile("" ::: "memory");
            if (publish_at(i - WIN + 1)) __builtin_amdgcn_s_barrier();
        }
#pragma unroll
        for (int done = NSTEP - WIN + 1; done <= NSTEP; ++done) {
            asm volatile("s_waitcnt vmcnt(%0)" :: "i"(NSTEP - done) : "memory");
            __builtin_amdgcn_s_barrier();
        }
        if (wave == 0) { STORE_DBG(2, off); STORE_DBG(3, latsum); STORE_DBG(4, latmax); }
        return;
    }

    if (b < N_PROD) {
        const unsigned src = (rank + 1u + (unsigned)(b - N_PUSH)) & 7u;
        if (NSPEC == 0 || wave >= 8 || src == rank) return;
        unsigned tk = 0;
        if (tid == 0) tk = draw_ticket(myws, b);
        const rsrc_t dsrd = make_rsrc(myws + OFF_ST + src * ST_BYTES + (NSTEP - NSPEC) * ST_STEP, NSPEC * ST_STEP);
        const u32x4 ones = {~0u, ~0u, ~0u, ~0u};
#pragma unroll 1
        for (unsigned i = 0; i < NSPEC * 5u; ++i) {
            bst16_sys(ones, tid * 16u, dsrd, i * 8192u);
            WAIT_VM(2);
        }
        asm volatile("s_waitcnt vmcnt(0)" : "+v"(tk) :: "memory");
        __builtin_amdgcn_s_barrier();
        if (tid == 0) st_sys(P.ws[src] + OFF_FILL + rank * 128u, tk + 1u);
        return;
    }

    const unsigned gi = (unsigned)b - N_PROD;
    const unsigned t = ((gi & 7u) >> 1) * 4u + ((gi >> 3) & 3u), c = (gi & 1u) * 2u + ((gi >> 5) & 1u) + (gi >> 6) * 4u;
    unsigned a0, a1, lu0, lu1;
    unit_src(2u * t, a0, lu0); unit_src(2u * t + 1u, a1, lu1);
    const bool own0 = a0 == rank, own1 = a1 == rank;
    if (tid < 32) ctl[tid] = ((tid == 8 && own0) || (tid == 9 && own1) || (tid == 10 && c < 4u)) ? (unsigned)NSTEP : 0u;
    BARRIER();
    if (wave == 8) STAMP(0);

    auto hrow = [&](unsigned u) {
        const unsigned i5 = u / 5u, i3 = (u - 20u) / 3u;
        return u < 20u ? (og + i5) * 512u + (u - 5u * i5) * 64u : (grp * 4u + i3) * 512u + (5u + u - 20u - 3u * i3) * 64u;
    };
    const unsigned hr0 = hrow(2u * t), hr1 = hrow(2u * t + 1u);
    f32x16 acc[2][2];
    if (wave < 4) {
        const unsigned wm = (unsigned)wave >> 1, wn = (unsigned)wave & 1u;
        const unsigned h = (unsigned)lane >> 5;
        unsigned xa[4], wa[2][2];
#pragma unroll
        for (int ks = 0; ks < 4; ++ks) xa[ks] = (wm * 64u + (lane & 31u)) * 128u + (((2u * ks + h) ^ (((unsigned)lane >> 1) & 7u)) << 4);
        {
            const unsigned blk = ((unsigned)lane >> 4) & 1u, q = ((unsigned)lane & 15u) >> 2, pp = (unsigned)lane & 3u;
#pragma unroll
            for (int nt = 0; nt < 2; ++nt)
#pragma unroll
                for (int tt = 0; tt < 2; ++tt) {
                    const unsigned row = 8u * h + 4u * tt + q, ch = 4u * (2u * wn + nt) + 2u * blk + (pp >> 1);
                    wa[nt][tt] = 16384u + 256u * row + ((ch ^ (((row & 3u) << 2) | ((row >> 2) & 3u))) << 4) + 8u * (pp & 1u);
                }
        }
#pragma unroll
        for (int i = 0; i < 2; ++i)
#pragma unroll
            for (int jj = 0; jj < 2; ++jj)
#pragma unroll
                for (int e = 0; e < 16; ++e) acc[i][jj][e] = 0.f;
        char* lanews = P.ws[lane & 7];
        asm volatile("" : "+v"(lanews));
        unsigned have = 0;
        [[maybe_unused]] unsigned long long dw = 0;
        s16x8 xf[4][2], wf[4][2];
        auto rd = [&](LAS char* S, int ks) {
#pragma unroll
            for (int nt = 0; nt < 2; ++nt) {
                const s16x4 lo = __builtin_amdgcn_ds_read_tr16_b64_v4i16((LAS s16x4*)(S + wa[nt][0] + ks * 4096));
                const s16x4 hi = __builtin_amdgcn_ds_read_tr16_b64_v4i16((LAS s16x4*)(S + wa[nt][1] + ks * 4096));
                wf[ks][nt] = __builtin_shufflevector(lo, hi, 0, 1, 2, 3, 4, 5, 6, 7);
            }
#pragma unroll
            for (int mt = 0; mt < 2; ++mt) xf[ks][mt] = *(LAS s16x8*)(S + xa[ks] + mt * 4096);
            __builtin_amdgcn_sched_barrier(0);
        };
        auto mm = [&](int ks) {
#pragma unroll
            for (int nt = 0; nt < 2; ++nt)
#pragma unroll
                for (int mt = 0; mt < 2; ++mt)
                    acc[nt][mt] = __builtin_amdgcn_mfma_f32_32x32x16_bf16((bf16x8)wf[ks][nt], (bf16x8)xf[ks][mt], acc[nt][mt], 0, 0, 0);
            __builtin_amdgcn_sched_barrier(0);
        };
        auto look = [&]() {
            const u32x4 f = *(volatile LAS u32x4*)(L + L_FULL);
            const unsigned fa = wm ? f.y : f.x, fb = f.z < f.w ? f.z : f.w;
            have = rfl(fa < fb ? fa : fb);
        };
        __builtin_amdgcn_s_waitcnt(0xC07F);
        bool pre = false;
#pragma unroll 1
        for (int k4 = 0; k4 < NSTEP; k4 += NSLOT) {
#pragma unroll
            for (int sl = 0; sl < (int)NSLOT; ++sl) {
                const unsigned k = k4 + sl;
                LAS char* const S = L + sl * SLOT;
                if (!AG_XPF || !pre) {
                    TIC();
                    while (have <= k) { look(); if (have <= k) __builtin_amdgcn_s_sleep(1); }
                    TOC(dw);
                    asm volatile("" ::: "memory");
                    rd(S, 0);
                }
                rd(S, 1); mm(0); rd(S, 2); mm(1); rd(S, 3);
                if (AG_XPF && have <= k + 1u) look();
                mm(2);
                __builtin_amdgcn_s_waitcnt(0xC07F);
                asm volatile("" ::: "memory");
                ctl[4 + wave] = k + 1;
                __builtin_amdgcn_sched_barrier(0);
                if (AG_XPF) {
                    pre = have > k + 1u && k + 1u < (unsigned)NSTEP;
                    if (pre) rd(L + ((sl + 1) & (NSLOT - 1)) * SLOT, 0);
                }
                mm(3);
            }
        }
        float m = 0.f;
#pragma unroll
        for (int i = 0; i < 2; ++i)
#pragma unroll
            for (int jj = 0; jj < 2; ++jj)
#pragma unroll
                for (int e = 0; e < 16; ++e) m = fmaxf(m, fabsf(acc[i][jj][e]));
        const unsigned mu = wave_max_u32(__float_as_uint(m));
        unsigned arr = 0;
        if (lane == 0) {
            __hip_atomic_fetch_max((LAS unsigned*)(L + L_WMAX), mu, __ATOMIC_RELAXED, __HIP_MEMORY_SCOPE_WORKGROUP);
            arr = __hip_atomic_fetch_add((LAS unsigned*)(L + L_WMAX + 4), 1u, __ATOMIC_RELAXED, __HIP_MEMORY_SCOPE_WORKGROUP);
        }
        if (rfl(arr) == 3u) {
            const unsigned mm = ctl[16];
            unsigned cl;
            while ((cl = rfl(ctl[13])) == 0u) { }
            if (lane < NR)
                __hip_atomic_store((gu64*)(lanews + OFF_AMAX + (rank * 128u + gi) * GSTR), ((unsigned long long)cl << 32) | mm,
                                   __ATOMIC_RELAXED, __HIP_MEMORY_SCOPE_SYSTEM);
        }
        if (wave == 0) { STAMP(2); STORE_DBG(5, dw); }
    } else if (wave < 8) {
        const unsigned li = (unsigned)wave - 4u, hf = li & 1u;
        const bool isx = li < 2u;
        const bool own = isx ? (hf ? own1 : own0) : (c < 4u);
        const unsigned luh = hf ? lu1 : lu0;
        const unsigned src = isx ? (hf ? a1 : a0) : og + (c & 3u), stn = isx ? (luh < 5u ? luh : luh - 5u) : 3u + hf;
        const char* base; unsigned bytes;
        if (!own) { base = myws + OFF_ST + src * ST_BYTES + stn * 1024u; bytes = ST_BYTES - stn * 1024u; }
        else if (isx) { base = P.x + luh * UNIT_BYTES; bytes = UNIT_BYTES; }
        else { base = P.w + c * 256u + hf * 32768u; bytes = SHARD_BYTES - c * 256u - hf * 32768u; }
        const u32x4 srd = make_srd(base, bytes);
        const unsigned lin = lane * 16u;
        const unsigned swx = (lane >> 3) * 8192u + ((((unsigned)lane & 7u) ^ ((unsigned)lane >> 4)) << 4);
        const unsigned sww = (lane >> 4) * 1024u + ((((unsigned)lane & 15u) ^ (((unsigned)lane >> 4) << 2)) << 4);
        unsigned vo[4];
#pragma unroll
        for (int i = 0; i < 4; ++i) vo[i] = own ? (isx ? (swx ^ ((i & 1) << 6)) : (sww ^ (i << 4))) : lin;
        const unsigned pstr = own ? (isx ? 65536u : 4096u) : ST_PIECE;
        const unsigned sstr = own ? (isx ? 128u : 65536u) : ST_STEP;
        const unsigned avw = isx ? 8u + hf : 10u;
        unsigned avail = 0, freed = 0, pub = 0;
        const unsigned nnorm = own ? (unsigned)NSTEP : (unsigned)(NSTEP - NSPEC);
        [[maybe_unused]] unsigned long long da = 0, df = 0, dv = 0;
        asm volatile("s_nop 4" ::: "memory");
#pragma unroll 1
        for (unsigned j = 0; j < nnorm; ++j) {
            { TIC();
            if (avail <= j) {
                avail = rfl(ctl[avw]);
                if (avail <= j) {
                    WAIT_VM(0);
                    if (pub < j) { pub = j; ctl[li] = pub; }
                    do { __builtin_amdgcn_s_sleep(1); avail = rfl(ctl[avw]); } while (avail <= j);
                }
            }
            TOC(da); }
            { TIC();
            while (freed + NSLOT <= j) {
                const u32x4 f = *(volatile LAS u32x4*)(L + L_FREE);
                const unsigned m0 = f.x < f.y ? f.x : f.y, m1 = f.z < f.w ? f.z : f.w;
                freed = rfl(isx ? (hf ? m1 : m0) : (m0 < m1 ? m0 : m1));
                if (freed + NSLOT <= j) __builtin_amdgcn_s_sleep(1);
            }
            TOC(df); }
            const unsigned dst = (unsigned)(size_t)(L + (j & (NSLOT - 1)) * SLOT) + li * 8192u;
#pragma unroll
            for (int p = 0; p < 8; ++p) dma16(dst + p * 1024u, vo[p & 3], srd, j * sstr + p * pstr);
            { TIC(); WAIT_VM(16); TOC(dv); }
            if (j >= 2 && pub < j - 1) { pub = j - 1; ctl[li] = pub; }
        }
        WAIT_VM(8); ctl[li] = nnorm - 1u;
        auto look_free = [&]() {
            const u32x4 f = *(volatile LAS u32x4*)(L + L_FREE);
            const unsigned m0 = f.x < f.y ? f.x : f.y, m1 = f.z < f.w ? f.z : f.w;
            freed = rfl(isx ? (hf ? m1 : m0) : (m0 < m1 ? m0 : m1));
        };
        auto landed = [&](unsigned j) {
            const LAS u32x4* q = (const LAS u32x4*)(L + (j & (NSLOT - 1)) * SLOT + li * 8192u + lin);
            unsigned m = 0;
#pragma unroll
            for (int p = 0; p < 8; ++p) { const u32x4 v = q[p * 64]; m = max(max(m, v.x), max(max(v.y, v.z), v.w)); }
            return !__any(m == ~0u);
        };
        pub = nnorm - 1u;
#pragma unroll 1
        for (unsigned j = nnorm; j < (unsigned)NSTEP; j = pub) {
            look_free();
            if (freed + NSLOT <= j) {
                WAIT_VM(0);
                if (pub < j) { pub = j; ctl[li] = pub; }
                do { __builtin_amdgcn_s_sleep(1); look_free(); } while (freed + NSLOT <= j);
            }
            avail = rfl(ctl[avw]);
            const bool two = j + 1u < (unsigned)NSTEP && freed + NSLOT > j + 1u;
            const unsigned d0 = (unsigned)(size_t)(L + (j & (NSLOT - 1)) * SLOT) + li * 8192u;
            const unsigned d1 = (unsigned)(size_t)(L + ((j + 1u) & (NSLOT - 1)) * SLOT) + li * 8192u;
#pragma unroll
            for (int p = 0; p < 8; ++p) dma16_sys(d0 + p * 1024u, lin, srd, j * ST_STEP + p * ST_PIECE);
            if (two) {
#pragma unroll
                for (int p = 0; p < 8; ++p) dma16_sys(d1 + p * 1024u, lin, srd, (j + 1u) * ST_STEP + p * ST_PIECE);
                WAIT_VM(16);
            } else WAIT_VM(8);
            if (pub < j) { pub = j; ctl[li] = pub; }
            if (two) WAIT_VM(8); else WAIT_VM(0);
            if (avail > j || landed(j)) {
                pub = j + 1u; ctl[li] = pub;
                WAIT_VM(0);
                if (two && (avail > j + 1u || landed(j + 1u))) { pub = j + 2u; ctl[li] = pub; }
            }
            WAIT_VM(0);
        }
        WAIT_VM(0); ctl[li] = NSTEP;
        STORE_DBG(8 + li * 3, da); STORE_DBG(9 + li * 3, df); STORE_DBG(10 + li * 3, dv);
    } else {
        __builtin_amdgcn_fence(__ATOMIC_ACQUIRE, "");
        unsigned tk = 0;
        if (lane == 0) tk = draw_ticket(myws, b);
        const unsigned call = rfl(tk) + 1u;
        WAIT_VM(0);
        if (lane == 0) ctl[12] = call;
        const unsigned fsrc = lane < 8 ? a0 : (lane < 16 ? a1 : og + (c & 3u));
        const char* fp = lane < 24 ? myws + OFF_FLAG + (fsrc * 8u + (lane & 7)) * FSTR : myws + OFF_TMO;
        const rsrc_t msrd = make_rsrc(P.out[rank], OUT_BYTES);
        const u32x4 mark = {0x80808080u, 0x80808080u, 0x80808080u, 0x80808080u};
        unsigned nmark = c < 4u ? 0u : 16u;
        auto put_mark = [&]() {
            bst16_sys(mark, ((unsigned)lane >> 3) * 512u + ((unsigned)lane & 7u) * 16u, msrd, (((nmark & 8u) ? hr1 : hr0) + (nmark & 7u) * 8u) * 512u + c * 128u);
            ++nmark;
        };
        const unsigned long long t0 = now();
        [[maybe_unused]] unsigned npoll = 0;
        for (;;) {
            if (nmark < 16u && (unsigned)(now() - t0) > AG_MARK0 + (nmark * 16u + (gi & 15u)) * AG_MARKD) put_mark();
            if (nmark == 16u) { WAIT_VM(0); if (lane == 0) ctl[13] = call; nmark = 17u; }
            const unsigned v = ld_sys(fp);
            const int d = (int)(v - call * 64u);
            const unsigned prog = d < 0 ? 0u : (d > NSTEP ? (unsigned)NSTEP : (unsigned)d);
            auto min8 = [&](int l) { const unsigned m0 = min4(prog, l), m1 = min4(prog, l + 4); return m0 < m1 ? m0 : m1; };
            unsigned av0 = own0 ? (unsigned)NSTEP : min8(0), av1 = own1 ? (unsigned)NSTEP : min8(8), avb = c < 4u ? (unsigned)NSTEP : min8(16);
            const bool dead = __builtin_amdgcn_readlane(v, 24) == call || now() - t0 > DEADLINE;
            if (dead) { if (lane == 0) { st_agent(myws + OFF_TMO, call); ctl[11] = 1u; } av0 = av1 = avb = NSTEP; }
            if (lane == 0) { ctl[8] = av0; ctl[9] = av1; ctl[10] = avb; }
            if (av0 == (unsigned)NSTEP && av1 == (unsigned)NSTEP && avb == (unsigned)NSTEP) break;
            const u32x4 fl = *(volatile LAS u32x4*)(L + L_FULL);
            if (rfl(fl.x + fl.y + fl.z + fl.w) == 4u * NSTEP) break;
            __builtin_amdgcn_s_sleep(AG_PSLEEP);
        }
        while (nmark < 16u) {
            if ((unsigned)(now() - t0) > AG_MARK0 + (nmark * 16u + (gi & 15u)) * AG_MARKD || rfl(ctl[4]) > 48u) put_mark(); else __builtin_amdgcn_s_sleep(8);
        }
        if (nmark == 16u) { WAIT_VM(0); if (lane == 0) ctl[13] = call; }
        STORE_DBG(6, npoll);
        STAMP(1);
    }

    if (wave >= 8) return;
    if (wave < 4) {
        for (;;) {
            const u32x4 f = *(volatile LAS u32x4*)(L + L_FREE);
            const unsigned m0 = f.x < f.y ? f.x : f.y, m1 = f.z < f.w ? f.z : f.w;
            if (rfl(m0 < m1 ? m0 : m1) >= (unsigned)NSTEP - 2u) break;
        }
        asm volatile("" ::: "memory");
        const unsigned wm = (unsigned)wave >> 1, wn = (unsigned)wave & 1u, h = (unsigned)lane >> 5;
#pragma unroll
        for (int nt = 0; nt < 2; ++nt)
#pragma unroll
            for (int mt = 0; mt < 2; ++mt)
#pragma unroll
                for (int g = 0; g < 4; ++g) {
                    const unsigned m = wm * 64u + mt * 32u + (lane & 31u), ch = 16u * wn + 8u * nt + 2u * g + h;
                    f32x4 q; q.x = acc[nt][mt][4 * g]; q.y = acc[nt][mt][4 * g + 1]; q.z = acc[nt][mt][4 * g + 2]; q.w = acc[nt][mt][4 * g + 3];
                    *(LAS f32x4*)(L + m * 512u + ((ch ^ (m & 7u)) << 4)) = q;
                }
    }
    const unsigned orank = (c < 4u ? rank : og + (c & 3u)) & 7u;
    const unsigned ocol = (c < 4u ? c : me) * 128u;
    char* optr = P.out[orank];
    asm volatile("" : "+s"(optr));
    unsigned call = 0, mx = 0;
    bool got = false;
    const char* gp = myws + OFF_AMAX + ((unsigned)tid & 255u) * GSTR;
    const unsigned own_g = rank * 128u + gi;
    const unsigned skip = (own_g & 255u) == ((unsigned)tid & 255u) ? own_g >> 8 : 4u;
    unsigned long long tg0 = 0;
    bool ok = false;
    auto sweep = [&]() {
        const unsigned tm = ld_agent(myws + OFF_TMO);
        if (!ok) {
            unsigned long long g[4];
#pragma unroll
            for (int j = 0; j < 4; ++j) g[j] = __hip_atomic_load((gu64*)(gp + j * 256 * GSTR), __ATOMIC_RELAXED, __HIP_MEMORY_SCOPE_SYSTEM);
            ok = true; mx = 0;
#pragma unroll
            for (int j = 0; j < 4; ++j) {
                const bool mine = skip == (unsigned)j;
                ok = ok && (mine || (unsigned)(g[j] >> 32) >= call);
                mx = max(mx, mine ? 0u : (unsigned)g[j]);
            }
        }
        if (rfl(tm) == call || now() - tg0 > DEADLINE) { if (lane == 0) { st_agent(myws + OFF_TMO, call); ctl[11] = 1u; } return true; }
        return (bool)__all(ok);
    };
    if (wave >= 4) {
        while ((call = rfl(ctl[12])) == 0u) { }
        tg0 = now();
        while (!got && rfl(ctl[17]) < 4u) got = sweep();
    }
    BARRIER();
    f32x4 y[2][4];
    const unsigned r0 = (unsigned)tid >> 3, q8 = (unsigned)tid & 7u;
#pragma unroll
    for (int hf = 0; hf < 2; ++hf)
#pragma unroll
        for (int e = 0; e < 4; ++e) y[hf][e] = *(LAS f32x4*)(L + (r0 + 64u * hf) * 512u + (((4u * q8 + e) ^ (r0 & 7u)) << 4));
    if (wave >= 4) {
        while (!got) got = sweep();
        mx = wave_max_u32(mx);
        if (lane == 0) ctl[20 + wave] = mx;
    }
    BARRIER();
    if (wave == 5) STAMP(3);
    const unsigned long long tq = now();
    const u32x4 gmx = *(volatile LAS u32x4*)(L + L_GMAX);
    const unsigned ga = max(max(gmx.x, gmx.y), max(gmx.z, gmx.w)), gb = ctl[16];
    const bool aborted = ctl[11] != 0u;
    const float amax = __uint_as_float(ga > gb ? ga : gb);
    const float inv = amax > 0.f ? 127.0f / amax : 0.f;
    const rsrc_t osrd = make_rsrc(optr, OUT_BYTES);
    const unsigned ovoff = r0 * 512u + q8 * 16u;
    constexpr float MAGIC = 12582912.0f;
#pragma unroll
    for (int hf = 0; hf < 2; ++hf) {
        u32x4 o;
#pragma unroll
        for (int e = 0; e < 4; ++e) {
            unsigned z[4];
#pragma unroll
            for (int i = 0; i < 4; ++i)
                z[i] = __float_as_uint(__builtin_amdgcn_fmed3f(__builtin_fmaf(y[hf][e][i], inv, MAGIC), MAGIC - 127.0f, MAGIC + 127.0f));
            const unsigned lo = __builtin_amdgcn_perm(z[1], z[0], 0x0c0c0400u), hi = __builtin_amdgcn_perm(z[3], z[2], 0x04000c0cu);
            o[e] = aborted ? 0x80808080u : (lo | hi);
        }
        const unsigned row = hf ? a1 * 512u + lu1 * 64u : a0 * 512u + lu0 * 64u;
        if (c < 4u) __builtin_amdgcn_raw_buffer_store_b128(o, osrd, ovoff, row * 512u + ocol, 0);
        else {
            const unsigned due = ((unsigned)(hf * 8 + wave) * 16u + t) * AG_ODELTA / 160u;
            asm volatile("" : "+v"(o));
            while ((unsigned)(now() - tq) < due) __builtin_amdgcn_s_sleep(1);
            bst16_sys(o, ovoff, osrd, row * 512u + ocol);
        }
    }
    if (c < 4u) {
        call = rfl(ctl[12]);
        while (now() - tq < AG_EDELAY) __builtin_amdgcn_s_sleep(2);
        bool in = false;
        for (;;) {
            const unsigned tm = ld_agent(myws + OFF_TMO);
            if (!in) {
                const u32x4 h0 = __builtin_amdgcn_raw_buffer_load_b128(osrd, ovoff, hr0 * 512u + ocol, 17);
                const u32x4 h1 = __builtin_amdgcn_raw_buffer_load_b128(osrd, ovoff, hr1 * 512u + ocol, 17);
                unsigned bad = 0;
#pragma unroll
                for (int e = 0; e < 4; ++e) {
                    const unsigned x0 = h0[e] ^ 0x80808080u, x1 = h1[e] ^ 0x80808080u;
                    bad |= ((x0 - 0x01010101u) & ~x0) | ((x1 - 0x01010101u) & ~x1);
                }
                in = (bad & 0x80808080u) == 0u;
            }
            if (__all(in)) break;
            if (rfl(tm) == call || now() - tq > DEADLINE) { if (lane == 0) st_agent(myws + OFF_TMO, call); break; }
            asm volatile("" ::: "memory");
        }
    }
    if (wave == 7) STAMP(4);
}

extern "C" void kernel_launch(int rank, int nranks, ncclComm_t comm,
                              hipStream_t stream,
                              void* const* d_in, const int* in_sizes, int n_in,
                              void* d_out, int out_size,
                              const int* in_dtypes, int out_dtype,
                              void* const* d_out_all,
                              void* const* d_ws_all, size_t ws_bytes) {
    (void)comm; (void)in_dtypes; (void)out_dtype; (void)d_out;
    if (nranks != NR || (unsigned)rank >= (unsigned)NR || n_in < 2 || ws_bytes < WS_NEEDED ||
        in_sizes[0] != 512 * 4096 || in_sizes[1] != 4096 * 512 || out_size != (int)OUT_BYTES) {
        fprintf(stderr, "kernel_launch: unexpected problem shape\n");
        return;
    }
    Params P;
    P.x = (const char*)d_in[0];
    P.w = (const char*)d_in[1];
    for (int p = 0; p < NR; ++p) { P.ws[p] = (char*)d_ws_all[p]; P.out[p] = (char*)d_out_all[p]; }
    P.rank = rank;
    hipLaunchKernelGGL(ag_gemm_q8, dim3(GRID), dim3(NTHREADS), 0, stream, P);
    HIP_CHECK(hipGetLastError());
}
```

```cpp
#include <hip/hip_runtime.h>
#include <rccl/rccl.h>
#include <cstdio>
#include <cstdint>

#define HIP_CHECK(expr) do { hipError_t _e = (expr); if (_e != hipSuccess) { \
    fprintf(stderr, "HIP error %s at %s:%d\n", hipGetErrorString(_e), __FILE__, __LINE__); \
    return; } } while (0)

typedef __attribute__((ext_vector_type(4))) unsigned u32x4;
typedef __attribute__((ext_vector_type(4))) float f32x4;
typedef __attribute__((ext_vector_type(16))) float f32x16;
typedef __attribute__((ext_vector_type(4))) short s16x4;
typedef __attribute__((ext_vector_type(8))) short s16x8;
typedef __attribute__((ext_vector_type(8))) __bf16 bf16x8;
#define LAS __attribute__((address_space(3)))
#define GAS __attribute__((address_space(1)))
typedef GAS unsigned gu32;
typedef GAS unsigned long long gu64;

constexpr int NR = 8;
constexpr int NSTEP = 64;
constexpr int N_PUSH = 56;
constexpr int N_PROD = N_PUSH + 8;
constexpr int N_GEMM = 128;
constexpr int GRID = N_PROD + N_GEMM;
constexpr int NTHREADS = 576;
constexpr int PUSH_T0 = 30, PUSH_TAU_NS = 685;
constexpr int PUSH_GAP = 64;
constexpr int LAT_HI = 85, OFF_INC = 2, OFF_DEC = 1, OFF_CAP = 80;
constexpr unsigned OUT_DELTA_NS = 270;
constexpr unsigned OUT_CHECK_DELAY = 250;
constexpr unsigned MARK_T0 = 300, MARK_DELTA = 8;
constexpr unsigned FSTR = 128, GSTR = 16;
constexpr int NSPEC = 24;
constexpr unsigned long long DEADLINE = 100000000ull;

constexpr unsigned OFF_CTR = 0;
constexpr unsigned OFF_TMO = 0x8000;
constexpr unsigned OFF_FILL = 0x9000;
constexpr unsigned OFF_FLAG = 0xE000;
constexpr unsigned OFF_AMAX = 0x10000;
constexpr unsigned AMAX_BYTES = NR * N_GEMM * GSTR;
constexpr unsigned UNIT_BYTES = 64u * 8192u;
constexpr unsigned OFF_ST = 1u << 20;
constexpr unsigned ST_PIECE = 5u * 1024u, ST_STEP = 8u * ST_PIECE, ST_BYTES = 64u * ST_STEP;
constexpr unsigned WS_NEEDED = OFF_ST + 8u * ST_BYTES;
constexpr unsigned SHARD_BYTES = 4u << 20;
constexpr unsigned OUT_BYTES = 4096u * 512u;
static_assert(NSPEC % 2 == 0 && NSPEC <= NSTEP - 6 && GRID * 128 <= (int)OFF_TMO && OFF_AMAX + 2 * AMAX_BYTES <= OFF_ST, "workspace map");

constexpr unsigned SLOT = 32768, NSLOT = 4;
constexpr unsigned L_CTRL = SLOT * NSLOT;
constexpr unsigned L_FULL = L_CTRL, L_FREE = L_CTRL + 16, L_WMAX = L_CTRL + 64, L_GMAX = L_CTRL + 96;
constexpr unsigned LDS_BYTES = L_CTRL + 128;

struct alignas(32) LinkArgs {
    const char* x;
    const char* wq;
    char* dst;
    unsigned long long wq_bytes;
};
struct Params {
    LinkArgs link[NR];
    const char* x;
    const char* w;
    char* ws[NR];
    char* out[NR];
    int rank;
};

__host__ __device__ __forceinline__ unsigned link_rank(unsigned rank, unsigned l) {
    return l < 3u ? (rank & 4u) | ((rank + 1u + l) & 3u) : ((rank & 4u) ^ 4u) | ((rank + l - 3u) & 3u);
}
__device__ __forceinline__ unsigned rfl(unsigned v) { return __builtin_amdgcn_readfirstlane(v); }
__device__ __forceinline__ u32x4 make_srd(const void* p, unsigned bytes) {
    const unsigned long long a = (unsigned long long)p;
    u32x4 r;
    r.x = rfl((unsigned)a); r.y = rfl((unsigned)(a >> 32) & 0xffffu); r.z = rfl(bytes); r.w = 0x00020000u;
    return r;
}
__device__ __forceinline__ void dma16(unsigned lds, unsigned voff, u32x4 srd, unsigned soff) {
    asm volatile("s_mov_b32 m0, %0\n\ts_nop 0\n\tbuffer_load_dwordx4 %1, %2, %3 offen lds"
                 :: "s"(lds), "v"(voff), "s"(srd), "s"(soff) : "memory");
}
__device__ __forceinline__ void dma16_sys(unsigned lds, unsigned voff, u32x4 srd, unsigned soff) {
    asm volatile("s_mov_b32 m0, %0\n\ts_nop 0\n\tbuffer_load_dwordx4 %1, %2, %3 offen sc0 sc1 lds"
                 :: "s"(lds), "v"(voff), "s"(srd), "s"(soff) : "memory");
}
typedef __amdgpu_buffer_rsrc_t rsrc_t;
__device__ __forceinline__ rsrc_t make_rsrc(const void* p, unsigned bytes) {
    const unsigned long long a = (unsigned long long)p;
    const unsigned long long u = ((unsigned long long)rfl((unsigned)(a >> 32)) << 32) | rfl((unsigned)a);
    return __builtin_amdgcn_make_buffer_rsrc((void*)u, 0, rfl(bytes), 0x00020000);
}
__device__ __forceinline__ u32x4 bld16(unsigned voff, rsrc_t r, unsigned soff) { return __builtin_amdgcn_raw_buffer_load_b128(r, voff, soff, 0); }
__device__ __forceinline__ void bst16_sys(u32x4 v, unsigned voff, rsrc_t r, unsigned soff) {
    __builtin_amdgcn_raw_buffer_store_b128(v, r, voff, soff, 17);
}
#define WAIT_VM(n) asm volatile("s_waitcnt vmcnt(" #n ")" ::: "memory")
#define BARRIER() do { asm volatile("s_waitcnt lgkmcnt(0)" ::: "memory"); __builtin_amdgcn_s_barrier(); asm volatile("" ::: "memory"); } while (0)
__device__ __forceinline__ unsigned long long now() { return __builtin_amdgcn_s_memrealtime(); }
__device__ __forceinline__ unsigned ld_sys(const void* p) { return __hip_atomic_load((gu32*)p, __ATOMIC_RELAXED, __HIP_MEMORY_SCOPE_SYSTEM); }
__device__ __forceinline__ void st_sys(void* p, unsigned v) { __hip_atomic_store((gu32*)p, v, __ATOMIC_RELAXED, __HIP_MEMORY_SCOPE_SYSTEM); }
__device__ __forceinline__ unsigned ld_agent(const void* p) { return __hip_atomic_load((gu32*)p, __ATOMIC_RELAXED, __HIP_MEMORY_SCOPE_AGENT); }
__device__ __forceinline__ void st_agent(void* p, unsigned v) { __hip_atomic_store((gu32*)p, v, __ATOMIC_RELAXED, __HIP_MEMORY_SCOPE_AGENT); }
__device__ __forceinline__ unsigned draw_ticket(char* ws, int b) {
    unsigned z = 0;
    asm volatile("" : "+v"(z));
    return __hip_atomic_fetch_add((gu32*)(ws + OFF_CTR + b * 128 + z), 1u, __ATOMIC_RELAXED, __HIP_MEMORY_SCOPE_AGENT);
}
__device__ __forceinline__ unsigned wave_max_u32(unsigned v) {
#define DPP_MAX(ctrl, rmask) do { const unsigned o_ = (unsigned)__builtin_amdgcn_update_dpp(0, (int)v, ctrl, rmask, 0xf, false); v = o_ > v ? o_ : v; } while (0)
    DPP_MAX(0x111, 0xf); DPP_MAX(0x112, 0xf); DPP_MAX(0x114, 0xf); DPP_MAX(0x118, 0xf);
    DPP_MAX(0x142, 0xa); DPP_MAX(0x143, 0xc);
#undef DPP_MAX
    return __builtin_amdgcn_readlane(v, 63);
}

__global__ void __launch_bounds__(NTHREADS, 3) ag_gemm_q8(Params P) {
    __shared__ __attribute__((aligned(16))) char smem[LDS_BYTES];
    LAS char* const L = (LAS char*)smem;
    const unsigned long long ttop = now();
    const int b = blockIdx.x;
    const int tid = threadIdx.x, lane = tid & 63;
    const int wave = (int)rfl(tid >> 6);
    if (__builtin_expect(b < N_PUSH && wave < 5, 1)) {
        const unsigned link = (unsigned)b >> 3, p = (unsigned)b & 7u, st = (unsigned)wave;
        const LinkArgs A = P.link[link];
        const bool intra = link < 3u, isw = !intra && st >= 3u;
        const unsigned lu = intra ? st : 5u + st;
        const unsigned pw = 8u * (st - 3u) + p;
        const rsrc_t ssrd = make_rsrc(isw ? A.wq : A.x, isw ? (unsigned)A.wq_bytes : SHARD_BYTES);
        const rsrc_t dsrd = make_rsrc(A.dst, ST_BYTES);
        const unsigned svoff = isw ? (4u * pw + (lane >> 4)) * 1024u + ((((unsigned)lane & 15u) ^ (((unsigned)lane >> 4) << 2) ^ (pw & 3u)) << 4)
                                   : (lu * 64u + 8u * p + (lane >> 3)) * 8192u + ((((unsigned)lane & 7u) ^ ((unsigned)lane >> 4) ^ ((p & 1u) << 2)) << 4);
        const unsigned dvoff = p * ST_PIECE + st * 1024u + lane * 16u;
        const unsigned sstr = isw ? 65536u : 128u;
        u32x4 v0 = bld16(svoff, ssrd, 0), v1 = bld16(svoff, ssrd, sstr);
        constexpr int TAU8 = PUSH_TAU_NS * 8 / 10;
        int sched = PUSH_T0 * 8 + (int)(p * 5u + st) * TAU8 / 40, floor_t = 0, off = 0, ti0 = 0, ti1 = 0;
        auto clock8 = [&]() { return (int)((unsigned)(now() - ttop) << 3); };
        auto pace = [&](int& ti, bool acked) {
            int tn = clock8();
            if (acked) {
                const int lat = tn - ti;
                if (lat > LAT_HI * 8) { if (off < OFF_CAP * 8) off += OFF_INC * 8; } else if (off >= OFF_DEC * 8) off -= OFF_DEC * 8;
            }
            const int due = sched + off > floor_t ? sched + off : floor_t;
            while (tn < due) { __builtin_amdgcn_s_sleep(1); tn = clock8(); }
            floor_t = tn + PUSH_GAP * 8;
            sched += TAU8;
            ti = tn;
        };
        asm volatile("" : "+v"(v0));
        pace(ti0, false);
        bst16_sys(v0, dvoff, dsrd, 0); v0 = bld16(svoff, ssrd, 2u * sstr);
        pace(ti1, false);
        bst16_sys(v1, dvoff, dsrd, ST_STEP); v1 = bld16(svoff, ssrd, 3u * sstr);
#pragma unroll 1
        for (unsigned i = 2; i < (unsigned)NSTEP; i += 2) {
            WAIT_VM(2);
            pace(ti0, true);
            bst16_sys(v0, dvoff, dsrd, i * ST_STEP);
            v0 = bld16(svoff, ssrd, (i + 2u < (unsigned)NSTEP ? i + 2u : i) * sstr);
            WAIT_VM(2);
            pace(ti1, true);
            bst16_sys(v1, dvoff, dsrd, (i + 1u) * ST_STEP);
            v1 = bld16(svoff, ssrd, (i + 3u < (unsigned)NSTEP ? i + 3u : i) * sstr);
            asm volatile("" ::: "memory");
            __builtin_amdgcn_s_barrier();
        }
        WAIT_VM(0);
        __builtin_amdgcn_s_barrier();
        asm volatile("" :: "v"(v0), "v"(v1));
        return;
    }
    const unsigned rank = (unsigned)P.rank & 7u;
    const unsigned grp = rank >> 2, me = rank & 3u, og = (grp ^ 1u) * 4u;
    char* const myws = P.ws[rank];

    volatile LAS unsigned* const ctl = (volatile LAS unsigned*)(L + L_CTRL);
    auto unit_src = [&](unsigned u, unsigned& a, unsigned& lu) {
        if (u < 20u) { const unsigned i = u / 5u; a = grp * 4u + i; lu = u - 5u * i; }
        else { const unsigned i = (u - 20u) / 3u; a = og + i; lu = 5u + (u - 20u) - 3u * i; }
        a &= 7u;
    };
    auto min4 = [](unsigned v, int l) {
        const unsigned p0 = __builtin_amdgcn_readlane(v, l), p1 = __builtin_amdgcn_readlane(v, l + 1),
                       p2 = __builtin_amdgcn_readlane(v, l + 2), p3 = __builtin_amdgcn_readlane(v, l + 3);
        const unsigned m0 = p0 < p1 ? p0 : p1, m1 = p2 < p3 ? p2 : p3;
        return m0 < m1 ? m0 : m1;
    };
    if (b < N_PUSH) {
        if (wave > 5) return;
        const unsigned link = (unsigned)b >> 3, p = (unsigned)b & 7u;
        const unsigned dr = link_rank(rank, link);
        char* const flag = P.ws[dr] + OFF_FLAG + (rank * 8u + p) * FSTR;
        unsigned tk = 0;
        if (lane == 0) tk = draw_ticket(myws, b);
        const unsigned call = rfl(tk) + 1u, base = call * 64u;
        const char* fillp = lane == 0 ? myws + OFF_FILL + dr * 128u : myws + OFF_TMO;
#pragma unroll 1
        for (int done = 2; done <= NSTEP; done += 2) {
            __builtin_amdgcn_s_barrier();
            if (done == NSTEP - NSPEC - 4) {
                const unsigned long long t0 = now();
                for (;;) {
                    const unsigned v = ld_sys(fillp);
                    if (__builtin_amdgcn_readlane(v, 0) >= call) break;
                    if (__builtin_amdgcn_readlane(v, 1) == call || now() - t0 > DEADLINE) { if (lane == 0) st_agent(myws + OFF_TMO, call); break; }
                }
            }
            const bool pub = done <= NSTEP - NSPEC ? (done % 4 == 0 || done > NSTEP - NSPEC - 12) : done == NSTEP;
            if (lane == 0 && pub) st_sys(flag, base + (unsigned)done);
        }
        return;
    }

    if (b < N_PROD) {
        const unsigned src = (rank + 1u + (unsigned)(b - N_PUSH)) & 7u;
        if (wave >= 8 || src == rank) return;
        unsigned tk = 0;
        if (tid == 0) tk = draw_ticket(myws, b);
        const rsrc_t dsrd = make_rsrc(myws + OFF_ST + src * ST_BYTES + (NSTEP - NSPEC) * ST_STEP, NSPEC * ST_STEP);
        const u32x4 ones = {~0u, ~0u, ~0u, ~0u};
#pragma unroll 1
        for (unsigned i = 0; i < NSPEC * 5u; ++i) {
            bst16_sys(ones, tid * 16u, dsrd, i * 8192u);
            WAIT_VM(8);
        }
        asm volatile("s_waitcnt vmcnt(0)" : "+v"(tk) :: "memory");
        __builtin_amdgcn_s_barrier();
        if (tid == 0) st_sys(P.ws[src] + OFF_FILL + rank * 128u, tk + 1u);
        return;
    }

    const unsigned gi = (unsigned)b - N_PROD;
    const unsigned t = ((gi & 7u) >> 1) * 4u + ((gi >> 3) & 3u), c = (gi & 1u) * 2u + ((gi >> 5) & 1u) + (gi >> 6) * 4u;
    unsigned a0, a1, lu0, lu1;
    unit_src(2u * t, a0, lu0); unit_src(2u * t + 1u, a1, lu1);
    const bool own0 = a0 == rank, own1 = a1 == rank;
    if (tid < 32) ctl[tid] = ((tid == 8 && own0) || (tid == 9 && own1) || (tid == 10 && c < 4u)) ? (unsigned)NSTEP : 0u;
    BARRIER();

    auto hrow = [&](unsigned u) {
        const unsigned i5 = u / 5u, i3 = (u - 20u) / 3u;
        return u < 20u ? (og + i5) * 512u + (u - 5u * i5) * 64u : (grp * 4u + i3) * 512u + (5u + u - 20u - 3u * i3) * 64u;
    };
    const unsigned hr0 = hrow(2u * t), hr1 = hrow(2u * t + 1u);
    f32x16 acc[2][2];
    if (wave < 4) {
        const unsigned wm = (unsigned)wave >> 1, wn = (unsigned)wave & 1u;
        const unsigned h = (unsigned)lane >> 5;
        unsigned xa[4], wa[2][2];
#pragma unroll
        for (int ks = 0; ks < 4; ++ks) xa[ks] = (wm * 64u + (lane & 31u)) * 128u + (((2u * ks + h) ^ (((unsigned)lane >> 1) & 7u)) << 4);
        {
            const unsigned blk = ((unsigned)lane >> 4) & 1u, q = ((unsigned)lane & 15u) >> 2, pp = (unsigned)lane & 3u;
#pragma unroll
            for (int nt = 0; nt < 2; ++nt)
#pragma unroll
                for (int tt = 0; tt < 2; ++tt) {
                    const unsigned row = 8u * h + 4u * tt + q, ch = 4u * (2u * wn + nt) + 2u * blk + (pp >> 1);
                    wa[nt][tt] = 16384u + 256u * row + ((ch ^ (((row & 3u) << 2) | ((row >> 2) & 3u))) << 4) + 8u * (pp & 1u);
                }
        }
#pragma unroll
        for (int i = 0; i < 2; ++i)
#pragma unroll
            for (int jj = 0; jj < 2; ++jj)
#pragma unroll
                for (int e = 0; e < 16; ++e) acc[i][jj][e] = 0.f;
        char* lanews = P.ws[lane & 7];
        asm volatile("" : "+v"(lanews));
        unsigned have = 0;
        s16x8 xf[4][2], wf[4][2];
        auto rd = [&](LAS char* S, int ks) {
#pragma unroll
            for (int nt = 0; nt < 2; ++nt) {
                const s16x4 lo = __builtin_amdgcn_ds_read_tr16_b64_v4i16((LAS s16x4*)(S + wa[nt][0] + ks * 4096));
                const s16x4 hi = __builtin_amdgcn_ds_read_tr16_b64_v4i16((LAS s16x4*)(S + wa[nt][1] + ks * 4096));
                wf[ks][nt] = __builtin_shufflevector(lo, hi, 0, 1, 2, 3, 4, 5, 6, 7);
            }
#pragma unroll
            for (int mt = 0; mt < 2; ++mt) xf[ks][mt] = *(LAS s16x8*)(S + xa[ks] + mt * 4096);
            __builtin_amdgcn_sched_barrier(0);
        };
        auto mm = [&](int ks) {
#pragma unroll
            for (int nt = 0; nt < 2; ++nt)
#pragma unroll
                for (int mt = 0; mt < 2; ++mt)
                    acc[nt][mt] = __builtin_amdgcn_mfma_f32_32x32x16_bf16((bf16x8)wf[ks][nt], (bf16x8)xf[ks][mt], acc[nt][mt], 0, 0, 0);
            __builtin_amdgcn_sched_barrier(0);
        };
        __builtin_amdgcn_s_waitcnt(0xC07F);
#pragma unroll 1
        for (int k4 = 0; k4 < NSTEP; k4 += NSLOT) {
#pragma unroll
            for (int sl = 0; sl < (int)NSLOT; ++sl) {
                const unsigned k = k4 + sl;
                LAS char* const S = L + sl * SLOT;
                while (have <= k) {
                    const u32x4 f = *(volatile LAS u32x4*)(L + L_FULL);
                    const unsigned fa = wm ? f.y : f.x, fb = f.z < f.w ? f.z : f.w;
                    have = rfl(fa < fb ? fa : fb);
                    if (have <= k) __builtin_amdgcn_s_sleep(1);
                }
                asm volatile("" ::: "memory");
                rd(S, 0); rd(S, 1); mm(0); rd(S, 2); mm(1); rd(S, 3); mm(2);
                __builtin_amdgcn_s_waitcnt(0xC07F);
                asm volatile("" ::: "memory");
                ctl[4 + wave] = k + 1;
                __builtin_amdgcn_sched_barrier(0);
                mm(3);
            }
        }
        float m = 0.f;
#pragma unroll
        for (int i = 0; i < 2; ++i)
#pragma unroll
            for (int jj = 0; jj < 2; ++jj)
#pragma unroll
                for (int e = 0; e < 16; ++e) m = fmaxf(m, fabsf(acc[i][jj][e]));
        const unsigned mu = wave_max_u32(__float_as_uint(m));
        unsigned arr = 0;
        if (lane == 0) {
            __hip_atomic_fetch_max((LAS unsigned*)(L + L_WMAX), mu, __ATOMIC_RELAXED, __HIP_MEMORY_SCOPE_WORKGROUP);
            arr = __hip_atomic_fetch_add((LAS unsigned*)(L + L_WMAX + 4), 1u, __ATOMIC_RELAXED, __HIP_MEMORY_SCOPE_WORKGROUP);
        }
        if (rfl(arr) == 3u) {
            const unsigned mm = ctl[16];
            unsigned cl;
            while ((cl = rfl(ctl[13])) == 0u) { }
            if (lane < NR)
                __hip_atomic_store((gu64*)(lanews + OFF_AMAX + (cl & 1u) * AMAX_BYTES + (rank * 128u + gi) * GSTR), ((unsigned long long)cl << 32) | mm,
                                   __ATOMIC_RELAXED, __HIP_MEMORY_SCOPE_SYSTEM);
        }
    } else if (wave < 8) {
        const unsigned li = (unsigned)wave - 4u, hf = li & 1u;
        const bool isx = li < 2u;
        const bool own = isx ? (hf ? own1 : own0) : (c < 4u);
        const unsigned luh = hf ? lu1 : lu0;
        const unsigned src = isx ? (hf ? a1 : a0) : og + (c & 3u), stn = isx ? (luh < 5u ? luh : luh - 5u) : 3u + hf;
        const char* base; unsigned bytes;
        if (!own) { base = myws + OFF_ST + src * ST_BYTES + stn * 1024u; bytes = ST_BYTES - stn * 1024u; }
        else if (isx) { base = P.x + luh * UNIT_BYTES; bytes = UNIT_BYTES; }
        else { base = P.w + c * 256u + hf * 32768u; bytes = SHARD_BYTES - c * 256u - hf * 32768u; }
        const u32x4 srd = make_srd(base, bytes);
        const unsigned lin = lane * 16u;
        const unsigned swx = (lane >> 3) * 8192u + ((((unsigned)lane & 7u) ^ ((unsigned)lane >> 4)) << 4);
        const unsigned sww = (lane >> 4) * 1024u + ((((unsigned)lane & 15u) ^ (((unsigned)lane >> 4) << 2)) << 4);
        unsigned vo[4];
#pragma unroll
        for (int i = 0; i < 4; ++i) vo[i] = isx ? (swx ^ ((i & 1) << 6)) : (sww ^ (i << 4));
        const unsigned pstr = isx ? 65536u : 4096u, sstr = isx ? 128u : 65536u;
        const unsigned avw = isx ? 8u + hf : 10u;
        unsigned avail = 0, freed = 0, pub = 0;
        const unsigned nnorm = own ? (unsigned)NSTEP : (unsigned)(NSTEP - NSPEC);
        auto look_free = [&]() {
            const u32x4 f = *(volatile LAS u32x4*)(L + L_FREE);
            const unsigned m0 = f.x < f.y ? f.x : f.y, m1 = f.z < f.w ? f.z : f.w;
            freed = rfl(isx ? (hf ? m1 : m0) : (m0 < m1 ? m0 : m1));
        };
        asm volatile("s_nop 4" ::: "memory");
#pragma unroll 1
        for (unsigned j = 0; j < nnorm; ++j) {
            if (avail <= j) {
                avail = rfl(ctl[avw]);
                if (avail <= j) {
                    WAIT_VM(0);
                    if (pub < j) { pub = j; ctl[li] = pub; }
                    do { __builtin_amdgcn_s_sleep(1); avail = rfl(ctl[avw]); } while (avail <= j);
                }
            }
            while (freed + NSLOT <= j) { look_free(); if (freed + NSLOT <= j) __builtin_amdgcn_s_sleep(1); }
            const unsigned dst = (unsigned)(size_t)(L + (j & (NSLOT - 1)) * SLOT) + li * 8192u;
            if (own) {
#pragma unroll
                for (int p = 0; p < 8; ++p) dma16(dst + p * 1024u, vo[p & 3], srd, j * sstr + p * pstr);
            } else {
#pragma unroll
                for (int p = 0; p < 8; ++p) dma16_sys(dst + p * 1024u, lin, srd, j * ST_STEP + p * ST_PIECE);
            }
            WAIT_VM(16);
            if (j >= 2 && pub < j - 1) { pub = j - 1; ctl[li] = pub; }
        }
        WAIT_VM(8); ctl[li] = nnorm - 1u;
        auto landed = [&](unsigned j) {
            const LAS u32x4* q = (const LAS u32x4*)(L + (j & (NSLOT - 1)) * SLOT + li * 8192u + lin);
            unsigned m = 0;
#pragma unroll
            for (int p = 0; p < 8; ++p) { const u32x4 v = q[p * 64]; m = max(max(m, v.x), max(max(v.y, v.z), v.w)); }
            return !__any(m == ~0u);
        };
        pub = nnorm - 1u;
#pragma unroll 1
        for (unsigned j = nnorm; j < (unsigned)NSTEP; j = pub) {
            look_free();
            if (freed + NSLOT <= j) {
                WAIT_VM(0);
                if (pub < j) { pub = j; ctl[li] = pub; }
                do { __builtin_amdgcn_s_sleep(1); look_free(); } while (freed + NSLOT <= j);
            }
            avail = rfl(ctl[avw]);
            const bool two = j + 1u < (unsigned)NSTEP && freed + NSLOT > j + 1u;
            const unsigned d0 = (unsigned)(size_t)(L + (j & (NSLOT - 1)) * SLOT) + li * 8192u;
            const unsigned d1 = (unsigned)(size_t)(L + ((j + 1u) & (NSLOT - 1)) * SLOT) + li * 8192u;
#pragma unroll
            for (int p = 0; p < 8; ++p) dma16_sys(d0 + p * 1024u, lin, srd, j * ST_STEP + p * ST_PIECE);
            if (two) {
#pragma unroll
                for (int p = 0; p < 8; ++p) dma16_sys(d1 + p * 1024u, lin, srd, (j + 1u) * ST_STEP + p * ST_PIECE);
                WAIT_VM(16);
            } else WAIT_VM(8);
            if (pub < j) { pub = j; ctl[li] = pub; }
            if (two) WAIT_VM(8); else WAIT_VM(0);
            if (avail > j || landed(j)) {
                pub = j + 1u; ctl[li] = pub;
                WAIT_VM(0);
                if (two && (avail > j + 1u || landed(j + 1u))) { pub = j + 2u; ctl[li] = pub; }
            }
            WAIT_VM(0);
        }
        WAIT_VM(0); ctl[li] = NSTEP;
    } else {
        unsigned tk = 0;
        if (lane == 0) tk = draw_ticket(myws, b);
        const unsigned call = rfl(tk) + 1u;
        WAIT_VM(0);
        if (lane == 0) ctl[12] = call;
        const unsigned fsrc = lane < 8 ? a0 : (lane < 16 ? a1 : og + (c & 3u));
        const char* fp = lane < 24 ? myws + OFF_FLAG + (fsrc * 8u + (lane & 7)) * FSTR : myws + OFF_TMO;
        const rsrc_t msrd = make_rsrc(P.out[rank], OUT_BYTES);
        const u32x4 mark = {0x80808080u, 0x80808080u, 0x80808080u, 0x80808080u};
        unsigned nmark = c < 4u ? 0u : 16u;
        auto put_mark = [&]() {
            bst16_sys(mark, ((unsigned)lane >> 3) * 512u + ((unsigned)lane & 7u) * 16u, msrd, (((nmark & 8u) ? hr1 : hr0) + (nmark & 7u) * 8u) * 512u + c * 128u);
            ++nmark;
        };
        const unsigned long long t0 = now();
        for (;;) {
            if (nmark < 16u && (unsigned)(now() - t0) > MARK_T0 + (nmark * 16u + (gi & 15u)) * MARK_DELTA) put_mark();
            if (nmark == 16u) { WAIT_VM(0); if (lane == 0) ctl[13] = call; nmark = 17u; }
            const unsigned v = ld_sys(fp);
            const int d = (int)(v - call * 64u);
            const unsigned prog = d < 0 ? 0u : (d > NSTEP ? (unsigned)NSTEP : (unsigned)d);
            auto min8 = [&](int l) { const unsigned m0 = min4(prog, l), m1 = min4(prog, l + 4); return m0 < m1 ? m0 : m1; };
            unsigned av0 = own0 ? (unsigned)NSTEP : min8(0), av1 = own1 ? (unsigned)NSTEP : min8(8), avb = c < 4u ? (unsigned)NSTEP : min8(16);
            const bool dead = __builtin_amdgcn_readlane(v, 24) == call || now() - t0 > DEADLINE;
            if (dead) { if (lane == 0) { st_agent(myws + OFF_TMO, call); ctl[11] = 1u; } av0 = av1 = avb = NSTEP; }
            if (lane == 0) { ctl[8] = av0; ctl[9] = av1; ctl[10] = avb; }
            if (av0 == (unsigned)NSTEP && av1 == (unsigned)NSTEP && avb == (unsigned)NSTEP) break;
            const u32x4 fl = *(volatile LAS u32x4*)(L + L_FULL);
            if (rfl(fl.x + fl.y + fl.z + fl.w) == 4u * NSTEP) break;
            __builtin_amdgcn_s_sleep(1);
        }
        while (nmark < 16u) {
            if ((unsigned)(now() - t0) > MARK_T0 + (nmark * 16u + (gi & 15u)) * MARK_DELTA || rfl(ctl[4]) > 48u) put_mark(); else __builtin_amdgcn_s_sleep(8);
        }
        if (nmark == 16u) { WAIT_VM(0); if (lane == 0) ctl[13] = call; }
    }

    if (wave >= 8) return;
    if (wave < 4) {
        for (;;) {
            const u32x4 f = *(volatile LAS u32x4*)(L + L_FREE);
            const unsigned m0 = f.x < f.y ? f.x : f.y, m1 = f.z < f.w ? f.z : f.w;
            if (rfl(m0 < m1 ? m0 : m1) >= (unsigned)NSTEP - 2u) break;
        }
        asm volatile("" ::: "memory");
        const unsigned wm = (unsigned)wave >> 1, wn = (unsigned)wave & 1u, h = (unsigned)lane >> 5;
#pragma unroll
        for (int nt = 0; nt < 2; ++nt)
#pragma unroll
            for (int mt = 0; mt < 2; ++mt)
#pragma unroll
                for (int g = 0; g < 4; ++g) {
                    const unsigned m = wm * 64u + mt * 32u + (lane & 31u), ch = 16u * wn + 8u * nt + 2u * g + h;
                    f32x4 q; q.x = acc[nt][mt][4 * g]; q.y = acc[nt][mt][4 * g + 1]; q.z = acc[nt][mt][4 * g + 2]; q.w = acc[nt][mt][4 * g + 3];
                    *(LAS f32x4*)(L + m * 512u + ((ch ^ (m & 7u)) << 4)) = q;
                }
    }
    const unsigned orank = (c < 4u ? rank : og + (c & 3u)) & 7u;
    const unsigned ocol = (c < 4u ? c : me) * 128u;
    char* optr = P.out[orank];
    asm volatile("" : "+s"(optr));
    unsigned call = 0, mx = 0;
    bool got = false;
    const char* gp = myws + OFF_AMAX + ((unsigned)tid & 255u) * GSTR;
    const unsigned own_g = rank * 128u + gi;
    const unsigned skip = (own_g & 255u) == ((unsigned)tid & 255u) ? own_g >> 8 : 4u;
    unsigned long long tg0 = 0;
    bool ok = false;
    auto sweep = [&]() {
        const unsigned tm = ld_agent(myws + OFF_TMO);
        if (!ok) {
            unsigned long long g[4];
#pragma unroll
            for (int j = 0; j < 4; ++j) g[j] = __hip_atomic_load((gu64*)(gp + j * 256 * GSTR), __ATOMIC_RELAXED, __HIP_MEMORY_SCOPE_SYSTEM);
            ok = true; mx = 0;
#pragma unroll
            for (int j = 0; j < 4; ++j) {
                const bool mine = skip == (unsigned)j;
                ok = ok && (mine || (unsigned)(g[j] >> 32) >= call);
                mx = max(mx, mine ? 0u : (unsigned)g[j]);
            }
        }
        if (rfl(tm) == call || now() - tg0 > DEADLINE) { if (lane == 0) { st_agent(myws + OFF_TMO, call); ctl[11] = 1u; } return true; }
        return (bool)__all(ok);
    };
    if (wave >= 4) {
        while ((call = rfl(ctl[12])) == 0u) { }
        gp += (call & 1u) * AMAX_BYTES;
        tg0 = now();
        while (!got && rfl(ctl[17]) < 4u) got = sweep();
    }
    BARRIER();
    f32x4 y[2][4];
    const unsigned r0 = (unsigned)tid >> 3, q8 = (unsigned)tid & 7u;
#pragma unroll
    for (int hf = 0; hf < 2; ++hf)
#pragma unroll
        for (int e = 0; e < 4; ++e) y[hf][e] = *(LAS f32x4*)(L + (r0 + 64u * hf) * 512u + (((4u * q8 + e) ^ (r0 & 7u)) << 4));
    if (wave >= 4) {
        while (!got) got = sweep();
        mx = wave_max_u32(mx);
        if (lane == 0) ctl[20 + wave] = mx;
    }
    BARRIER();
    const unsigned long long tq = now();
    const u32x4 gmx = *(volatile LAS u32x4*)(L + L_GMAX);
    const unsigned ga = max(max(gmx.x, gmx.y), max(gmx.z, gmx.w)), gb = ctl[16];
    const bool aborted = ctl[11] != 0u;
    const float amax = __uint_as_float(ga > gb ? ga : gb);
    const float inv = amax > 0.f ? 127.0f / amax : 0.f;
    const rsrc_t osrd = make_rsrc(optr, OUT_BYTES);
    const unsigned ovoff = r0 * 512u + q8 * 16u;
    constexpr float MAGIC = 12582912.0f;
#pragma unroll
    for (int hf = 0; hf < 2; ++hf) {
        u32x4 o;
#pragma unroll
        for (int e = 0; e < 4; ++e) {
            unsigned z[4];
#pragma unroll
            for (int i = 0; i < 4; ++i)
                z[i] = __float_as_uint(__builtin_amdgcn_fmed3f(__builtin_fmaf(y[hf][e][i], inv, MAGIC), MAGIC - 127.0f, MAGIC + 127.0f));
            const unsigned lo = __builtin_amdgcn_perm(z[1], z[0], 0x0c0c0400u), hi = __builtin_amdgcn_perm(z[3], z[2], 0x04000c0cu);
            o[e] = aborted ? 0x80808080u : (lo | hi);
        }
        const unsigned row = hf ? a1 * 512u + lu1 * 64u : a0 * 512u + lu0 * 64u;
        if (c < 4u) __builtin_amdgcn_raw_buffer_store_b128(o, osrd, ovoff, row * 512u + ocol, 16);
        else {
            const unsigned due = ((unsigned)(hf * 8 + wave) * 16u + t) * OUT_DELTA_NS / 160u;
            asm volatile("" : "+v"(o));
            while ((unsigned)(now() - tq) < due) __builtin_amdgcn_s_sleep(1);
            bst16_sys(o, ovoff, osrd, row * 512u + ocol);
        }
    }
    if (c < 4u) {
        call = rfl(ctl[12]);
        while (now() - tq < OUT_CHECK_DELAY) __builtin_amdgcn_s_sleep(2);
        bool in = false;
        for (;;) {
            const unsigned tm = ld_agent(myws + OFF_TMO);
            if (!in) {
                const u32x4 h0 = __builtin_amdgcn_raw_buffer_load_b128(osrd, ovoff, hr0 * 512u + ocol, 17);
                const u32x4 h1 = __builtin_amdgcn_raw_buffer_load_b128(osrd, ovoff, hr1 * 512u + ocol, 17);
                unsigned bad = 0;
#pragma unroll
                for (int e = 0; e < 4; ++e) {
                    const unsigned x0 = h0[e] ^ 0x80808080u, x1 = h1[e] ^ 0x80808080u;
                    bad |= ((x0 - 0x01010101u) & ~x0) | ((x1 - 0x01010101u) & ~x1);
                }
                in = (bad & 0x80808080u) == 0u;
            }
            if (__all(in)) break;
            if (rfl(tm) == call || now() - tq > DEADLINE) { if (lane == 0) st_agent(myws + OFF_TMO, call); break; }
            asm volatile("" ::: "memory");
        }
    }
}

extern "C" void kernel_launch(int rank, int nranks, ncclComm_t comm,
                              hipStream_t stream,
                              void* const* d_in, const int* in_sizes, int n_in,
                              void* d_out, int out_size,
                              const int* in_dtypes, int out_dtype,
                              void* const* d_out_all,
                              void* const* d_ws_all, size_t ws_bytes) {
    (void)comm; (void)in_dtypes; (void)out_dtype; (void)d_out;
    if (nranks != NR || (unsigned)rank >= (unsigned)NR || n_in < 2 || ws_bytes < WS_NEEDED ||
        in_sizes[0] != 512 * 4096 || in_sizes[1] != 4096 * 512 || out_size != (int)OUT_BYTES) {
        fprintf(stderr, "kernel_launch: unexpected problem shape\n");
        return;
    }
    int dev = 0, cus = 0;
    HIP_CHECK(hipGetDevice(&dev));
    HIP_CHECK(hipDeviceGetAttribute(&cus, hipDeviceAttributeMultiprocessorCount, dev));
    if (cus < GRID) { fprintf(stderr, "kernel_launch: needs %d CUs\n", GRID); return; }
    Params P;
    P.x = (const char*)d_in[0];
    P.w = (const char*)d_in[1];
    for (int p = 0; p < NR; ++p) { P.ws[p] = (char*)d_ws_all[p]; P.out[p] = (char*)d_out_all[p]; }
    for (unsigned l = 0; l < (unsigned)NR; ++l) {
        const unsigned dr = link_rank((unsigned)rank, l < 7u ? l : 0u);
        P.link[l].x = P.x;
        P.link[l].wq = P.w + (dr & 3u) * 256u;
        P.link[l].wq_bytes = SHARD_BYTES - (dr & 3u) * 256u;
        P.link[l].dst = P.ws[dr] + OFF_ST + (unsigned)rank * ST_BYTES;
    }
    P.rank = rank;
    hipLaunchKernelGGL(ag_gemm_q8, dim3(GRID), dim3(NTHREADS), 0, stream, P);
    HIP_CHECK(hipGetLastError());
}
```

```cpp
#include <hip/hip_runtime.h>
#include <rccl/rccl.h>
#include <cstdio>
#include <cstdint>

#define HIP_CHECK(expr) do { hipError_t _e = (expr); if (_e != hipSuccess) { \
    fprintf(stderr, "HIP error %s at %s:%d\n", hipGetErrorString(_e), __FILE__, __LINE__); \
    return; } } while (0)

typedef __attribute__((ext_vector_type(4))) unsigned u32x4;
typedef __attribute__((ext_vector_type(4))) float f32x4;
typedef __attribute__((ext_vector_type(16))) float f32x16;
typedef __attribute__((ext_vector_type(4))) short s16x4;
typedef __attribute__((ext_vector_type(8))) short s16x8;
typedef __attribute__((ext_vector_type(8))) __bf16 bf16x8;
#define LAS __attribute__((address_space(3)))
#define GAS __attribute__((address_space(1)))
typedef GAS unsigned gu32;
typedef GAS unsigned long long gu64;

constexpr int NR = 8;
constexpr int NSTEP = 64;
constexpr int N_PROD = 64;
constexpr int N_GEMM = 128;
constexpr int GRID = N_PROD + N_GEMM;
constexpr int NTHREADS = 576;
constexpr int PUSH_T0 = 30, PUSH_TAU_NS = 682;
constexpr int PUSH_GAP = 64;
constexpr int LAT_HI = 85, OFF_INC = 2, OFF_DEC = 1, OFF_CAP = 80;
constexpr unsigned OUT_DELTA_NS = 270;
constexpr unsigned OUT_CHECK_DELAY = 60;
constexpr unsigned MARK_T0 = 300, MARK_DELTA = 8;
constexpr unsigned FSTR = 128, GSTR = 64;
constexpr int NSPEC = 24;
constexpr unsigned long long DEADLINE = 100000000ull;

constexpr unsigned OFF_CTR = 0;
constexpr unsigned OFF_TMO = 0x8000;
constexpr unsigned OFF_FILL = 0x9000;
constexpr unsigned OFF_FLAG = 0xE000;
constexpr unsigned OFF_AMAX = 0x10000;
constexpr unsigned AMAX_BYTES = NR * N_GEMM * GSTR;
constexpr unsigned UNIT_BYTES = 64u * 8192u;
constexpr unsigned OFF_ST = 1u << 20;
constexpr unsigned ST_PIECE = 5u * 1024u, ST_STEP = 8u * ST_PIECE, ST_BYTES = 64u * ST_STEP;
constexpr unsigned WS_NEEDED = OFF_ST + 8u * ST_BYTES;
constexpr unsigned SHARD_BYTES = 4u << 20;
constexpr unsigned OUT_BYTES = 4096u * 512u;
static_assert(NSPEC % 2 == 0 && NSPEC <= NSTEP - 6 && GRID * 128 <= (int)OFF_TMO && OFF_AMAX + 2 * AMAX_BYTES <= OFF_ST, "workspace map");

constexpr unsigned SLOT = 32768, NSLOT = 4;
constexpr unsigned L_CTRL = SLOT * NSLOT;
constexpr unsigned L_FULL = L_CTRL, L_FREE = L_CTRL + 16, L_WMAX = L_CTRL + 64, L_GMAX = L_CTRL + 96;
constexpr unsigned LDS_BYTES = L_CTRL + 128;

struct alignas(32) LinkArgs {
    const char* x;
    const char* wq;
    char* dst;
    unsigned long long wq_bytes;
};
struct Params {
    LinkArgs link[NR];
    const char* x;
    const char* w;
    char* ws[NR];
    char* out[NR];
    int rank;
};

__host__ __device__ __forceinline__ unsigned link_rank(unsigned rank, unsigned l) {
    return l < 3u ? (rank & 4u) | ((rank + 1u + l) & 3u) : ((rank & 4u) ^ 4u) | ((rank + l - 3u) & 3u);
}
__device__ __forceinline__ unsigned rfl(unsigned v) { return __builtin_amdgcn_readfirstlane(v); }
__device__ __forceinline__ u32x4 make_srd(const void* p, unsigned bytes) {
    const unsigned long long a = (unsigned long long)p;
    u32x4 r;
    r.x = rfl((unsigned)a); r.y = rfl((unsigned)(a >> 32) & 0xffffu); r.z = rfl(bytes); r.w = 0x00020000u;
    return r;
}
__device__ __forceinline__ void dma16(unsigned lds, unsigned voff, u32x4 srd, unsigned soff) {
    asm volatile("s_mov_b32 m0, %0\n\ts_nop 0\n\tbuffer_load_dwordx4 %1, %2, %3 offen lds"
                 :: "s"(lds), "v"(voff), "s"(srd), "s"(soff) : "memory");
}
__device__ __forceinline__ void dma16_sys(unsigned lds, unsigned voff, u32x4 srd, unsigned soff) {
    asm volatile("s_mov_b32 m0, %0\n\ts_nop 0\n\tbuffer_load_dwordx4 %1, %2, %3 offen sc0 sc1 lds"
                 :: "s"(lds), "v"(voff), "s"(srd), "s"(soff) : "memory");
}
typedef __amdgpu_buffer_rsrc_t rsrc_t;
__device__ __forceinline__ rsrc_t make_rsrc(const void* p, unsigned bytes) {
    const unsigned long long a = (unsigned long long)p;
    const unsigned long long u = ((unsigned long long)rfl((unsigned)(a >> 32)) << 32) | rfl((unsigned)a);
    return __builtin_amdgcn_make_buffer_rsrc((void*)u, 0, rfl(bytes), 0x00020000);
}
__device__ __forceinline__ u32x4 bld16(unsigned voff, rsrc_t r, unsigned soff) { return __builtin_amdgcn_raw_buffer_load_b128(r, voff, soff, 0); }
__device__ __forceinline__ void bst16_sys(u32x4 v, unsigned voff, rsrc_t r, unsigned soff) {
    __builtin_amdgcn_raw_buffer_store_b128(v, r, voff, soff, 17);
}
#define WAIT_VM(n) asm volatile("s_waitcnt vmcnt(" #n ")" ::: "memory")
#define BARRIER() do { asm volatile("s_waitcnt lgkmcnt(0)" ::: "memory"); __builtin_amdgcn_s_barrier(); asm volatile("" ::: "memory"); } while (0)
__device__ __forceinline__ unsigned long long now() { return __builtin_amdgcn_s_memrealtime(); }
__device__ __forceinline__ unsigned ld_sys(const void* p) { return __hip_atomic_load((gu32*)p, __ATOMIC_RELAXED, __HIP_MEMORY_SCOPE_SYSTEM); }
__device__ __forceinline__ void st_sys(void* p, unsigned v) { __hip_atomic_store((gu32*)p, v, __ATOMIC_RELAXED, __HIP_MEMORY_SCOPE_SYSTEM); }
__device__ __forceinline__ unsigned ld_agent(const void* p) { return __hip_atomic_load((gu32*)p, __ATOMIC_RELAXED, __HIP_MEMORY_SCOPE_AGENT); }
__device__ __forceinline__ void st_agent(void* p, unsigned v) { __hip_atomic_store((gu32*)p, v, __ATOMIC_RELAXED, __HIP_MEMORY_SCOPE_AGENT); }
__device__ __forceinline__ unsigned draw_ticket(char* ws, int b) {
    unsigned z = 0;
    asm volatile("" : "+v"(z));
    return __hip_atomic_fetch_add((gu32*)(ws + OFF_CTR + b * 128 + z), 1u, __ATOMIC_RELAXED, __HIP_MEMORY_SCOPE_AGENT);
}
__device__ __forceinline__ unsigned wave_max_u32(unsigned v) {
#define DPP_MAX(ctrl, rmask) do { const unsigned o_ = (unsigned)__builtin_amdgcn_update_dpp(0, (int)v, ctrl, rmask, 0xf, false); v = o_ > v ? o_ : v; } while (0)
    DPP_MAX(0x111, 0xf); DPP_MAX(0x112, 0xf); DPP_MAX(0x114, 0xf); DPP_MAX(0x118, 0xf);
    DPP_MAX(0x142, 0xa); DPP_MAX(0x143, 0xc);
#undef DPP_MAX
    return __builtin_amdgcn_readlane(v, 63);
}

__global__ void __launch_bounds__(NTHREADS, 3) ag_gemm_q8(Params P) {
    __shared__ __attribute__((aligned(16))) char smem[LDS_BYTES];
    LAS char* const L = (LAS char*)smem;
    const unsigned long long ttop = now();
    const int b = blockIdx.x;
    const int tid = threadIdx.x, lane = tid & 63;
    const int wave = (int)rfl(tid >> 6);
    const unsigned link = ((unsigned)b >> 4) + ((unsigned)b & 4u), p = (((unsigned)b >> 1) & 4u) | ((unsigned)b & 3u);
    const bool pusher = b < N_PROD && link < 7u;
    if (__builtin_expect(pusher && wave < 5, 1)) {
        const unsigned st = (unsigned)wave;
        const LinkArgs A = P.link[link];
        const bool intra = link < 3u, isw = !intra && st >= 3u;
        const unsigned lu = intra ? st : 5u + st;
        const unsigned pw = 8u * (st - 3u) + p;
        const rsrc_t ssrd = make_rsrc(isw ? A.wq : A.x, isw ? (unsigned)A.wq_bytes : SHARD_BYTES);
        const rsrc_t dsrd = make_rsrc(A.dst, ST_BYTES);
        const unsigned svoff = isw ? (4u * pw + (lane >> 4)) * 1024u + ((((unsigned)lane & 15u) ^ (((unsigned)lane >> 4) << 2) ^ (pw & 3u)) << 4)
                                   : (lu * 64u + 8u * p + (lane >> 3)) * 8192u + ((((unsigned)lane & 7u) ^ ((unsigned)lane >> 4) ^ ((p & 1u) << 2)) << 4);
        const unsigned dvoff = p * ST_PIECE + st * 1024u + lane * 16u;
        const unsigned sstr = isw ? 65536u : 128u;
        u32x4 v0 = bld16(svoff, ssrd, 0), v1 = bld16(svoff, ssrd, sstr);
        constexpr int TAU8 = PUSH_TAU_NS * 8 / 10;
        int sched = PUSH_T0 * 8 + (int)((((p & 3u) << 1) | (p >> 2)) * 5u + st) * TAU8 / 40, floor_t = 0, off = 0, ti0 = 0, ti1 = 0;
        auto clock8 = [&]() { return (int)((unsigned)(now() - ttop) << 3); };
        auto pace = [&](int& ti, bool acked) {
            int tn = clock8();
            if (acked) {
                const int lat = tn - ti;
                if (lat > LAT_HI * 8) { if (off < OFF_CAP * 8) off += OFF_INC * 8; } else if (off >= OFF_DEC * 8) off -= OFF_DEC * 8;
            }
            const int due = sched + off > floor_t ? sched + off : floor_t;
            while (tn < due) { __builtin_amdgcn_s_sleep(1); tn = clock8(); }
            floor_t = tn + PUSH_GAP * 8;
            sched += TAU8;
            ti = tn;
        };
        asm volatile("" : "+v"(v0));
        pace(ti0, false);
        bst16_sys(v0, dvoff, dsrd, 0); v0 = bld16(svoff, ssrd, 2u * sstr);
        pace(ti1, false);
        bst16_sys(v1, dvoff, dsrd, ST_STEP); v1 = bld16(svoff, ssrd, 3u * sstr);
#pragma unroll 1
        for (unsigned i = 2; i < (unsigned)NSTEP; i += 2) {
            WAIT_VM(2);
            pace(ti0, true);
            bst16_sys(v0, dvoff, dsrd, i * ST_STEP);
            v0 = bld16(svoff, ssrd, (i + 2u < (unsigned)NSTEP ? i + 2u : i) * sstr);
            WAIT_VM(2);
            pace(ti1, true);
            bst16_sys(v1, dvoff, dsrd, (i + 1u) * ST_STEP);
            v1 = bld16(svoff, ssrd, (i + 3u < (unsigned)NSTEP ? i + 3u : i) * sstr);
            asm volatile("" ::: "memory");
            __builtin_amdgcn_s_barrier();
        }
        WAIT_VM(0);
        __builtin_amdgcn_s_barrier();
        asm volatile("" :: "v"(v0), "v"(v1));
        return;
    }
    const unsigned rank = (unsigned)P.rank & 7u;
    const unsigned grp = rank >> 2, me = rank & 3u, og = (grp ^ 1u) * 4u;
    char* const myws = P.ws[rank];

    volatile LAS unsigned* const ctl = (volatile LAS unsigned*)(L + L_CTRL);
    auto unit_src = [&](unsigned u, unsigned& a, unsigned& lu) {
        if (u < 20u) { const unsigned i = u / 5u; a = grp * 4u + i; lu = u - 5u * i; }
        else { const unsigned i = (u - 20u) / 3u; a = og + i; lu = 5u + (u - 20u) - 3u * i; }
        a &= 7u;
    };
    auto min4 = [](unsigned v, int l) {
        const unsigned p0 = __builtin_amdgcn_readlane(v, l), p1 = __builtin_amdgcn_readlane(v, l + 1),
                       p2 = __builtin_amdgcn_readlane(v, l + 2), p3 = __builtin_amdgcn_readlane(v, l + 3);
        const unsigned m0 = p0 < p1 ? p0 : p1, m1 = p2 < p3 ? p2 : p3;
        return m0 < m1 ? m0 : m1;
    };
    if (pusher) {
        if (wave > 5) return;
        const unsigned dr = link_rank(rank, link);
        char* const flag = P.ws[dr] + OFF_FLAG + (rank * 8u + p) * FSTR;
        unsigned tk = 0;
        if (lane == 0) tk = draw_ticket(myws, b);
        const unsigned call = rfl(tk) + 1u, base = call * 64u;
        const char* fillp = lane == 0 ? myws + OFF_FILL + dr * 128u : myws + OFF_TMO;
#pragma unroll 1
        for (int done = 2; done <= NSTEP; done += 2) {
            __builtin_amdgcn_s_barrier();
            if (done == NSTEP - NSPEC - 4) {
                const unsigned long long t0 = now();
                for (;;) {
                    const unsigned v = ld_sys(fillp);
                    if (__builtin_amdgcn_readlane(v, 0) >= call) break;
                    if (__builtin_amdgcn_readlane(v, 1) == call || now() - t0 > DEADLINE) { if (lane == 0) st_agent(myws + OFF_TMO, call); break; }
                }
            }
            const bool pub = done <= NSTEP - NSPEC ? (done % 4 == 0 || done > NSTEP - NSPEC - 12) : done == NSTEP;
            if (lane == 0 && pub) st_sys(flag, base + (unsigned)done);
        }
        return;
    }

    if (b < N_PROD) {
        const unsigned src = (rank + 1u + p) & 7u;
        if (wave >= 8 || src == rank) return;
        unsigned tk = 0;
        if (tid == 0) tk = draw_ticket(myws, b);
        const rsrc_t dsrd = make_rsrc(myws + OFF_ST + src * ST_BYTES + (NSTEP - NSPEC) * ST_STEP, NSPEC * ST_STEP);
        const u32x4 ones = {~0u, ~0u, ~0u, ~0u};
#pragma unroll 1
        for (unsigned i = 0; i < NSPEC * 5u; ++i) {
            bst16_sys(ones, tid * 16u, dsrd, i * 8192u);
            WAIT_VM(8);
        }
        asm volatile("s_waitcnt vmcnt(0)" : "+v"(tk) :: "memory");
        __builtin_amdgcn_s_barrier();
        if (tid == 0) st_sys(P.ws[src] + OFF_FILL + rank * 128u, tk + 1u);
        return;
    }

    const unsigned gi = (unsigned)b - N_PROD;
    const unsigned t = ((gi & 7u) >> 1) * 4u + ((gi >> 3) & 3u), c = (gi & 1u) * 2u + ((gi >> 5) & 1u) + (gi >> 6) * 4u;
    unsigned a0, a1, lu0, lu1;
    unit_src(2u * t, a0, lu0); unit_src(2u * t + 1u, a1, lu1);
    const bool own0 = a0 == rank, own1 = a1 == rank;
    if (tid < 32) ctl[tid] = ((tid == 8 && own0) || (tid == 9 && own1) || (tid == 10 && c < 4u)) ? (unsigned)NSTEP : 0u;
    BARRIER();

    auto hrow = [&](unsigned u) {
        const unsigned i5 = u / 5u, i3 = (u - 20u) / 3u;
        return u < 20u ? (og + i5) * 512u + (u - 5u * i5) * 64u : (grp * 4u + i3) * 512u + (5u + u - 20u - 3u * i3) * 64u;
    };
    const unsigned hr0 = hrow(2u * t), hr1 = hrow(2u * t + 1u);
    f32x16 acc[2][2];
    if (wave < 4) {
        const unsigned wm = (unsigned)wave >> 1, wn = (unsigned)wave & 1u;
        const unsigned h = (unsigned)lane >> 5;
        unsigned xa[4], wa[2][2];
#pragma unroll
        for (int ks = 0; ks < 4; ++ks) xa[ks] = (wm * 64u + (lane & 31u)) * 128u + (((2u * ks + h) ^ (((unsigned)lane >> 1) & 7u)) << 4);
        {
            const unsigned blk = ((unsigned)lane >> 4) & 1u, q = ((unsigned)lane & 15u) >> 2, pp = (unsigned)lane & 3u;
#pragma unroll
            for (int nt = 0; nt < 2; ++nt)
#pragma unroll
                for (int tt = 0; tt < 2; ++tt) {
                    const unsigned row = 8u * h + 4u * tt + q, ch = 4u * (2u * wn + nt) + 2u * blk + (pp >> 1);
                    wa[nt][tt] = 16384u + 256u * row + ((ch ^ (((row & 3u) << 2) | ((row >> 2) & 3u))) << 4) + 8u * (pp & 1u);
                }
        }
#pragma unroll
        for (int i = 0; i < 2; ++i)
#pragma unroll
            for (int jj = 0; jj < 2; ++jj)
#pragma unroll
                for (int e = 0; e < 16; ++e) acc[i][jj][e] = 0.f;
        char* lanews = P.ws[lane & 7];
        asm volatile("" : "+v"(lanews));
        unsigned have = 0;
        s16x8 xf[4][2], wf[4][2];
        auto rd = [&](LAS char* S, int ks) {
#pragma unroll
            for (int nt = 0; nt < 2; ++nt) {
                const s16x4 lo = __builtin_amdgcn_ds_read_tr16_b64_v4i16((LAS s16x4*)(S + wa[nt][0] + ks * 4096));
                const s16x4 hi = __builtin_amdgcn_ds_read_tr16_b64_v4i16((LAS s16x4*)(S + wa[nt][1] + ks * 4096));
                wf[ks][nt] = __builtin_shufflevector(lo, hi, 0, 1, 2, 3, 4, 5, 6, 7);
            }
#pragma unroll
            for (int mt = 0; mt < 2; ++mt) xf[ks][mt] = *(LAS s16x8*)(S + xa[ks] + mt * 4096);
            __builtin_amdgcn_sched_barrier(0);
        };
        auto mm = [&](int ks) {
#pragma unroll
            for (int nt = 0; nt < 2; ++nt)
#pragma unroll
                for (int mt = 0; mt < 2; ++mt)
                    acc[nt][mt] = __builtin_amdgcn_mfma_f32_32x32x16_bf16((bf16x8)wf[ks][nt], (bf16x8)xf[ks][mt], acc[nt][mt], 0, 0, 0);
            __builtin_amdgcn_sched_barrier(0);
        };
        __builtin_amdgcn_s_waitcnt(0xC07F);
#pragma unroll 1
        for (int k4 = 0; k4 < NSTEP; k4 += NSLOT) {
#pragma unroll
            for (int sl = 0; sl < (int)NSLOT; ++sl) {
                const unsigned k = k4 + sl;
                LAS char* const S = L + sl * SLOT;
                while (have <= k) {
                    const u32x4 f = *(volatile LAS u32x4*)(L + L_FULL);
                    const unsigned fa = wm ? f.y : f.x, fb = f.z < f.w ? f.z : f.w;
                    have = rfl(fa < fb ? fa : fb);
                    if (have <= k) __builtin_amdgcn_s_sleep(1);
                }
                asm volatile("" ::: "memory");
                rd(S, 0); rd(S, 1); mm(0); rd(S, 2); mm(1); rd(S, 3); mm(2);
                __builtin_amdgcn_s_waitcnt(0xC07F);
                asm volatile("" ::: "memory");
                ctl[4 + wave] = k + 1;
                __builtin_amdgcn_sched_barrier(0);
                mm(3);
            }
        }
        float m = 0.f;
#pragma unroll
        for (int i = 0; i < 2; ++i)
#pragma unroll
            for (int jj = 0; jj < 2; ++jj)
#pragma unroll
                for (int e = 0; e < 16; ++e) m = fmaxf(m, fabsf(acc[i][jj][e]));
        const unsigned mu = wave_max_u32(__float_as_uint(m));
        unsigned arr = 0;
        if (lane == 0) {
            __hip_atomic_fetch_max((LAS unsigned*)(L + L_WMAX), mu, __ATOMIC_RELAXED, __HIP_MEMORY_SCOPE_WORKGROUP);
            arr = __hip_atomic_fetch_add((LAS unsigned*)(L + L_WMAX + 4), 1u, __ATOMIC_RELAXED, __HIP_MEMORY_SCOPE_WORKGROUP);
        }
        if (rfl(arr) == 3u) {
            const unsigned mm = ctl[16];
            unsigned cl;
            while ((cl = rfl(ctl[13])) == 0u) { }
            if (lane < NR)
                __hip_atomic_store((gu64*)(lanews + OFF_AMAX + (cl & 1u) * AMAX_BYTES + (rank * 128u + gi) * GSTR), ((unsigned long long)cl << 32) | mm,
                                   __ATOMIC_RELAXED, __HIP_MEMORY_SCOPE_SYSTEM);
        }
    } else if (wave < 8) {
        const unsigned li = (unsigned)wave - 4u, hf = li & 1u;
        const bool isx = li < 2u;
        const bool own = isx ? (hf ? own1 : own0) : (c < 4u);
        const unsigned luh = hf ? lu1 : lu0;
        const unsigned src = isx ? (hf ? a1 : a0) : og + (c & 3u), stn = isx ? (luh < 5u ? luh : luh - 5u) : 3u + hf;
        const char* base; unsigned bytes;
        if (!own) { base = myws + OFF_ST + src * ST_BYTES + stn * 1024u; bytes = ST_BYTES - stn * 1024u; }
        else if (isx) { base = P.x + luh * UNIT_BYTES; bytes = UNIT_BYTES; }
        else { base = P.w + c * 256u + hf * 32768u; bytes = SHARD_BYTES - c * 256u - hf * 32768u; }
        const u32x4 srd = make_srd(base, bytes);
        const unsigned lin = lane * 16u;
        const unsigned swx = (lane >> 3) * 8192u + ((((unsigned)lane & 7u) ^ ((unsigned)lane >> 4)) << 4);
        const unsigned sww = (lane >> 4) * 1024u + ((((unsigned)lane & 15u) ^ (((unsigned)lane >> 4) << 2)) << 4);
        unsigned vo[4];
#pragma unroll
        for (int i = 0; i < 4; ++i) vo[i] = isx ? (swx ^ ((i & 1) << 6)) : (sww ^ (i << 4));
        const unsigned pstr = isx ? 65536u : 4096u, sstr = isx ? 128u : 65536u;
        const unsigned avw = isx ? 8u + hf : 10u;
        unsigned avail = 0, freed = 0, pub = 0;
        const unsigned nnorm = own ? (unsigned)NSTEP : (unsigned)(NSTEP - NSPEC);
        auto look_free = [&]() {
            const u32x4 f = *(volatile LAS u32x4*)(L + L_FREE);
            const unsigned m0 = f.x < f.y ? f.x : f.y, m1 = f.z < f.w ? f.z : f.w;
            freed = rfl(isx ? (hf ? m1 : m0) : (m0 < m1 ? m0 : m1));
        };
        asm volatile("s_nop 4" ::: "memory");
#pragma unroll 1
        for (unsigned j = 0; j < nnorm; ++j) {
            if (avail <= j) {
                avail = rfl(ctl[avw]);
                if (avail <= j) {
                    WAIT_VM(0);
                    if (pub < j) { pub = j; ctl[li] = pub; }
                    do { __builtin_amdgcn_s_sleep(1); avail = rfl(ctl[avw]); } while (avail <= j);
                }
            }
            while (freed + NSLOT <= j) { look_free(); if (freed + NSLOT <= j) __builtin_amdgcn_s_sleep(1); }
            const unsigned dst = (unsigned)(size_t)(L + (j & (NSLOT - 1)) * SLOT) + li * 8192u;
            if (own) {
#pragma unroll
                for (int p = 0; p < 8; ++p) dma16(dst + p * 1024u, vo[p & 3], srd, j * sstr + p * pstr);
            } else {
#pragma unroll
                for (int p = 0; p < 8; ++p) dma16_sys(dst + p * 1024u, lin, srd, j * ST_STEP + p * ST_PIECE);
            }
            WAIT_VM(16);
            if (j >= 2 && pub < j - 1) { pub = j - 1; ctl[li] = pub; }
        }
        WAIT_VM(8); ctl[li] = nnorm - 1u;
        auto landed = [&](unsigned j) {
            const LAS u32x4* q = (const LAS u32x4*)(L + (j & (NSLOT - 1)) * SLOT + li * 8192u + lin);
            unsigned m = 0;
#pragma unroll
            for (int p = 0; p < 8; ++p) { const u32x4 v = q[p * 64]; m = max(max(m, v.x), max(max(v.y, v.z), v.w)); }
            return !__any(m == ~0u);
        };
        pub = nnorm - 1u;
#pragma unroll 1
        for (unsigned j = nnorm; j < (unsigned)NSTEP; j = pub) {
            look_free();
            if (freed + NSLOT <= j) {
                WAIT_VM(0);
                if (pub < j) { pub = j; ctl[li] = pub; }
                do { __builtin_amdgcn_s_sleep(1); look_free(); } while (freed + NSLOT <= j);
            }
            avail = rfl(ctl[avw]);
            const bool two = j + 1u < (unsigned)NSTEP && freed + NSLOT > j + 1u;
            const unsigned d0 = (unsigned)(size_t)(L + (j & (NSLOT - 1)) * SLOT) + li * 8192u;
            const unsigned d1 = (unsigned)(size_t)(L + ((j + 1u) & (NSLOT - 1)) * SLOT) + li * 8192u;
#pragma unroll
            for (int p = 0; p < 8; ++p) dma16_sys(d0 + p * 1024u, lin, srd, j * ST_STEP + p * ST_PIECE);
            if (two) {
#pragma unroll
                for (int p = 0; p < 8; ++p) dma16_sys(d1 + p * 1024u, lin, srd, (j + 1u) * ST_STEP + p * ST_PIECE);
                WAIT_VM(16);
            } else WAIT_VM(8);
            if (pub < j) { pub = j; ctl[li] = pub; }
            if (two) WAIT_VM(8); else WAIT_VM(0);
            if (avail > j || landed(j)) {
                pub = j + 1u; ctl[li] = pub;
                WAIT_VM(0);
                if (two && (avail > j + 1u || landed(j + 1u))) { pub = j + 2u; ctl[li] = pub; }
            }
            WAIT_VM(0);
        }
        WAIT_VM(0); ctl[li] = NSTEP;
    } else {
        unsigned tk = 0;
        if (lane == 0) tk = draw_ticket(myws, b);
        const unsigned call = rfl(tk) + 1u;
        WAIT_VM(0);
        if (lane == 0) ctl[12] = call;
        const unsigned fsrc = lane < 8 ? a0 : (lane < 16 ? a1 : og + (c & 3u));
        const char* fp = lane < 24 ? myws + OFF_FLAG + (fsrc * 8u + (lane & 7)) * FSTR : myws + OFF_TMO;
        const rsrc_t msrd = make_rsrc(P.out[rank], OUT_BYTES);
        const u32x4 mark = {0x80808080u, 0x80808080u, 0x80808080u, 0x80808080u};
        unsigned nmark = c < 4u ? 0u : 16u;
        auto put_mark = [&]() {
            bst16_sys(mark, ((unsigned)lane >> 3) * 512u + ((unsigned)lane & 7u) * 16u, msrd, (((nmark & 8u) ? hr1 : hr0) + (nmark & 7u) * 8u) * 512u + c * 128u);
            ++nmark;
        };
        const unsigned long long t0 = now();
        for (;;) {
            if (nmark < 16u && (unsigned)(now() - t0) > MARK_T0 + (nmark * 16u + (gi & 15u)) * MARK_DELTA) put_mark();
            if (nmark == 16u) { WAIT_VM(0); if (lane == 0) ctl[13] = call; nmark = 17u; }
            const unsigned v = ld_sys(fp);
            const int d = (int)(v - call * 64u);
            const unsigned prog = d < 0 ? 0u : (d > NSTEP ? (unsigned)NSTEP : (unsigned)d);
            auto min8 = [&](int l) { const unsigned m0 = min4(prog, l), m1 = min4(prog, l + 4); return m0 < m1 ? m0 : m1; };
            unsigned av0 = own0 ? (unsigned)NSTEP : min8(0), av1 = own1 ? (unsigned)NSTEP : min8(8), avb = c < 4u ? (unsigned)NSTEP : min8(16);
            const bool dead = __builtin_amdgcn_readlane(v, 24) == call || now() - t0 > DEADLINE;
            if (dead) { if (lane == 0) { st_agent(myws + OFF_TMO, call); ctl[11] = 1u; } av0 = av1 = avb = NSTEP; }
            if (lane == 0) { ctl[8] = av0; ctl[9] = av1; ctl[10] = avb; }
            if (av0 == (unsigned)NSTEP && av1 == (unsigned)NSTEP && avb == (unsigned)NSTEP) break;
            const bool lazy = min(min(av0, av1), avb) >= (unsigned)(NSTEP - NSPEC) && nmark > 16u;
            bool done = false;
            for (int i = lazy ? 40 : 1; i > 0 && !done; --i) {
                const u32x4 fl = *(volatile LAS u32x4*)(L + L_FULL);
                done = rfl(fl.x + fl.y + fl.z + fl.w) == 4u * NSTEP;
                __builtin_amdgcn_s_sleep(1);
            }
            if (done) break;
        }
        while (nmark < 16u) {
            if ((unsigned)(now() - t0) > MARK_T0 + (nmark * 16u + (gi & 15u)) * MARK_DELTA || rfl(ctl[4]) > 48u) put_mark(); else __builtin_amdgcn_s_sleep(8);
        }
        if (nmark == 16u) { WAIT_VM(0); if (lane == 0) ctl[13] = call; }
    }

    if (wave >= 8) return;
    if (wave < 4) {
        for (;;) {
            const u32x4 f = *(volatile LAS u32x4*)(L + L_FREE);
            const unsigned m0 = f.x < f.y ? f.x : f.y, m1 = f.z < f.w ? f.z : f.w;
            if (rfl(m0 < m1 ? m0 : m1) >= (unsigned)NSTEP - 2u) break;
        }
        asm volatile("" ::: "memory");
        const unsigned wm = (unsigned)wave >> 1, wn = (unsigned)wave & 1u, h = (unsigned)lane >> 5;
#pragma unroll
        for (int nt = 0; nt < 2; ++nt)
#pragma unroll
            for (int mt = 0; mt < 2; ++mt)
#pragma unroll
                for (int g = 0; g < 4; ++g) {
                    const unsigned m = wm * 64u + mt * 32u + (lane & 31u), ch = 16u * wn + 8u * nt + 2u * g + h;
                    f32x4 q; q.x = acc[nt][mt][4 * g]; q.y = acc[nt][mt][4 * g + 1]; q.z = acc[nt][mt][4 * g + 2]; q.w = acc[nt][mt][4 * g + 3];
                    *(LAS f32x4*)(L + m * 512u + ((ch ^ (m & 7u)) << 4)) = q;
                }
    }
    const unsigned orank = (c < 4u ? rank : og + (c & 3u)) & 7u;
    const unsigned ocol = (c < 4u ? c : me) * 128u;
    char* optr = P.out[orank];
    asm volatile("" : "+s"(optr));
    unsigned call = 0, mx = 0;
    const char* gp = myws + OFF_AMAX + ((unsigned)tid & 255u) * GSTR;
    const unsigned own_g = rank * 128u + gi;
    const unsigned skip = (own_g & 255u) == ((unsigned)tid & 255u) ? own_g >> 8 : 4u;
    unsigned long long tg0 = 0;
    bool ok = false;
    auto sweep = [&]() {
        const unsigned tm = ld_agent(myws + OFF_TMO);
        if (!ok) {
            unsigned long long g[4];
#pragma unroll
            for (int j = 0; j < 4; ++j) g[j] = __hip_atomic_load((gu64*)(gp + j * 256 * GSTR), __ATOMIC_RELAXED, __HIP_MEMORY_SCOPE_SYSTEM);
            ok = true; mx = 0;
#pragma unroll
            for (int j = 0; j < 4; ++j) {
                const bool mine = skip == (unsigned)j;
                ok = ok && (mine || (unsigned)(g[j] >> 32) >= call);
                mx = max(mx, mine ? 0u : (unsigned)g[j]);
            }
        }
        if (rfl(tm) == call || now() - tg0 > DEADLINE) { if (lane == 0) { st_agent(myws + OFF_TMO, call); ctl[11] = 1u; } return true; }
        return (bool)__all(ok);
    };
    if (wave >= 4) {
        while ((call = rfl(ctl[12])) == 0u) { }
        gp += (call & 1u) * AMAX_BYTES;
    }
    BARRIER();
    f32x4 y[2][4];
    const unsigned r0 = (unsigned)tid >> 3, q8 = (unsigned)tid & 7u;
#pragma unroll
    for (int hf = 0; hf < 2; ++hf)
#pragma unroll
        for (int e = 0; e < 4; ++e) y[hf][e] = *(LAS f32x4*)(L + (r0 + 64u * hf) * 512u + (((4u * q8 + e) ^ (r0 & 7u)) << 4));
    if (wave >= 4) {
        tg0 = now();
        while (!sweep()) { }
        mx = wave_max_u32(mx);
        if (lane == 0) ctl[20 + wave] = mx;
    }
    BARRIER();
    const unsigned long long tq = now();
    const u32x4 gmx = *(volatile LAS u32x4*)(L + L_GMAX);
    const unsigned ga = max(max(gmx.x, gmx.y), max(gmx.z, gmx.w)), gb = ctl[16];
    const bool aborted = ctl[11] != 0u;
    const float amax = __uint_as_float(ga > gb ? ga : gb);
    const float inv = amax > 0.f ? 127.0f / amax : 0.f;
    const rsrc_t osrd = make_rsrc(optr, OUT_BYTES);
    const unsigned ovoff = r0 * 512u + q8 * 16u;
    constexpr float MAGIC = 12582912.0f;
#pragma unroll
    for (int hf = 0; hf < 2; ++hf) {
        u32x4 o;
#pragma unroll
        for (int e = 0; e < 4; ++e) {
            unsigned z[4];
#pragma unroll
            for (int i = 0; i < 4; ++i)
                z[i] = __float_as_uint(__builtin_amdgcn_fmed3f(__builtin_fmaf(y[hf][e][i], inv, MAGIC), MAGIC - 127.0f, MAGIC + 127.0f));
            const unsigned lo = __builtin_amdgcn_perm(z[1], z[0], 0x0c0c0400u), hi = __builtin_amdgcn_perm(z[3], z[2], 0x04000c0cu);
            o[e] = aborted ? 0x80808080u : (lo | hi);
        }
        const unsigned row = hf ? a1 * 512u + lu1 * 64u : a0 * 512u + lu0 * 64u;
        if (c < 4u) __builtin_amdgcn_raw_buffer_store_b128(o, osrd, ovoff, row * 512u + ocol, 16);
        else {
            const unsigned due = ((unsigned)(hf * 8 + wave) * 16u + t) * OUT_DELTA_NS / 160u;
            asm volatile("" : "+v"(o));
            while ((unsigned)(now() - tq) < due) __builtin_amdgcn_s_sleep(1);
            bst16_sys(o, ovoff, osrd, row * 512u + ocol);
        }
    }
    if (c < 4u) {
        call = rfl(ctl[12]);
        const unsigned due = ((unsigned)(8 + wave) * 16u + t) * OUT_DELTA_NS / 160u + OUT_CHECK_DELAY;
        while ((unsigned)(now() - tq) < due) __builtin_amdgcn_s_sleep(2);
        bool in = false;
        for (;;) {
            const unsigned tm = ld_agent(myws + OFF_TMO);
            if (!in) {
                const u32x4 h0 = __builtin_amdgcn_raw_buffer_load_b128(osrd, ovoff, hr0 * 512u + ocol, 17);
                const u32x4 h1 = __builtin_amdgcn_raw_buffer_load_b128(osrd, ovoff, hr1 * 512u + ocol, 17);
                unsigned bad = 0;
#pragma unroll
                for (int e = 0; e < 4; ++e) {
                    const unsigned x0 = h0[e] ^ 0x80808080u, x1 = h1[e] ^ 0x80808080u;
                    bad |= ((x0 - 0x01010101u) & ~x0) | ((x1 - 0x01010101u) & ~x1);
                }
                in = (bad & 0x80808080u) == 0u;
            }
            if (__all(in)) break;
            if (rfl(tm) == call || now() - tq > DEADLINE) { if (lane == 0) st_agent(myws + OFF_TMO, call); break; }
            asm volatile("" ::: "memory");
        }
    }
}

extern "C" void kernel_launch(int rank, int nranks, ncclComm_t comm,
                              hipStream_t stream,
                              void* const* d_in, const int* in_sizes, int n_in,
                              void* d_out, int out_size,
                              const int* in_dtypes, int out_dtype,
                              void* const* d_out_all,
                              void* const* d_ws_all, size_t ws_bytes) {
    (void)comm; (void)in_dtypes; (void)out_dtype; (void)d_out;
    if (nranks != NR || (unsigned)rank >= (unsigned)NR || n_in < 2 || ws_bytes < WS_NEEDED ||
        in_sizes[0] != 512 * 4096 || in_sizes[1] != 4096 * 512 || out_size != (int)OUT_BYTES) {
        fprintf(stderr, "kernel_launch: unexpected problem shape\n");
        return;
    }
    int dev = 0, cus = 0;
    HIP_CHECK(hipGetDevice(&dev));
    HIP_CHECK(hipDeviceGetAttribute(&cus, hipDeviceAttributeMultiprocessorCount, dev));
    if (cus < GRID) { fprintf(stderr, "kernel_launch: needs %d CUs\n", GRID); return; }
    Params P;
    P.x = (const char*)d_in[0];
    P.w = (const char*)d_in[1];
    for (int p = 0; p < NR; ++p) { P.ws[p] = (char*)d_ws_all[p]; P.out[p] = (char*)d_out_all[p]; }
    for (unsigned l = 0; l < (unsigned)NR; ++l) {
        const unsigned dr = link_rank((unsigned)rank, l < 7u ? l : 0u);
        P.link[l].x = P.x;
        P.link[l].wq = P.w + (dr & 3u) * 256u;
        P.link[l].wq_bytes = SHARD_BYTES - (dr & 3u) * 256u;
        P.link[l].dst = P.ws[dr] + OFF_ST + (unsigned)rank * ST_BYTES;
    }
    P.rank = rank;
    hipLaunchKernelGGL(ag_gemm_q8, dim3(GRID), dim3(NTHREADS), 0, stream, P);
    HIP_CHECK(hipGetLastError());
}
```

```cpp
#include <hip/hip_runtime.h>
#include <rccl/rccl.h>
#include <cstdio>
#include <cstdint>

#define HIP_CHECK(expr) do { hipError_t _e = (expr); if (_e != hipSuccess) { \
    fprintf(stderr, "HIP error %s at %s:%d\n", hipGetErrorString(_e), __FILE__, __LINE__); \
    return; } } while (0)

typedef __attribute__((ext_vector_type(4))) unsigned u32x4;
typedef __attribute__((ext_vector_type(4))) float f32x4;
typedef __attribute__((ext_vector_type(16))) float f32x16;
typedef __attribute__((ext_vector_type(4))) short s16x4;
typedef __attribute__((ext_vector_type(8))) short s16x8;
typedef __attribute__((ext_vector_type(8))) __bf16 bf16x8;
#define LAS __attribute__((address_space(3)))
#define GAS __attribute__((address_space(1)))
typedef GAS unsigned gu32;
typedef GAS unsigned long long gu64;

constexpr int NR = 8;
constexpr int NSTEP = 64;
constexpr int N_PROD = 64;
constexpr int N_GEMM = 136;
constexpr int GRID = N_PROD + N_GEMM;
constexpr int NTHREADS = 576;
constexpr int PUSH_T0 = 30, PUSH_TAU_NS = 688;
constexpr int PUSH_GAP = 64;
constexpr int LAT_HI = 85, OFF_INC = 2, OFF_DEC = 1, OFF_CAP = 80;
constexpr unsigned OUT_DELTA_NS = 262;
constexpr unsigned OUT_CHECK_DELAY = 60;
constexpr unsigned MARK_T0 = 300, MARK_DELTA = 8;
constexpr unsigned FSTR = 128, GSTR = 64;
constexpr int NSPEC = 24;
constexpr unsigned long long DEADLINE = 100000000ull;

constexpr unsigned OFF_CTR = 0;
constexpr unsigned OFF_TMO = 0x8000;
constexpr unsigned OFF_FILL = 0x9000;
constexpr unsigned OFF_FLAG = 0xE000;
constexpr unsigned OFF_AMAX = 0x10000;
constexpr unsigned AMAX_BYTES = 1280 * GSTR;
constexpr unsigned UNIT_BYTES = 64u * 8192u;
constexpr unsigned OFF_ST = 1u << 20;
constexpr unsigned ST_PIECE = 5u * 1024u, ST_STEP = 8u * ST_PIECE, ST_BYTES = 64u * ST_STEP;
constexpr unsigned WS_NEEDED = OFF_ST + 8u * ST_BYTES;
constexpr unsigned SHARD_BYTES = 4u << 20;
constexpr unsigned OUT_BYTES = 4096u * 512u;
static_assert(NSPEC % 2 == 0 && NSPEC <= NSTEP - 6 && GRID * 128 <= (int)OFF_TMO && OFF_AMAX + 2 * AMAX_BYTES <= OFF_ST, "workspace map");

constexpr unsigned SLOT = 32768, NSLOT = 4;
constexpr unsigned L_CTRL = SLOT * NSLOT;
constexpr unsigned L_FULL = L_CTRL, L_FREE = L_CTRL + 16, L_WMAX = L_CTRL + 64, L_GMAX = L_CTRL + 96;
constexpr unsigned LDS_BYTES = L_CTRL + 128;

struct alignas(32) LinkArgs {
    const char* x;
    const char* wq;
    char* dst;
    unsigned long long wq_bytes;
};
struct Params {
    LinkArgs link[NR];
    const char* x;
    const char* w;
    char* ws[NR];
    char* out[NR];
    int rank;
};

__host__ __device__ __forceinline__ unsigned link_rank(unsigned rank, unsigned l) {
    return l < 3u ? (rank & 4u) | ((rank + 1u + l) & 3u) : ((rank & 4u) ^ 4u) | ((rank + l - 3u) & 3u);
}
__device__ __forceinline__ unsigned rfl(unsigned v) { return __builtin_amdgcn_readfirstlane(v); }
__device__ __forceinline__ u32x4 make_srd(const void* p, unsigned bytes) {
    const unsigned long long a = (unsigned long long)p;
    u32x4 r;
    r.x = rfl((unsigned)a); r.y = rfl((unsigned)(a >> 32) & 0xffffu); r.z = rfl(bytes); r.w = 0x00020000u;
    return r;
}
__device__ __forceinline__ void dma16(unsigned lds, unsigned voff, u32x4 srd, unsigned soff) {
    asm volatile("s_mov_b32 m0, %0\n\ts_nop 0\n\tbuffer_load_dwordx4 %1, %2, %3 offen lds"
                 :: "s"(lds), "v"(voff), "s"(srd), "s"(soff) : "memory");
}
__device__ __forceinline__ void dma16_sys(unsigned lds, unsigned voff, u32x4 srd, unsigned soff) {
    asm volatile("s_mov_b32 m0, %0\n\ts_nop 0\n\tbuffer_load_dwordx4 %1, %2, %3 offen sc0 sc1 lds"
                 :: "s"(lds), "v"(voff), "s"(srd), "s"(soff) : "memory");
}
typedef __amdgpu_buffer_rsrc_t rsrc_t;
__device__ __forceinline__ rsrc_t make_rsrc(const void* p, unsigned bytes) {
    const unsigned long long a = (unsigned long long)p;
    const unsigned long long u = ((unsigned long long)rfl((unsigned)(a >> 32)) << 32) | rfl((unsigned)a);
    return __builtin_amdgcn_make_buffer_rsrc((void*)u, 0, rfl(bytes), 0x00020000);
}
__device__ __forceinline__ u32x4 bld16(unsigned voff, rsrc_t r, unsigned soff) { return __builtin_amdgcn_raw_buffer_load_b128(r, voff, soff, 0); }
__device__ __forceinline__ void bst16_sys(u32x4 v, unsigned voff, rsrc_t r, unsigned soff) {
    __builtin_amdgcn_raw_buffer_store_b128(v, r, voff, soff, 17);
}
#define WAIT_VM(n) asm volatile("s_waitcnt vmcnt(" #n ")" ::: "memory")
#define BARRIER() do { asm volatile("s_waitcnt lgkmcnt(0)" ::: "memory"); __builtin_amdgcn_s_barrier(); asm volatile("" ::: "memory"); } while (0)
__device__ __forceinline__ unsigned long long now() { return __builtin_amdgcn_s_memrealtime(); }
__device__ __forceinline__ unsigned ld_sys(const void* p) { return __hip_atomic_load((gu32*)p, __ATOMIC_RELAXED, __HIP_MEMORY_SCOPE_SYSTEM); }
__device__ __forceinline__ void st_sys(void* p, unsigned v) { __hip_atomic_store((gu32*)p, v, __ATOMIC_RELAXED, __HIP_MEMORY_SCOPE_SYSTEM); }
__device__ __forceinline__ unsigned ld_agent(const void* p) { return __hip_atomic_load((gu32*)p, __ATOMIC_RELAXED, __HIP_MEMORY_SCOPE_AGENT); }
__device__ __forceinline__ void st_agent(void* p, unsigned v) { __hip_atomic_store((gu32*)p, v, __ATOMIC_RELAXED, __HIP_MEMORY_SCOPE_AGENT); }
__device__ __forceinline__ unsigned draw_ticket(char* ws, int b) {
    unsigned z = 0;
    asm volatile("" : "+v"(z));
    return __hip_atomic_fetch_add((gu32*)(ws + OFF_CTR + b * 128 + z), 1u, __ATOMIC_RELAXED, __HIP_MEMORY_SCOPE_AGENT);
}
__device__ __forceinline__ unsigned wave_max_u32(unsigned v) {
#define DPP_MAX(ctrl, rmask) do { const unsigned o_ = (unsigned)__builtin_amdgcn_update_dpp(0, (int)v, ctrl, rmask, 0xf, false); v = o_ > v ? o_ : v; } while (0)
    DPP_MAX(0x111, 0xf); DPP_MAX(0x112, 0xf); DPP_MAX(0x114, 0xf); DPP_MAX(0x118, 0xf);
    DPP_MAX(0x142, 0xa); DPP_MAX(0x143, 0xc);
#undef DPP_MAX
    return __builtin_amdgcn_readlane(v, 63);
}

__global__ void __launch_bounds__(NTHREADS, 3) ag_gemm_q8(Params P) {
    __shared__ __attribute__((aligned(16))) char smem[LDS_BYTES];
    LAS char* const L = (LAS char*)smem;
    const unsigned long long ttop = now();
    const int b = blockIdx.x;
    const int tid = threadIdx.x, lane = tid & 63;
    const int wave = (int)rfl(tid >> 6);
    const unsigned link = ((unsigned)b >> 4) + ((unsigned)b & 4u), p = (((unsigned)b >> 1) & 4u) | ((unsigned)b & 3u);
    const bool pusher = b < N_PROD && link < 7u;
    if (__builtin_expect(pusher && wave < 5, 1)) {
        const unsigned st = (unsigned)wave;
        const LinkArgs A = P.link[link];
        const bool intra = link < 3u, isw = !intra && st >= 3u;
        const unsigned lu = intra ? st : 5u + st;
        const unsigned pw = 8u * (st - 3u) + p;
        const rsrc_t ssrd = make_rsrc(isw ? A.wq : A.x, isw ? (unsigned)A.wq_bytes : SHARD_BYTES);
        const rsrc_t dsrd = make_rsrc(A.dst, ST_BYTES);
        const unsigned svoff = isw ? (4u * pw + (lane >> 4)) * 1024u + ((((unsigned)lane & 15u) ^ (((unsigned)lane >> 4) << 2) ^ (pw & 3u)) << 4)
                                   : (lu * 64u + 8u * p + (lane >> 3)) * 8192u + ((((unsigned)lane & 7u) ^ ((unsigned)lane >> 4) ^ ((p & 1u) << 2)) << 4);
        const unsigned dvoff = p * ST_PIECE + st * 1024u + lane * 16u;
        const unsigned sstr = isw ? 65536u : 128u;
        u32x4 v0 = bld16(svoff, ssrd, 0), v1 = bld16(svoff, ssrd, sstr);
        constexpr int TAU8 = PUSH_TAU_NS * 8 / 10;
        int sched = PUSH_T0 * 8 + (int)((((p & 3u) << 1) | (p >> 2)) * 5u + st) * TAU8 / 40, floor_t = 0, off = 0, ti0 = 0, ti1 = 0;
        auto clock8 = [&]() { return (int)((unsigned)(now() - ttop) << 3); };
        auto pace = [&](int& ti, bool acked) {
            int tn = clock8();
            if (acked) {
                const int lat = tn - ti;
                if (lat > LAT_HI * 8) { if (off < OFF_CAP * 8) off += OFF_INC * 8; } else if (off >= OFF_DEC * 8) off -= OFF_DEC * 8;
            }
            const int due = sched + off > floor_t ? sched + off : floor_t;
            while (tn < due) { __builtin_amdgcn_s_sleep(1); tn = clock8(); }
            floor_t = tn + PUSH_GAP * 8;
            sched += TAU8;
            ti = tn;
        };
        asm volatile("" : "+v"(v0));
        pace(ti0, false);
        bst16_sys(v0, dvoff, dsrd, 0); v0 = bld16(svoff, ssrd, 2u * sstr);
        pace(ti1, false);
        bst16_sys(v1, dvoff, dsrd, ST_STEP); v1 = bld16(svoff, ssrd, 3u * sstr);
#pragma unroll 1
        for (unsigned i = 2; i < (unsigned)NSTEP; i += 2) {
            WAIT_VM(2);
            pace(ti0, true);
            bst16_sys(v0, dvoff, dsrd, i * ST_STEP);
            v0 = bld16(svoff, ssrd, (i + 2u < (unsigned)NSTEP ? i + 2u : i) * sstr);
            WAIT_VM(2);
            pace(ti1, true);
            bst16_sys(v1, dvoff, dsrd, (i + 1u) * ST_STEP);
            v1 = bld16(svoff, ssrd, (i + 3u < (unsigned)NSTEP ? i + 3u : i) * sstr);
            asm volatile("" ::: "memory");
            __builtin_amdgcn_s_barrier();
        }
        WAIT_VM(0);
        __builtin_amdgcn_s_barrier();
        asm volatile("" :: "v"(v0), "v"(v1));
        return;
    }
    const unsigned rank = (unsigned)P.rank & 7u;
    const unsigned grp = rank >> 2, me = rank & 3u, og = (grp ^ 1u) * 4u;
    char* const myws = P.ws[rank];

    volatile LAS unsigned* const ctl = (volatile LAS unsigned*)(L + L_CTRL);
    auto unit_src = [&](unsigned u, unsigned& a, unsigned& lu) {
        if (u < 20u) { const unsigned i = u / 5u; a = grp * 4u + i; lu = u - 5u * i; }
        else if (u < 32u) { const unsigned i = (u - 20u) / 3u; a = og + i; lu = 5u + (u - 20u) - 3u * i; }
        else { a = rank; lu = u < 35u ? u - 27u : 0u; }
        a &= 7u;
    };
    auto min4 = [](unsigned v, int l) {
        const unsigned p0 = __builtin_amdgcn_readlane(v, l), p1 = __builtin_amdgcn_readlane(v, l + 1),
                       p2 = __builtin_amdgcn_readlane(v, l + 2), p3 = __builtin_amdgcn_readlane(v, l + 3);
        const unsigned m0 = p0 < p1 ? p0 : p1, m1 = p2 < p3 ? p2 : p3;
        return m0 < m1 ? m0 : m1;
    };
    if (pusher) {
        if (wave > 5) return;
        const unsigned dr = link_rank(rank, link);
        char* const flag = P.ws[dr] + OFF_FLAG + (rank * 8u + p) * FSTR;
        unsigned tk = 0;
        if (lane == 0) tk = draw_ticket(myws, b);
        const unsigned call = rfl(tk) + 1u, base = call * 64u;
        const char* fillp = lane == 0 ? myws + OFF_FILL + dr * 128u : myws + OFF_TMO;
#pragma unroll 1
        for (int done = 2; done <= NSTEP; done += 2) {
            __builtin_amdgcn_s_barrier();
            if (done == NSTEP - NSPEC - 4) {
                const unsigned long long t0 = now();
                for (;;) {
                    const unsigned v = ld_sys(fillp);
                    if (__builtin_amdgcn_readlane(v, 0) >= call) break;
                    if (__builtin_amdgcn_readlane(v, 1) == call || now() - t0 > DEADLINE) { if (lane == 0) st_agent(myws + OFF_TMO, call); break; }
                }
            }
            const bool pub = done <= NSTEP - NSPEC ? (done % 4 == 0 || done > NSTEP - NSPEC - 12) : done == NSTEP;
            if (lane == 0 && pub) st_sys(flag, base + (unsigned)done);
        }
        return;
    }

    if (b < N_PROD) {
        const unsigned src = (rank + 1u + p) & 7u;
        if (wave >= 8 || src == rank) return;
        unsigned tk = 0;
        if (tid == 0) tk = draw_ticket(myws, b);
        const rsrc_t dsrd = make_rsrc(myws + OFF_ST + src * ST_BYTES + (NSTEP - NSPEC) * ST_STEP, NSPEC * ST_STEP);
        const u32x4 ones = {~0u, ~0u, ~0u, ~0u};
#pragma unroll 1
        for (unsigned i = 0; i < NSPEC * 5u; ++i) {
            bst16_sys(ones, tid * 16u, dsrd, i * 8192u);
            WAIT_VM(8);
        }
        asm volatile("s_waitcnt vmcnt(0)" : "+v"(tk) :: "memory");
        __builtin_amdgcn_s_barrier();
        if (tid == 0) st_sys(P.ws[src] + OFF_FILL + rank * 128u, tk + 1u);
        return;
    }

    const unsigned gi = (unsigned)b - N_PROD;
    auto tile_unit = [](unsigned t, unsigned cu, unsigned h) {
        const unsigned e = 2u * t + h;
        if (e < 20u) return e;
        if (cu < 4u) return e;
        return e < 29u ? e + (e - 20u >= 3u * (cu - 4u) ? 3u : 0u) : 35u;
    };
    const unsigned xq = (gi & 7u) >> 1, slot = gi >> 3, n = xq * 17u - (xq == 3u ? 1u : 0u) + slot;
    if (slot == 16u && xq >= 2u) return;
    const unsigned t = n < 40u ? n >> 2 : (n < 56u ? 10u + ((n - 40u) >> 1) : 10u + ((n - 56u) >> 1));
    const unsigned c = (gi & 1u) * 2u + (n & 1u) + ((n < 40u ? (n >> 1) & 1u : (n < 56u ? 0u : 1u)) << 2);
    const unsigned gidx = rank * 132u + (gi & 1u) * 66u + n;
    unsigned a0, a1, lu0, lu1;
    const unsigned u1 = tile_unit(t, c, 1);
    const bool half = u1 == 35u;
    unit_src(tile_unit(t, c, 0), a0, lu0); unit_src(u1, a1, lu1);
    const bool own0 = a0 == rank, own1 = a1 == rank;
    if (tid < 32) ctl[tid] = ((tid == 8 && own0) || (tid == 9 && own1) || (tid == 10 && c < 4u) || (half && (tid == 1 || tid == 6 || tid == 7))) ? (unsigned)NSTEP : 0u;
    BARRIER();

    auto hrow = [&](unsigned u) {
        const unsigned i5 = u / 5u, i3 = (u - 20u) / 3u;
        return u < 20u ? (og + i5) * 512u + (u - 5u * i5) * 64u : (grp * 4u + i3) * 512u + (5u + u - 20u - 3u * i3) * 64u;
    };
    const unsigned nh = c >= 4u || t > 14u ? 0u : (t == 14u ? 1u : 2u);
    const unsigned hr0 = nh > 0u ? hrow(tile_unit(t, 4u + me, 0)) : 0u, hr1 = nh > 1u ? hrow(tile_unit(t, 4u + me, 1)) : hr0;
    f32x16 acc[2][2];
    if (wave < 4) {
        const unsigned wm = (unsigned)wave >> 1, wn = (unsigned)wave & 1u;
        const unsigned h = (unsigned)lane >> 5;
        unsigned xa[4], wa[2][2];
#pragma unroll
        for (int ks = 0; ks < 4; ++ks) xa[ks] = (wm * 64u + (lane & 31u)) * 128u + (((2u * ks + h) ^ (((unsigned)lane >> 1) & 7u)) << 4);
        {
            const unsigned blk = ((unsigned)lane >> 4) & 1u, q = ((unsigned)lane & 15u) >> 2, pp = (unsigned)lane & 3u;
#pragma unroll
            for (int nt = 0; nt < 2; ++nt)
#pragma unroll
                for (int tt = 0; tt < 2; ++tt) {
                    const unsigned row = 8u * h + 4u * tt + q, ch = 4u * (2u * wn + nt) + 2u * blk + (pp >> 1);
                    wa[nt][tt] = 16384u + 256u * row + ((ch ^ (((row & 3u) << 2) | ((row >> 2) & 3u))) << 4) + 8u * (pp & 1u);
                }
        }
#pragma unroll
        for (int i = 0; i < 2; ++i)
#pragma unroll
            for (int jj = 0; jj < 2; ++jj)
#pragma unroll
                for (int e = 0; e < 16; ++e) acc[i][jj][e] = 0.f;
        char* lanews = P.ws[lane & 7];
        asm volatile("" : "+v"(lanews));
        unsigned have = 0;
        s16x8 xf[4][2], wf[4][2];
        auto rd = [&](LAS char* S, int ks) {
#pragma unroll
            for (int nt = 0; nt < 2; ++nt) {
                const s16x4 lo = __builtin_amdgcn_ds_read_tr16_b64_v4i16((LAS s16x4*)(S + wa[nt][0] + ks * 4096));
                const s16x4 hi = __builtin_amdgcn_ds_read_tr16_b64_v4i16((LAS s16x4*)(S + wa[nt][1] + ks * 4096));
                wf[ks][nt] = __builtin_shufflevector(lo, hi, 0, 1, 2, 3, 4, 5, 6, 7);
            }
#pragma unroll
            for (int mt = 0; mt < 2; ++mt) xf[ks][mt] = *(LAS s16x8*)(S + xa[ks] + mt * 4096);
            __builtin_amdgcn_sched_barrier(0);
        };
        auto mm = [&](int ks) {
#pragma unroll
            for (int nt = 0; nt < 2; ++nt)
#pragma unroll
                for (int mt = 0; mt < 2; ++mt)
                    acc[nt][mt] = __builtin_amdgcn_mfma_f32_32x32x16_bf16((bf16x8)wf[ks][nt], (bf16x8)xf[ks][mt], acc[nt][mt], 0, 0, 0);
            __builtin_amdgcn_sched_barrier(0);
        };
        __builtin_amdgcn_s_waitcnt(0xC07F);
#pragma unroll 1
        for (int k4 = (half && wm) ? NSTEP : 0; k4 < NSTEP; k4 += NSLOT) {
#pragma unroll
            for (int sl = 0; sl < (int)NSLOT; ++sl) {
                const unsigned k = k4 + sl;
                LAS char* const S = L + sl * SLOT;
                while (have <= k) {
                    const u32x4 f = *(volatile LAS u32x4*)(L + L_FULL);
                    const unsigned fa = wm ? f.y : f.x, fb = f.z < f.w ? f.z : f.w;
                    have = rfl(fa < fb ? fa : fb);
                    if (have <= k) __builtin_amdgcn_s_sleep(1);
                }
                asm volatile("" ::: "memory");
                rd(S, 0); rd(S, 1); mm(0); rd(S, 2); mm(1); rd(S, 3); mm(2);
                __builtin_amdgcn_s_waitcnt(0xC07F);
                asm volatile("" ::: "memory");
                ctl[4 + wave] = k + 1;
                __builtin_amdgcn_sched_barrier(0);
                mm(3);
            }
        }
        float m = 0.f;
#pragma unroll
        for (int i = 0; i < 2; ++i)
#pragma unroll
            for (int jj = 0; jj < 2; ++jj)
#pragma unroll
                for (int e = 0; e < 16; ++e) m = fmaxf(m, fabsf(acc[i][jj][e]));
        const unsigned mu = wave_max_u32(__float_as_uint(m));
        unsigned arr = 0;
        if (lane == 0) {
            __hip_atomic_fetch_max((LAS unsigned*)(L + L_WMAX), mu, __ATOMIC_RELAXED, __HIP_MEMORY_SCOPE_WORKGROUP);
            arr = __hip_atomic_fetch_add((LAS unsigned*)(L + L_WMAX + 4), 1u, __ATOMIC_RELAXED, __HIP_MEMORY_SCOPE_WORKGROUP);
        }
        if (rfl(arr) == 3u) {
            const unsigned mm = ctl[16];
            unsigned cl;
            while ((cl = rfl(ctl[13])) == 0u) { }
            if (lane < NR)
                __hip_atomic_store((gu64*)(lanews + OFF_AMAX + (cl & 1u) * AMAX_BYTES + gidx * GSTR), ((unsigned long long)cl << 32) | mm,
                                   __ATOMIC_RELAXED, __HIP_MEMORY_SCOPE_SYSTEM);
        }
    } else if (wave < 8 && !(half && wave == 5)) {
        const unsigned li = (unsigned)wave - 4u, hf = li & 1u;
        const bool isx = li < 2u;
        const bool own = isx ? (hf ? own1 : own0) : (c < 4u);
        const unsigned luh = hf ? lu1 : lu0;
        const unsigned src = isx ? (hf ? a1 : a0) : og + (c & 3u), stn = isx ? (luh < 5u ? luh : luh - 5u) : 3u + hf;
        const char* base; unsigned bytes;
        if (!own) { base = myws + OFF_ST + src * ST_BYTES + stn * 1024u; bytes = ST_BYTES - stn * 1024u; }
        else if (isx) { base = P.x + luh * UNIT_BYTES; bytes = UNIT_BYTES; }
        else { base = P.w + c * 256u + hf * 32768u; bytes = SHARD_BYTES - c * 256u - hf * 32768u; }
        const u32x4 srd = make_srd(base, bytes);
        const unsigned lin = lane * 16u;
        const unsigned swx = (lane >> 3) * 8192u + ((((unsigned)lane & 7u) ^ ((unsigned)lane >> 4)) << 4);
        const unsigned sww = (lane >> 4) * 1024u + ((((unsigned)lane & 15u) ^ (((unsigned)lane >> 4) << 2)) << 4);
        unsigned vo[4];
#pragma unroll
        for (int i = 0; i < 4; ++i) vo[i] = isx ? (swx ^ ((i & 1) << 6)) : (sww ^ (i << 4));
        const unsigned pstr = isx ? 65536u : 4096u, sstr = isx ? 128u : 65536u;
        const unsigned avw = isx ? 8u + hf : 10u;
        unsigned avail = 0, freed = 0, pub = 0;
        const unsigned nnorm = own ? (unsigned)NSTEP : (unsigned)(NSTEP - NSPEC);
        auto look_free = [&]() {
            const u32x4 f = *(volatile LAS u32x4*)(L + L_FREE);
            const unsigned m0 = f.x < f.y ? f.x : f.y, m1 = f.z < f.w ? f.z : f.w;
            freed = rfl(isx ? (hf ? m1 : m0) : (m0 < m1 ? m0 : m1));
        };
        asm volatile("s_nop 4" ::: "memory");
#pragma unroll 1
        for (unsigned j = 0; j < nnorm; ++j) {
            if (avail <= j) {
                avail = rfl(ctl[avw]);
                if (avail <= j) {
                    WAIT_VM(0);
                    if (pub < j) { pub = j; ctl[li] = pub; }
                    do { __builtin_amdgcn_s_sleep(1); avail = rfl(ctl[avw]); } while (avail <= j);
                }
            }
            while (freed + NSLOT <= j) { look_free(); if (freed + NSLOT <= j) __builtin_amdgcn_s_sleep(1); }
            const unsigned dst = (unsigned)(size_t)(L + (j & (NSLOT - 1)) * SLOT) + li * 8192u;
            if (own) {
#pragma unroll
                for (int p = 0; p < 8; ++p) dma16(dst + p * 1024u, vo[p & 3], srd, j * sstr + p * pstr);
            } else {
#pragma unroll
                for (int p = 0; p < 8; ++p) dma16_sys(dst + p * 1024u, lin, srd, j * ST_STEP + p * ST_PIECE);
            }
            WAIT_VM(16);
            if (j >= 2 && pub < j - 1) { pub = j - 1; ctl[li] = pub; }
        }
        WAIT_VM(8); ctl[li] = nnorm - 1u;
        auto landed = [&](unsigned j) {
            const LAS u32x4* q = (const LAS u32x4*)(L + (j & (NSLOT - 1)) * SLOT + li * 8192u + lin);
            unsigned m = 0;
#pragma unroll
            for (int p = 0; p < 8; ++p) { const u32x4 v = q[p * 64]; m = max(max(m, v.x), max(max(v.y, v.z), v.w)); }
            return !__any(m == ~0u);
        };
        pub = nnorm - 1u;
#pragma unroll 1
        for (unsigned j = nnorm; j < (unsigned)NSTEP; j = pub) {
            look_free();
            if (freed + NSLOT <= j) {
                WAIT_VM(0);
                if (pub < j) { pub = j; ctl[li] = pub; }
                do { __builtin_amdgcn_s_sleep(1); look_free(); } while (freed + NSLOT <= j);
            }
            avail = rfl(ctl[avw]);
            const bool two = j + 1u < (unsigned)NSTEP && freed + NSLOT > j + 1u;
            const unsigned d0 = (unsigned)(size_t)(L + (j & (NSLOT - 1)) * SLOT) + li * 8192u;
            const unsigned d1 = (unsigned)(size_t)(L + ((j + 1u) & (NSLOT - 1)) * SLOT) + li * 8192u;
#pragma unroll
            for (int p = 0; p < 8; ++p) dma16_sys(d0 + p * 1024u, lin, srd, j * ST_STEP + p * ST_PIECE);
            if (two) {
#pragma unroll
                for (int p = 0; p < 8; ++p) dma16_sys(d1 + p * 1024u, lin, srd, (j + 1u) * ST_STEP + p * ST_PIECE);
                WAIT_VM(16);
            } else WAIT_VM(8);
            if (pub < j) { pub = j; ctl[li] = pub; }
            if (two) WAIT_VM(8); else WAIT_VM(0);
            if (avail > j || landed(j)) {
                pub = j + 1u; ctl[li] = pub;
                WAIT_VM(0);
                if (two && (avail > j + 1u || landed(j + 1u))) { pub = j + 2u; ctl[li] = pub; }
            }
            WAIT_VM(0);
        }
        WAIT_VM(0); ctl[li] = NSTEP;
    } else if (wave == 8) {
        unsigned tk = 0;
        if (lane == 0) tk = draw_ticket(myws, b);
        const unsigned call = rfl(tk) + 1u;
        WAIT_VM(0);
        if (lane == 0) ctl[12] = call;
        const unsigned fsrc = lane < 8 ? a0 : (lane < 16 ? a1 : og + (c & 3u));
        const char* fp = lane < 24 ? myws + OFF_FLAG + (fsrc * 8u + (lane & 7)) * FSTR : myws + OFF_TMO;
        const rsrc_t msrd = make_rsrc(P.out[rank], OUT_BYTES);
        const u32x4 mark = {0x80808080u, 0x80808080u, 0x80808080u, 0x80808080u};
        unsigned nmark = 16u - 8u * nh;
        auto put_mark = [&]() {
            bst16_sys(mark, ((unsigned)lane >> 3) * 512u + ((unsigned)lane & 7u) * 16u, msrd, (((nmark & 8u) ? hr0 : hr1) + (nmark & 7u) * 8u) * 512u + c * 128u);
            ++nmark;
        };
        const unsigned long long t0 = now();
        for (;;) {
            if (nmark < 16u && (unsigned)(now() - t0) > MARK_T0 + (nmark * 16u + (gi & 15u)) * MARK_DELTA) put_mark();
            if (nmark == 16u) { WAIT_VM(0); if (lane == 0) ctl[13] = call; nmark = 17u; }
            const unsigned v = ld_sys(fp);
            const int d = (int)(v - call * 64u);
            const unsigned prog = d < 0 ? 0u : (d > NSTEP ? (unsigned)NSTEP : (unsigned)d);
            auto min8 = [&](int l) { const unsigned m0 = min4(prog, l), m1 = min4(prog, l + 4); return m0 < m1 ? m0 : m1; };
            unsigned av0 = own0 ? (unsigned)NSTEP : min8(0), av1 = own1 ? (unsigned)NSTEP : min8(8), avb = c < 4u ? (unsigned)NSTEP : min8(16);
            const bool dead = __builtin_amdgcn_readlane(v, 24) == call || now() - t0 > DEADLINE;
            if (dead) { if (lane == 0) { st_agent(myws + OFF_TMO, call); ctl[11] = 1u; } av0 = av1 = avb = NSTEP; }
            if (lane == 0) { ctl[8] = av0; ctl[9] = av1; ctl[10] = avb; }
            if (av0 == (unsigned)NSTEP && av1 == (unsigned)NSTEP && avb == (unsigned)NSTEP) break;
            const bool lazy = min(min(av0, av1), avb) >= (unsigned)(NSTEP - NSPEC) && nmark > 16u;
            bool done = false;
            for (int i = lazy ? 40 : 1; i > 0 && !done; --i) {
                const u32x4 fl = *(volatile LAS u32x4*)(L + L_FULL);
                done = rfl(fl.x + fl.y + fl.z + fl.w) == 4u * NSTEP;
                __builtin_amdgcn_s_sleep(1);
            }
            if (done) break;
        }
        while (nmark < 16u) {
            if ((unsigned)(now() - t0) > MARK_T0 + (nmark * 16u + (gi & 15u)) * MARK_DELTA || rfl(ctl[4]) > 48u) put_mark(); else __builtin_amdgcn_s_sleep(8);
        }
        if (nmark == 16u) { WAIT_VM(0); if (lane == 0) ctl[13] = call; }
    }

    if (wave >= 8) return;
    if (wave < 4) {
        for (;;) {
            const u32x4 f = *(volatile LAS u32x4*)(L + L_FREE);
            const unsigned m0 = f.x < f.y ? f.x : f.y, m1 = f.z < f.w ? f.z : f.w;
            if (rfl(m0 < m1 ? m0 : m1) >= (unsigned)NSTEP - 2u) break;
            __builtin_amdgcn_s_sleep(2);
        }
        asm volatile("" ::: "memory");
        const unsigned wm = (unsigned)wave >> 1, wn = (unsigned)wave & 1u, h = (unsigned)lane >> 5;
#pragma unroll
        for (int nt = 0; nt < 2; ++nt)
#pragma unroll
            for (int mt = 0; mt < 2; ++mt)
#pragma unroll
                for (int g = 0; g < 4; ++g) {
                    const unsigned m = wm * 64u + mt * 32u + (lane & 31u), ch = 16u * wn + 8u * nt + 2u * g + h;
                    f32x4 q; q.x = acc[nt][mt][4 * g]; q.y = acc[nt][mt][4 * g + 1]; q.z = acc[nt][mt][4 * g + 2]; q.w = acc[nt][mt][4 * g + 3];
                    *(LAS f32x4*)(L + m * 512u + ((ch ^ (m & 7u)) << 4)) = q;
                }
    }
    const unsigned orank = (c < 4u ? rank : og + (c & 3u)) & 7u;
    const unsigned ocol = (c < 4u ? c : me) * 128u;
    char* optr = P.out[orank];
    asm volatile("" : "+s"(optr));
    unsigned call = 0, mx = 0;
    const char* gp = myws + OFF_AMAX + ((unsigned)tid & 255u) * GSTR;
    unsigned skip = 0;
#pragma unroll
    for (int j = 0; j < 5; ++j) { const unsigned g = j * 256u + ((unsigned)tid & 255u); skip |= (g >= 8u * 132u || g == gidx) ? 1u << j : 0u; }
    unsigned long long tg0 = 0;
    bool ok = false;
    auto sweep = [&]() {
        const unsigned tm = ld_agent(myws + OFF_TMO);
        if (!ok) {
            unsigned long long g[5];
#pragma unroll
            for (int j = 0; j < 4; ++j) g[j] = __hip_atomic_load((gu64*)(gp + j * 256 * GSTR), __ATOMIC_RELAXED, __HIP_MEMORY_SCOPE_SYSTEM);
            g[4] = 0;
            if (tid < 256 + 32) g[4] = __hip_atomic_load((gu64*)(gp + 4 * 256 * GSTR), __ATOMIC_RELAXED, __HIP_MEMORY_SCOPE_SYSTEM);
            ok = true; mx = 0;
#pragma unroll
            for (int j = 0; j < 5; ++j) {
                const bool mine = (skip >> j) & 1u;
                ok = ok && (mine || (unsigned)(g[j] >> 32) >= call);
                mx = max(mx, mine ? 0u : (unsigned)g[j]);
            }
        }
        if (rfl(tm) == call || now() - tg0 > DEADLINE) { if (lane == 0) { st_agent(myws + OFF_TMO, call); ctl[11] = 1u; } return true; }
        return (bool)__all(ok);
    };
    if (wave >= 4) {
        while ((call = rfl(ctl[12])) == 0u) __builtin_amdgcn_s_sleep(2);
        gp += (call & 1u) * AMAX_BYTES;
    }
    BARRIER();
    f32x4 y[2][4];
    const unsigned r0 = (unsigned)tid >> 3, q8 = (unsigned)tid & 7u;
#pragma unroll
    for (int hf = 0; hf < 2; ++hf)
#pragma unroll
        for (int e = 0; e < 4; ++e) y[hf][e] = *(LAS f32x4*)(L + (r0 + 64u * hf) * 512u + (((4u * q8 + e) ^ (r0 & 7u)) << 4));
    if (wave >= 4) {
        tg0 = now();
        while (!sweep()) { }
        mx = wave_max_u32(mx);
        if (lane == 0) ctl[20 + wave] = mx;
    }
    BARRIER();
    const unsigned long long tq = now();
    const u32x4 gmx = *(volatile LAS u32x4*)(L + L_GMAX);
    const unsigned ga = max(max(gmx.x, gmx.y), max(gmx.z, gmx.w)), gb = ctl[16];
    const bool aborted = ctl[11] != 0u;
    const float amax = __uint_as_float(ga > gb ? ga : gb);
    const float inv = amax > 0.f ? 127.0f / amax : 0.f;
    const rsrc_t osrd = make_rsrc(optr, OUT_BYTES);
    const unsigned ovoff = r0 * 512u + q8 * 16u;
    constexpr float MAGIC = 12582912.0f;
#pragma unroll
    for (int hf = 0; hf < 2; ++hf) {
        u32x4 o;
#pragma unroll
        for (int e = 0; e < 4; ++e) {
            unsigned z[4];
#pragma unroll
            for (int i = 0; i < 4; ++i)
                z[i] = __float_as_uint(__builtin_amdgcn_fmed3f(__builtin_fmaf(y[hf][e][i], inv, MAGIC), MAGIC - 127.0f, MAGIC + 127.0f));
            const unsigned lo = __builtin_amdgcn_perm(z[1], z[0], 0x0c0c0400u), hi = __builtin_amdgcn_perm(z[3], z[2], 0x04000c0cu);
            o[e] = aborted ? 0x80808080u : (lo | hi);
        }
        const unsigned row = hf ? a1 * 512u + lu1 * 64u : a0 * 512u + lu0 * 64u;
        if (hf && half) break;
        if (c < 4u) __builtin_amdgcn_raw_buffer_store_b128(o, osrd, ovoff, row * 512u + ocol, 16);
        else {
            const unsigned due = ((unsigned)(hf * 8 + wave) * 15u + t) * OUT_DELTA_NS / 160u;
            asm volatile("" : "+v"(o));
            while ((unsigned)(now() - tq) < due) __builtin_amdgcn_s_sleep(1);
            bst16_sys(o, ovoff, osrd, row * 512u + ocol);
        }
    }
    if (nh > 0u) {
        call = rfl(ctl[12]);
        const unsigned due = ((unsigned)((nh > 1u ? 8 : 0) + wave) * 15u + t) * OUT_DELTA_NS / 160u + OUT_CHECK_DELAY;
        while ((unsigned)(now() - tq) < due) __builtin_amdgcn_s_sleep(2);
        bool in = false;
        for (;;) {
            const unsigned tm = ld_agent(myws + OFF_TMO);
            if (!in) {
                const u32x4 h0 = __builtin_amdgcn_raw_buffer_load_b128(osrd, ovoff, hr0 * 512u + ocol, 17);
                const u32x4 h1 = __builtin_amdgcn_raw_buffer_load_b128(osrd, ovoff, hr1 * 512u + ocol, 17);
                unsigned bad = 0;
#pragma unroll
                for (int e = 0; e < 4; ++e) {
                    const unsigned x0 = h0[e] ^ 0x80808080u, x1 = h1[e] ^ 0x80808080u;
                    bad |= ((x0 - 0x01010101u) & ~x0) | ((x1 - 0x01010101u) & ~x1);
                }
                in = (bad & 0x80808080u) == 0u;
            }
            if (__all(in)) break;
            if (rfl(tm) == call || now() - tq > DEADLINE) { if (lane == 0) st_agent(myws + OFF_TMO, call); break; }
            asm volatile("" ::: "memory");
        }
    }
}

extern "C" void kernel_launch(int rank, int nranks, ncclComm_t comm,
                              hipStream_t stream,
                              void* const* d_in, const int* in_sizes, int n_in,
                              void* d_out, int out_size,
                              const int* in_dtypes, int out_dtype,
                              void* const* d_out_all,
                              void* const* d_ws_all, size_t ws_bytes) {
    (void)comm; (void)in_dtypes; (void)out_dtype; (void)d_out;
    if (nranks != NR || (unsigned)rank >= (unsigned)NR || n_in < 2 || ws_bytes < WS_NEEDED ||
        in_sizes[0] != 512 * 4096 || in_sizes[1] != 4096 * 512 || out_size != (int)OUT_BYTES) {
        fprintf(stderr, "kernel_launch: unexpected problem shape\n");
        return;
    }
    int dev = 0, cus = 0;
    HIP_CHECK(hipGetDevice(&dev));
    HIP_CHECK(hipDeviceGetAttribute(&cus, hipDeviceAttributeMultiprocessorCount, dev));
    if (cus < GRID) { fprintf(stderr, "kernel_launch: needs %d CUs\n", GRID); return; }
    Params P;
    P.x = (const char*)d_in[0];
    P.w = (const char*)d_in[1];
    for (int p = 0; p < NR; ++p) { P.ws[p] = (char*)d_ws_all[p]; P.out[p] = (char*)d_out_all[p]; }
    for (unsigned l = 0; l < (unsigned)NR; ++l) {
        const unsigned dr = link_rank((unsigned)rank, l < 7u ? l : 0u);
        P.link[l].x = P.x;
        P.link[l].wq = P.w + (dr & 3u) * 256u;
        P.link[l].wq_bytes = SHARD_BYTES - (dr & 3u) * 256u;
        P.link[l].dst = P.ws[dr] + OFF_ST + (unsigned)rank * ST_BYTES;
    }
    P.rank = rank;
    hipLaunchKernelGGL(ag_gemm_q8, dim3(GRID), dim3(NTHREADS), 0, stream, P);
    HIP_CHECK(hipGetLastError());
}
```

```cpp
#include <hip/hip_runtime.h>
#include <rccl/rccl.h>
#include <cstdio>
#include <cstdint>

#define HIP_CHECK(expr) do { hipError_t _e = (expr); if (_e != hipSuccess) { \
    fprintf(stderr, "HIP error %s at %s:%d\n", hipGetErrorString(_e), __FILE__, __LINE__); \
    return; } } while (0)

typedef __attribute__((ext_vector_type(4))) unsigned u32x4;
typedef __attribute__((ext_vector_type(4))) float f32x4;
typedef __attribute__((ext_vector_type(16))) float f32x16;
typedef __attribute__((ext_vector_type(4))) short s16x4;
typedef __attribute__((ext_vector_type(8))) short s16x8;
typedef __attribute__((ext_vector_type(8))) __bf16 bf16x8;
#define LAS __attribute__((address_space(3)))
#define GAS __attribute__((address_space(1)))
typedef GAS unsigned gu32;
typedef GAS unsigned long long gu64;

constexpr int NR = 8;
constexpr int NSTEP = 64;
constexpr int N_PROD = 64;
constexpr int N_GEMM = 136;
constexpr int GRID = N_PROD + N_GEMM;
constexpr int NTHREADS = 576;
constexpr int PUSH_T0 = 30, PUSH_TAU_NS = 688;
constexpr int PUSH_GAP = 64;
constexpr int LAT_HI = 85, OFF_INC = 2, OFF_INC_MAX = 8, OFF_SHIFT = 3, OFF_DEC = 1, OFF_CAP = 400;
constexpr unsigned OUT_DELTA_NS = 262;
constexpr unsigned OUT_CHECK_DELAY = 60;
constexpr unsigned MARK_T0 = 300, MARK_DELTA = 8;
constexpr unsigned FSTR = 128, GSTR = 64;
constexpr int NSPEC = 24;
constexpr unsigned long long DEADLINE = 100000000ull;

constexpr unsigned OFF_CTR = 0;
constexpr unsigned OFF_TMO = 0x8000;
constexpr unsigned OFF_FILL = 0x9000;
constexpr unsigned OFF_FLAG = 0xE000;
constexpr unsigned OFF_AMAX = 0x10000;
constexpr unsigned AMAX_BYTES = 1280 * GSTR;
constexpr unsigned UNIT_BYTES = 64u * 8192u;
constexpr unsigned OFF_ST = 1u << 20;
constexpr unsigned ST_PIECE = 5u * 1024u, ST_STEP = 8u * ST_PIECE, ST_BYTES = 64u * ST_STEP;
constexpr unsigned WS_NEEDED = OFF_ST + 8u * ST_BYTES;
constexpr unsigned SHARD_BYTES = 4u << 20;
constexpr unsigned OUT_BYTES = 4096u * 512u;
static_assert(NSPEC % 2 == 0 && NSPEC <= NSTEP - 6 && GRID * 128 <= (int)OFF_TMO && OFF_AMAX + 2 * AMAX_BYTES <= OFF_ST, "workspace map");

constexpr unsigned SLOT = 32768, NSLOT = 4;
constexpr unsigned L_CTRL = SLOT * NSLOT;
constexpr unsigned L_FULL = L_CTRL, L_FREE = L_CTRL + 16, L_WMAX = L_CTRL + 64, L_GMAX = L_CTRL + 96;
constexpr unsigned LDS_BYTES = L_CTRL + 128;

struct alignas(32) LinkArgs {
    const char* x;
    const char* wq;
    char* dst;
    unsigned long long wq_bytes;
};
struct Params {
    LinkArgs link[NR];
    const char* x;
    const char* w;
    char* ws[NR];
    char* out[NR];
    int rank;
};

__host__ __device__ __forceinline__ unsigned link_rank(unsigned rank, unsigned l) {
    return l < 3u ? (rank & 4u) | ((rank + 1u + l) & 3u) : ((rank & 4u) ^ 4u) | ((rank + l - 3u) & 3u);
}
__device__ __forceinline__ unsigned rfl(unsigned v) { return __builtin_amdgcn_readfirstlane(v); }
__device__ __forceinline__ u32x4 make_srd(const void* p, unsigned bytes) {
    const unsigned long long a = (unsigned long long)p;
    u32x4 r;
    r.x = rfl((unsigned)a); r.y = rfl((unsigned)(a >> 32) & 0xffffu); r.z = rfl(bytes); r.w = 0x00020000u;
    return r;
}
__device__ __forceinline__ void dma16(unsigned lds, unsigned voff, u32x4 srd, unsigned soff) {
    asm volatile("s_mov_b32 m0, %0\n\ts_nop 0\n\tbuffer_load_dwordx4 %1, %2, %3 offen lds"
                 :: "s"(lds), "v"(voff), "s"(srd), "s"(soff) : "memory");
}
__device__ __forceinline__ void dma16_sys(unsigned lds, unsigned voff, u32x4 srd, unsigned soff) {
    asm volatile("s_mov_b32 m0, %0\n\ts_nop 0\n\tbuffer_load_dwordx4 %1, %2, %3 offen sc0 sc1 lds"
                 :: "s"(lds), "v"(voff), "s"(srd), "s"(soff) : "memory");
}
typedef __amdgpu_buffer_rsrc_t rsrc_t;
__device__ __forceinline__ rsrc_t make_rsrc(const void* p, unsigned bytes) {
    const unsigned long long a = (unsigned long long)p;
    const unsigned long long u = ((unsigned long long)rfl((unsigned)(a >> 32)) << 32) | rfl((unsigned)a);
    return __builtin_amdgcn_make_buffer_rsrc((void*)u, 0, rfl(bytes), 0x00020000);
}
__device__ __forceinline__ u32x4 bld16(unsigned voff, rsrc_t r, unsigned soff) { return __builtin_amdgcn_raw_buffer_load_b128(r, voff, soff, 0); }
__device__ __forceinline__ void bst16_sys(u32x4 v, unsigned voff, rsrc_t r, unsigned soff) {
    __builtin_amdgcn_raw_buffer_store_b128(v, r, voff, soff, 17);
}
#define WAIT_VM(n) asm volatile("s_waitcnt vmcnt(" #n ")" ::: "memory")
#define BARRIER() do { asm volatile("s_waitcnt lgkmcnt(0)" ::: "memory"); __builtin_amdgcn_s_barrier(); asm volatile("" ::: "memory"); } while (0)
__device__ __forceinline__ unsigned long long now() { return __builtin_amdgcn_s_memrealtime(); }
__device__ __forceinline__ unsigned ld_sys(const void* p) { return __hip_atomic_load((gu32*)p, __ATOMIC_RELAXED, __HIP_MEMORY_SCOPE_SYSTEM); }
__device__ __forceinline__ void st_sys(void* p, unsigned v) { __hip_atomic_store((gu32*)p, v, __ATOMIC_RELAXED, __HIP_MEMORY_SCOPE_SYSTEM); }
__device__ __forceinline__ unsigned ld_agent(const void* p) { return __hip_atomic_load((gu32*)p, __ATOMIC_RELAXED, __HIP_MEMORY_SCOPE_AGENT); }
__device__ __forceinline__ void st_agent(void* p, unsigned v) { __hip_atomic_store((gu32*)p, v, __ATOMIC_RELAXED, __HIP_MEMORY_SCOPE_AGENT); }
__device__ __forceinline__ unsigned draw_ticket(char* ws, int b) {
    unsigned z = 0;
    asm volatile("" : "+v"(z));
    return __hip_atomic_fetch_add((gu32*)(ws + OFF_CTR + b * 128 + z), 1u, __ATOMIC_RELAXED, __HIP_MEMORY_SCOPE_AGENT);
}
__device__ __forceinline__ unsigned wave_max_u32(unsigned v) {
#define DPP_MAX(ctrl, rmask) do { const unsigned o_ = (unsigned)__builtin_amdgcn_update_dpp(0, (int)v, ctrl, rmask, 0xf, false); v = o_ > v ? o_ : v; } while (0)
    DPP_MAX(0x111, 0xf); DPP_MAX(0x112, 0xf); DPP_MAX(0x114, 0xf); DPP_MAX(0x118, 0xf);
    DPP_MAX(0x142, 0xa); DPP_MAX(0x143, 0xc);
#undef DPP_MAX
    return __builtin_amdgcn_readlane(v, 63);
}

__global__ void __launch_bounds__(NTHREADS, 3) ag_gemm_q8(Params P) {
    __shared__ __attribute__((aligned(16))) char smem[LDS_BYTES];
    LAS char* const L = (LAS char*)smem;
    const unsigned long long ttop = now();
    const int b = blockIdx.x;
    const int tid = threadIdx.x, lane = tid & 63;
    const int wave = (int)rfl(tid >> 6);
    const unsigned link = ((unsigned)b + 7u) & 7u, p = ((unsigned)b >> 3) & 7u;
    const bool pusher = b < N_PROD && link < 7u;
    if (__builtin_expect(pusher && wave < 5, 1)) {
        const unsigned st = (unsigned)wave;
        const LinkArgs A = P.link[link];
        const bool intra = link < 3u, isw = !intra && st >= 3u;
        const unsigned lu = intra ? st : 5u + st;
        const unsigned pw = 8u * (st - 3u) + p;
        const rsrc_t ssrd = make_rsrc(isw ? A.wq : A.x, isw ? (unsigned)A.wq_bytes : SHARD_BYTES);
        const rsrc_t dsrd = make_rsrc(A.dst, ST_BYTES);
        const unsigned svoff = isw ? (4u * pw + (lane >> 4)) * 1024u + ((((unsigned)lane & 15u) ^ (((unsigned)lane >> 4) << 2) ^ (pw & 3u)) << 4)
                                   : (lu * 64u + 8u * p + (lane >> 3)) * 8192u + ((((unsigned)lane & 7u) ^ ((unsigned)lane >> 4) ^ ((p & 1u) << 2)) << 4);
        const unsigned dvoff = p * ST_PIECE + st * 1024u + lane * 16u;
        const unsigned sstr = isw ? 65536u : 128u;
        u32x4 v0 = bld16(svoff, ssrd, 0), v1 = bld16(svoff, ssrd, sstr);
        constexpr int TAU8 = PUSH_TAU_NS * 8 / 10;
        int sched = PUSH_T0 * 8 + (int)(p * 5u + st) * TAU8 / 40, floor_t = 0, off = 0, ti0 = 0, ti1 = 0;
        auto clock8 = [&]() { return (int)((unsigned)(now() - ttop) << 3); };
        auto pace = [&](int& ti, bool acked) {
            int tn = clock8();
            if (acked) {
                const int lat = tn - ti;
                const int inc = OFF_INC * 8 + ((lat - LAT_HI * 8) >> OFF_SHIFT);
                if (lat > LAT_HI * 8) { if (off < OFF_CAP * 8) off += inc < OFF_INC_MAX * 8 ? inc : OFF_INC_MAX * 8; } else if (off >= OFF_DEC * 8) off -= OFF_DEC * 8;
            }
            const int due = sched + off > floor_t ? sched + off : floor_t;
            while (tn < due) { __builtin_amdgcn_s_sleep(1); tn = clock8(); }
            floor_t = tn + PUSH_GAP * 8;
            sched += TAU8;
            ti = tn;
        };
        asm volatile("" : "+v"(v0));
        pace(ti0, false);
        bst16_sys(v0, dvoff, dsrd, 0); v0 = bld16(svoff, ssrd, 2u * sstr);
        pace(ti1, false);
        bst16_sys(v1, dvoff, dsrd, ST_STEP); v1 = bld16(svoff, ssrd, 3u * sstr);
#pragma unroll 1
        for (unsigned i = 2; i < (unsigned)NSTEP; i += 2) {
            WAIT_VM(2);
            pace(ti0, true);
            bst16_sys(v0, dvoff, dsrd, i * ST_STEP);
            v0 = bld16(svoff, ssrd, (i + 2u < (unsigned)NSTEP ? i + 2u : i) * sstr);
            WAIT_VM(2);
            pace(ti1, true);
            bst16_sys(v1, dvoff, dsrd, (i + 1u) * ST_STEP);
            v1 = bld16(svoff, ssrd, (i + 3u < (unsigned)NSTEP ? i + 3u : i) * sstr);
            asm volatile("" ::: "memory");
            __builtin_amdgcn_s_barrier();
        }
        WAIT_VM(0);
        __builtin_amdgcn_s_barrier();
        asm volatile("" :: "v"(v0), "v"(v1));
        return;
    }
    const unsigned rank = (unsigned)P.rank & 7u;
    const unsigned grp = rank >> 2, me = rank & 3u, og = (grp ^ 1u) * 4u;
    char* const myws = P.ws[rank];

    volatile LAS unsigned* const ctl = (volatile LAS unsigned*)(L + L_CTRL);
    auto unit_src = [&](unsigned u, unsigned& a, unsigned& lu) {
        if (u < 20u) { const unsigned i = u / 5u; a = grp * 4u + i; lu = u - 5u * i; }
        else if (u < 32u) { const unsigned i = (u - 20u) / 3u; a = og + i; lu = 5u + (u - 20u) - 3u * i; }
        else { a = rank; lu = u < 35u ? u - 27u : 0u; }
        a &= 7u;
    };
    auto min4 = [](unsigned v, int l) {
        const unsigned p0 = __builtin_amdgcn_readlane(v, l), p1 = __builtin_amdgcn_readlane(v, l + 1),
                       p2 = __builtin_amdgcn_readlane(v, l + 2), p3 = __builtin_amdgcn_readlane(v, l + 3);
        const unsigned m0 = p0 < p1 ? p0 : p1, m1 = p2 < p3 ? p2 : p3;
        return m0 < m1 ? m0 : m1;
    };
    if (pusher) {
        if (wave > 5) return;
        const unsigned dr = link_rank(rank, link);
        char* const flag = P.ws[dr] + OFF_FLAG + (rank * 8u + p) * FSTR;
        unsigned tk = 0;
        if (lane == 0) tk = draw_ticket(myws, b);
        const unsigned call = rfl(tk) + 1u, base = call * 64u;
        const char* fillp = lane == 0 ? myws + OFF_FILL + dr * 128u : myws + OFF_TMO;
#pragma unroll 1
        for (int done = 2; done <= NSTEP; done += 2) {
            __builtin_amdgcn_s_barrier();
            if (done == NSTEP - NSPEC - 4) {
                const unsigned long long t0 = now();
                for (;;) {
                    const unsigned v = ld_sys(fillp);
                    if (__builtin_amdgcn_readlane(v, 0) >= call) break;
                    if (__builtin_amdgcn_readlane(v, 1) == call || now() - t0 > DEADLINE) { if (lane == 0) st_agent(myws + OFF_TMO, call); break; }
                }
            }
            const bool pub = done <= NSTEP - NSPEC ? (done % 4 == 0 || done > NSTEP - NSPEC - 12) : done == NSTEP;
            if (lane == 0 && pub) st_sys(flag, base + (unsigned)done);
        }
        return;
    }

    if (b < N_PROD) {
        const unsigned src = (rank + 1u + p) & 7u;
        if (wave >= 8 || src == rank) return;
        unsigned tk = 0;
        if (tid == 0) tk = draw_ticket(myws, b);
        const rsrc_t dsrd = make_rsrc(myws + OFF_ST + src * ST_BYTES + (NSTEP - NSPEC) * ST_STEP, NSPEC * ST_STEP);
        const u32x4 ones = {~0u, ~0u, ~0u, ~0u};
#pragma unroll 1
        for (unsigned i = 0; i < NSPEC * 5u; ++i) {
            bst16_sys(ones, tid * 16u, dsrd, i * 8192u);
            WAIT_VM(8);
        }
        asm volatile("s_waitcnt vmcnt(0)" : "+v"(tk) :: "memory");
        __builtin_amdgcn_s_barrier();
        if (tid == 0) st_sys(P.ws[src] + OFF_FILL + rank * 128u, tk + 1u);
        return;
    }

    const unsigned gi = (unsigned)b - N_PROD;
    auto tile_unit = [](unsigned t, unsigned cu, unsigned h) {
        const unsigned e = 2u * t + h;
        if (e < 20u) return e;
        if (cu < 4u) return e;
        return e < 29u ? e + (e - 20u >= 3u * (cu - 4u) ? 3u : 0u) : 35u;
    };
    const unsigned xq = (gi & 7u) >> 1, slot = gi >> 3, n = xq * 17u - (xq == 3u ? 1u : 0u) + slot;
    if (slot == 16u && xq >= 2u) return;
    const unsigned t = n < 40u ? n >> 2 : (n < 56u ? 10u + ((n - 40u) >> 1) : 10u + ((n - 56u) >> 1));
    const unsigned c = (gi & 1u) * 2u + (n & 1u) + ((n < 40u ? (n >> 1) & 1u : (n < 56u ? 0u : 1u)) << 2);
    const unsigned gidx = rank * 132u + (gi & 1u) * 66u + n;
    unsigned a0, a1, lu0, lu1;
    const unsigned u1 = tile_unit(t, c, 1);
    const bool half = u1 == 35u;
    unit_src(tile_unit(t, c, 0), a0, lu0); unit_src(u1, a1, lu1);
    const bool own0 = a0 == rank, own1 = a1 == rank;
    if (tid < 32) ctl[tid] = ((tid == 8 && own0) || (tid == 9 && own1) || (tid == 10 && c < 4u) || (half && (tid == 1 || tid == 6 || tid == 7))) ? (unsigned)NSTEP : 0u;
    BARRIER();

    auto hrow = [&](unsigned u) {
        const unsigned i5 = u / 5u, i3 = (u - 20u) / 3u;
        return u < 20u ? (og + i5) * 512u + (u - 5u * i5) * 64u : (grp * 4u + i3) * 512u + (5u + u - 20u - 3u * i3) * 64u;
    };
    const unsigned nh = c >= 4u || t > 14u ? 0u : (t == 14u ? 1u : 2u);
    const unsigned hr0 = nh > 0u ? hrow(tile_unit(t, 4u + me, 0)) : 0u, hr1 = nh > 1u ? hrow(tile_unit(t, 4u + me, 1)) : hr0;
    f32x16 acc[2][2];
    if (wave < 4) {
        const unsigned wm = (unsigned)wave >> 1, wn = (unsigned)wave & 1u;
        const unsigned h = (unsigned)lane >> 5;
        unsigned xa[4], wa[2][2];
#pragma unroll
        for (int ks = 0; ks < 4; ++ks) xa[ks] = (wm * 64u + (lane & 31u)) * 128u + (((2u * ks + h) ^ (((unsigned)lane >> 1) & 7u)) << 4);
        {
            const unsigned blk = ((unsigned)lane >> 4) & 1u, q = ((unsigned)lane & 15u) >> 2, pp = (unsigned)lane & 3u;
#pragma unroll
            for (int nt = 0; nt < 2; ++nt)
#pragma unroll
                for (int tt = 0; tt < 2; ++tt) {
                    const unsigned row = 8u * h + 4u * tt + q, ch = 4u * (2u * wn + nt) + 2u * blk + (pp >> 1);
                    wa[nt][tt] = 16384u + 256u * row + ((ch ^ (((row & 3u) << 2) | ((row >> 2) & 3u))) << 4) + 8u * (pp & 1u);
                }
        }
#pragma unroll
        for (int i = 0; i < 2; ++i)
#pragma unroll
            for (int jj = 0; jj < 2; ++jj)
#pragma unroll
                for (int e = 0; e < 16; ++e) acc[i][jj][e] = 0.f;
        char* lanews = P.ws[lane & 7];
        asm volatile("" : "+v"(lanews));
        unsigned have = 0;
        s16x8 xf[4][2], wf[4][2];
        auto rd = [&](LAS char* S, int ks) {
#pragma unroll
            for (int nt = 0; nt < 2; ++nt) {
                const s16x4 lo = __builtin_amdgcn_ds_read_tr16_b64_v4i16((LAS s16x4*)(S + wa[nt][0] + ks * 4096));
                const s16x4 hi = __builtin_amdgcn_ds_read_tr16_b64_v4i16((LAS s16x4*)(S + wa[nt][1] + ks * 4096));
                wf[ks][nt] = __builtin_shufflevector(lo, hi, 0, 1, 2, 3, 4, 5, 6, 7);
            }
#pragma unroll
            for (int mt = 0; mt < 2; ++mt) xf[ks][mt] = *(LAS s16x8*)(S + xa[ks] + mt * 4096);
            __builtin_amdgcn_sched_barrier(0);
        };
        auto mm = [&](int ks) {
#pragma unroll
            for (int nt = 0; nt < 2; ++nt)
#pragma unroll
                for (int mt = 0; mt < 2; ++mt)
                    acc[nt][mt] = __builtin_amdgcn_mfma_f32_32x32x16_bf16((bf16x8)wf[ks][nt], (bf16x8)xf[ks][mt], acc[nt][mt], 0, 0, 0);
            __builtin_amdgcn_sched_barrier(0);
        };
        __builtin_amdgcn_s_waitcnt(0xC07F);
#pragma unroll 1
        for (int k4 = (half && wm) ? NSTEP : 0; k4 < NSTEP; k4 += NSLOT) {
#pragma unroll
            for (int sl = 0; sl < (int)NSLOT; ++sl) {
                const unsigned k = k4 + sl;
                LAS char* const S = L + sl * SLOT;
                while (have <= k) {
                    const u32x4 f = *(volatile LAS u32x4*)(L + L_FULL);
                    const unsigned fa = wm ? f.y : f.x, fb = f.z < f.w ? f.z : f.w;
                    have = rfl(fa < fb ? fa : fb);
                    if (have <= k) __builtin_amdgcn_s_sleep(1);
                }
                asm volatile("" ::: "memory");
                rd(S, 0); rd(S, 1); mm(0); rd(S, 2); mm(1); rd(S, 3); mm(2);
                __builtin_amdgcn_s_waitcnt(0xC07F);
                asm volatile("" ::: "memory");
                ctl[4 + wave] = k + 1;
                __builtin_amdgcn_sched_barrier(0);
                mm(3);
            }
        }
        float m = 0.f;
#pragma unroll
        for (int i = 0; i < 2; ++i)
#pragma unroll
            for (int jj = 0; jj < 2; ++jj)
#pragma unroll
                for (int e = 0; e < 16; ++e) m = fmaxf(m, fabsf(acc[i][jj][e]));
        const unsigned mu = wave_max_u32(__float_as_uint(m));
        unsigned arr = 0;
        if (lane == 0) {
            __hip_atomic_fetch_max((LAS unsigned*)(L + L_WMAX), mu, __ATOMIC_RELAXED, __HIP_MEMORY_SCOPE_WORKGROUP);
            arr = __hip_atomic_fetch_add((LAS unsigned*)(L + L_WMAX + 4), 1u, __ATOMIC_RELAXED, __HIP_MEMORY_SCOPE_WORKGROUP);
        }
        if (rfl(arr) == 3u) {
            const unsigned mm = ctl[16];
            unsigned cl;
            while ((cl = rfl(ctl[13])) == 0u) { }
            if (lane < NR)
                __hip_atomic_store((gu64*)(lanews + OFF_AMAX + (cl & 1u) * AMAX_BYTES + gidx * GSTR), ((unsigned long long)cl << 32) | mm,
                                   __ATOMIC_RELAXED, __HIP_MEMORY_SCOPE_SYSTEM);
        }
    } else if (wave < 8 && !(half && wave == 5)) {
        const unsigned li = (unsigned)wave - 4u, hf = li & 1u;
        const bool isx = li < 2u;
        const bool own = isx ? (hf ? own1 : own0) : (c < 4u);
        const unsigned luh = hf ? lu1 : lu0;
        const unsigned src = isx ? (hf ? a1 : a0) : og + (c & 3u), stn = isx ? (luh < 5u ? luh : luh - 5u) : 3u + hf;
        const char* base; unsigned bytes;
        if (!own) { base = myws + OFF_ST + src * ST_BYTES + stn * 1024u; bytes = ST_BYTES - stn * 1024u; }
        else if (isx) { base = P.x + luh * UNIT_BYTES; bytes = UNIT_BYTES; }
        else { base = P.w + c * 256u + hf * 32768u; bytes = SHARD_BYTES - c * 256u - hf * 32768u; }
        const u32x4 srd = make_srd(base, bytes);
        const unsigned lin = lane * 16u;
        const unsigned swx = (lane >> 3) * 8192u + ((((unsigned)lane & 7u) ^ ((unsigned)lane >> 4)) << 4);
        const unsigned sww = (lane >> 4) * 1024u + ((((unsigned)lane & 15u) ^ (((unsigned)lane >> 4) << 2)) << 4);
        unsigned vo[4];
#pragma unroll
        for (int i = 0; i < 4; ++i) vo[i] = isx ? (swx ^ ((i & 1) << 6)) : (sww ^ (i << 4));
        const unsigned pstr = isx ? 65536u : 4096u, sstr = isx ? 128u : 65536u;
        const unsigned avw = isx ? 8u + hf : 10u;
        unsigned avail = 0, freed = 0, pub = 0;
        const unsigned nnorm = own ? (unsigned)NSTEP : (unsigned)(NSTEP - NSPEC);
        auto look_free = [&]() {
            const u32x4 f = *(volatile LAS u32x4*)(L + L_FREE);
            const unsigned m0 = f.x < f.y ? f.x : f.y, m1 = f.z < f.w ? f.z : f.w;
            freed = rfl(isx ? (hf ? m1 : m0) : (m0 < m1 ? m0 : m1));
        };
        asm volatile("s_nop 4" ::: "memory");
#pragma unroll 1
        for (unsigned j = 0; j < nnorm; ++j) {
            if (avail <= j) {
                avail = rfl(ctl[avw]);
                if (avail <= j) {
                    WAIT_VM(0);
                    if (pub < j) { pub = j; ctl[li] = pub; }
                    do { __builtin_amdgcn_s_sleep(1); avail = rfl(ctl[avw]); } while (avail <= j);
                }
            }
            while (freed + NSLOT <= j) { look_free(); if (freed + NSLOT <= j) __builtin_amdgcn_s_sleep(1); }
            const unsigned dst = (unsigned)(size_t)(L + (j & (NSLOT - 1)) * SLOT) + li * 8192u;
            if (own) {
#pragma unroll
                for (int p = 0; p < 8; ++p) dma16(dst + p * 1024u, vo[p & 3], srd, j * sstr + p * pstr);
            } else {
#pragma unroll
                for (int p = 0; p < 8; ++p) dma16_sys(dst + p * 1024u, lin, srd, j * ST_STEP + p * ST_PIECE);
            }
            WAIT_VM(16);
            if (j >= 2 && pub < j - 1) { pub = j - 1; ctl[li] = pub; }
        }
        WAIT_VM(8); ctl[li] = nnorm - 1u;
        auto landed = [&](unsigned j) {
            const LAS u32x4* q = (const LAS u32x4*)(L + (j & (NSLOT - 1)) * SLOT + li * 8192u + lin);
            unsigned m = 0;
#pragma unroll
            for (int p = 0; p < 8; ++p) { const u32x4 v = q[p * 64]; m = max(max(m, v.x), max(max(v.y, v.z), v.w)); }
            return !__any(m == ~0u);
        };
        pub = nnorm - 1u;
#pragma unroll 1
        for (unsigned j = nnorm; j < (unsigned)NSTEP; j = pub) {
            look_free();
            if (freed + NSLOT <= j) {
                WAIT_VM(0);
                if (pub < j) { pub = j; ctl[li] = pub; }
                do { __builtin_amdgcn_s_sleep(1); look_free(); } while (freed + NSLOT <= j);
            }
            avail = rfl(ctl[avw]);
            const bool two = j + 1u < (unsigned)NSTEP && freed + NSLOT > j + 1u;
            const unsigned d0 = (unsigned)(size_t)(L + (j & (NSLOT - 1)) * SLOT) + li * 8192u;
            const unsigned d1 = (unsigned)(size_t)(L + ((j + 1u) & (NSLOT - 1)) * SLOT) + li * 8192u;
#pragma unroll
            for (int p = 0; p < 8; ++p) dma16_sys(d0 + p * 1024u, lin, srd, j * ST_STEP + p * ST_PIECE);
            if (two) {
#pragma unroll
                for (int p = 0; p < 8; ++p) dma16_sys(d1 + p * 1024u, lin, srd, (j + 1u) * ST_STEP + p * ST_PIECE);
                WAIT_VM(16);
            } else WAIT_VM(8);
            if (pub < j) { pub = j; ctl[li] = pub; }
            if (two) WAIT_VM(8); else WAIT_VM(0);
            if (avail > j || landed(j)) {
                pub = j + 1u; ctl[li] = pub;
                WAIT_VM(0);
                if (two && (avail > j + 1u || landed(j + 1u))) { pub = j + 2u; ctl[li] = pub; }
            }
            WAIT_VM(0);
        }
        WAIT_VM(0); ctl[li] = NSTEP;
    } else if (wave == 8) {
        unsigned tk = 0;
        if (lane == 0) tk = draw_ticket(myws, b);
        const unsigned call = rfl(tk) + 1u;
        WAIT_VM(0);
        if (lane == 0) ctl[12] = call;
        const unsigned fsrc = lane < 8 ? a0 : (lane < 16 ? a1 : og + (c & 3u));
        const char* fp = lane < 24 ? myws + OFF_FLAG + (fsrc * 8u + (lane & 7)) * FSTR : myws + OFF_TMO;
        const rsrc_t msrd = make_rsrc(P.out[rank], OUT_BYTES);
        const u32x4 mark = {0x80808080u, 0x80808080u, 0x80808080u, 0x80808080u};
        unsigned nmark = 16u - 8u * nh;
        auto put_mark = [&]() {
            bst16_sys(mark, ((unsigned)lane >> 3) * 512u + ((unsigned)lane & 7u) * 16u, msrd, (((nmark & 8u) ? hr0 : hr1) + (nmark & 7u) * 8u) * 512u + c * 128u);
            ++nmark;
        };
        const unsigned long long t0 = now();
        for (;;) {
            if (nmark < 16u && (unsigned)(now() - t0) > MARK_T0 + (nmark * 16u + (gi & 15u)) * MARK_DELTA) put_mark();
            if (nmark == 16u) { WAIT_VM(0); if (lane == 0) ctl[13] = call; nmark = 17u; }
            const unsigned v = ld_sys(fp);
            const int d = (int)(v - call * 64u);
            const unsigned prog = d < 0 ? 0u : (d > NSTEP ? (unsigned)NSTEP : (unsigned)d);
            auto min8 = [&](int l) { const unsigned m0 = min4(prog, l), m1 = min4(prog, l + 4); return m0 < m1 ? m0 : m1; };
            unsigned av0 = own0 ? (unsigned)NSTEP : min8(0), av1 = own1 ? (unsigned)NSTEP : min8(8), avb = c < 4u ? (unsigned)NSTEP : min8(16);
            const bool dead = __builtin_amdgcn_readlane(v, 24) == call || now() - t0 > DEADLINE;
            if (dead) { if (lane == 0) { st_agent(myws + OFF_TMO, call); ctl[11] = 1u; } av0 = av1 = avb = NSTEP; }
            if (lane == 0) { ctl[8] = av0; ctl[9] = av1; ctl[10] = avb; }
            if (av0 == (unsigned)NSTEP && av1 == (unsigned)NSTEP && avb == (unsigned)NSTEP) break;
            const bool lazy = min(min(av0, av1), avb) >= (unsigned)(NSTEP - NSPEC) && nmark > 16u;
            bool done = false;
            for (int i = lazy ? 40 : 1; i > 0 && !done; --i) {
                const u32x4 fl = *(volatile LAS u32x4*)(L + L_FULL);
                done = rfl(fl.x + fl.y + fl.z + fl.w) == 4u * NSTEP;
                __builtin_amdgcn_s_sleep(1);
            }
            if (done) break;
        }
        while (nmark < 16u) {
            if ((unsigned)(now() - t0) > MARK_T0 + (nmark * 16u + (gi & 15u)) * MARK_DELTA || rfl(ctl[4]) > 48u) put_mark(); else __builtin_amdgcn_s_sleep(8);
        }
        if (nmark == 16u) { WAIT_VM(0); if (lane == 0) ctl[13] = call; }
    }

    if (wave >= 8) return;
    if (wave < 4) {
        for (;;) {
            const u32x4 f = *(volatile LAS u32x4*)(L + L_FREE);
            const unsigned m0 = f.x < f.y ? f.x : f.y, m1 = f.z < f.w ? f.z : f.w;
            if (rfl(m0 < m1 ? m0 : m1) >= (unsigned)NSTEP - 2u) break;
            __builtin_amdgcn_s_sleep(2);
        }
        asm volatile("" ::: "memory");
        const unsigned wm = (unsigned)wave >> 1, wn = (unsigned)wave & 1u, h = (unsigned)lane >> 5;
#pragma unroll
        for (int nt = 0; nt < 2; ++nt)
#pragma unroll
            for (int mt = 0; mt < 2; ++mt)
#pragma unroll
                for (int g = 0; g < 4; ++g) {
                    const unsigned m = wm * 64u + mt * 32u + (lane & 31u), ch = 16u * wn + 8u * nt + 2u * g + h;
                    f32x4 q; q.x = acc[nt][mt][4 * g]; q.y = acc[nt][mt][4 * g + 1]; q.z = acc[nt][mt][4 * g + 2]; q.w = acc[nt][mt][4 * g + 3];
                    *(LAS f32x4*)(L + m * 512u + ((ch ^ (m & 7u)) << 4)) = q;
                }
    }
    const unsigned orank = (c < 4u ? rank : og + (c & 3u)) & 7u;
    const unsigned ocol = (c < 4u ? c : me) * 128u;
    char* optr = P.out[orank];
    asm volatile("" : "+s"(optr));
    unsigned call = 0, mx = 0;
    const char* gp = myws + OFF_AMAX + ((unsigned)tid & 255u) * GSTR;
    unsigned skip = 0;
#pragma unroll
    for (int j = 0; j < 5; ++j) { const unsigned g = j * 256u + ((unsigned)tid & 255u); skip |= (g >= 8u * 132u || g == gidx) ? 1u << j : 0u; }
    unsigned long long tg0 = 0;
    bool ok = false;
    auto sweep = [&]() {
        const unsigned tm = ld_agent(myws + OFF_TMO);
        if (!ok) {
            unsigned long long g[5];
#pragma unroll
            for (int j = 0; j < 4; ++j) g[j] = __hip_atomic_load((gu64*)(gp + j * 256 * GSTR), __ATOMIC_RELAXED, __HIP_MEMORY_SCOPE_SYSTEM);
            g[4] = 0;
            if (tid < 256 + 32) g[4] = __hip_atomic_load((gu64*)(gp + 4 * 256 * GSTR), __ATOMIC_RELAXED, __HIP_MEMORY_SCOPE_SYSTEM);
            ok = true; mx = 0;
#pragma unroll
            for (int j = 0; j < 5; ++j) {
                const bool mine = (skip >> j) & 1u;
                ok = ok && (mine || (unsigned)(g[j] >> 32) >= call);
                mx = max(mx, mine ? 0u : (unsigned)g[j]);
            }
        }
        if (rfl(tm) == call || now() - tg0 > DEADLINE) { if (lane == 0) { st_agent(myws + OFF_TMO, call); ctl[11] = 1u; } return true; }
        return (bool)__all(ok);
    };
    if (wave >= 4) {
        while ((call = rfl(ctl[12])) == 0u) __builtin_amdgcn_s_sleep(2);
        gp += (call & 1u) * AMAX_BYTES;
    }
    BARRIER();
    f32x4 y[2][4];
    const unsigned r0 = (unsigned)tid >> 3, q8 = (unsigned)tid & 7u;
#pragma unroll
    for (int hf = 0; hf < 2; ++hf)
#pragma unroll
        for (int e = 0; e < 4; ++e) y[hf][e] = *(LAS f32x4*)(L + (r0 + 64u * hf) * 512u + (((4u * q8 + e) ^ (r0 & 7u)) << 4));
    if (wave >= 4) {
        tg0 = now();
        while (!sweep()) { }
        mx = wave_max_u32(mx);
        if (lane == 0) ctl[20 + wave] = mx;
    }
    BARRIER();
    const unsigned long long tq = now();
    const u32x4 gmx = *(volatile LAS u32x4*)(L + L_GMAX);
    const unsigned ga = max(max(gmx.x, gmx.y), max(gmx.z, gmx.w)), gb = ctl[16];
    const bool aborted = ctl[11] != 0u;
    const float amax = __uint_as_float(ga > gb ? ga : gb);
    const float inv = amax > 0.f ? 127.0f / amax : 0.f;
    const rsrc_t osrd = make_rsrc(optr, OUT_BYTES);
    const unsigned ovoff = r0 * 512u + q8 * 16u;
    constexpr float MAGIC = 12582912.0f;
#pragma unroll
    for (int hf = 0; hf < 2; ++hf) {
        u32x4 o;
#pragma unroll
        for (int e = 0; e < 4; ++e) {
            unsigned z[4];
#pragma unroll
            for (int i = 0; i < 4; ++i)
                z[i] = __float_as_uint(__builtin_amdgcn_fmed3f(__builtin_fmaf(y[hf][e][i], inv, MAGIC), MAGIC - 127.0f, MAGIC + 127.0f));
            const unsigned lo = __builtin_amdgcn_perm(z[1], z[0], 0x0c0c0400u), hi = __builtin_amdgcn_perm(z[3], z[2], 0x04000c0cu);
            o[e] = aborted ? 0x80808080u : (lo | hi);
        }
        const unsigned row = hf ? a1 * 512u + lu1 * 64u : a0 * 512u + lu0 * 64u;
        if (hf && half) break;
        if (c < 4u) __builtin_amdgcn_raw_buffer_store_b128(o, osrd, ovoff, row * 512u + ocol, 16);
        else {
            const unsigned due = ((unsigned)(hf * 8 + wave) * 15u + t) * OUT_DELTA_NS / 160u;
            asm volatile("" : "+v"(o));
            while ((unsigned)(now() - tq) < due) __builtin_amdgcn_s_sleep(1);
            bst16_sys(o, ovoff, osrd, row * 512u + ocol);
        }
    }
    if (nh > 0u) {
        call = rfl(ctl[12]);
        const unsigned due = ((unsigned)((nh > 1u ? 8 : 0) + wave) * 15u + t) * OUT_DELTA_NS / 160u + OUT_CHECK_DELAY;
        while ((unsigned)(now() - tq) < due) __builtin_amdgcn_s_sleep(2);
        bool in = false;
        for (;;) {
            const unsigned tm = ld_agent(myws + OFF_TMO);
            if (!in) {
                const u32x4 h0 = __builtin_amdgcn_raw_buffer_load_b128(osrd, ovoff, hr0 * 512u + ocol, 17);
                const u32x4 h1 = __builtin_amdgcn_raw_buffer_load_b128(osrd, ovoff, hr1 * 512u + ocol, 17);
                unsigned bad = 0;
#pragma unroll
                for (int e = 0; e < 4; ++e) {
                    const unsigned x0 = h0[e] ^ 0x80808080u, x1 = h1[e] ^ 0x80808080u;
                    bad |= ((x0 - 0x01010101u) & ~x0) | ((x1 - 0x01010101u) & ~x1);
                }
                in = (bad & 0x80808080u) == 0u;
            }
            if (__all(in)) break;
            if (rfl(tm) == call || now() - tq > DEADLINE) { if (lane == 0) st_agent(myws + OFF_TMO, call); break; }
            asm volatile("" ::: "memory");
        }
    }
}

extern "C" void kernel_launch(int rank, int nranks, ncclComm_t comm,
                              hipStream_t stream,
                              void* const* d_in, const int* in_sizes, int n_in,
                              void* d_out, int out_size,
                              const int* in_dtypes, int out_dtype,
                              void* const* d_out_all,
                              void* const* d_ws_all, size_t ws_bytes) {
    (void)comm; (void)in_dtypes; (void)out_dtype; (void)d_out;
    if (nranks != NR || (unsigned)rank >= (unsigned)NR || n_in < 2 || ws_bytes < WS_NEEDED ||
        in_sizes[0] != 512 * 4096 || in_sizes[1] != 4096 * 512 || out_size != (int)OUT_BYTES) {
        fprintf(stderr, "kernel_launch: unexpected problem shape\n");
        return;
    }
    int dev = 0, cus = 0;
    HIP_CHECK(hipGetDevice(&dev));
    HIP_CHECK(hipDeviceGetAttribute(&cus, hipDeviceAttributeMultiprocessorCount, dev));
    if (cus < GRID) { fprintf(stderr, "kernel_launch: needs %d CUs\n", GRID); return; }
    Params P;
    P.x = (const char*)d_in[0];
    P.w = (const char*)d_in[1];
    for (int p = 0; p < NR; ++p) { P.ws[p] = (char*)d_ws_all[p]; P.out[p] = (char*)d_out_all[p]; }
    for (unsigned l = 0; l < (unsigned)NR; ++l) {
        const unsigned dr = link_rank((unsigned)rank, l < 7u ? l : 0u);
        P.link[l].x = P.x;
        P.link[l].wq = P.w + (dr & 3u) * 256u;
        P.link[l].wq_bytes = SHARD_BYTES - (dr & 3u) * 256u;
        P.link[l].dst = P.ws[dr] + OFF_ST + (unsigned)rank * ST_BYTES;
    }
    P.rank = rank;
    hipLaunchKernelGGL(ag_gemm_q8, dim3(GRID), dim3(NTHREADS), 0, stream, P);
    HIP_CHECK(hipGetLastError());
}
```
